```python
import numpy as np

kernel_name = "gpu_gemm_rs_m4096_k4096_n16384_e5m2e5m2_dqrelu"


ABSMAX_RTOL = 0.05

COMPUTE_DTYPE = "float32"
EPILOGUE = "dequant_relu"
IN_QUANT = None
OUT_QUANT = None
IN_FP8 = "e5m2e5m2"
IN_DTYPES = {"x": "e5m2", "w_mat": "e5m2",
             "scale_x": "float32", "scale_w": "float32"}
WORLD_SIZE = 8

def _snap_fp8(x, mant_bits, max_abs, min_normal):
    a = np.abs(x.astype(np.float64))
    _m, _e = np.frexp(a)
    q = np.where(a >= min_normal, np.ldexp(1.0, _e - 1 - mant_bits),
                 min_normal * 2.0 ** -mant_bits)
    snapped = np.minimum(np.round(a / q) * q, max_abs)
    return (np.sign(x) * snapped).astype(np.float32)

def setup_inputs():

    x = _snap_fp8(np.random.randn(4096, 4096).astype(np.float32), 2, 57344.0, 2.0 ** -14)
    w_mat = _snap_fp8(np.random.randn(4096, 16384).astype(np.float32), 2, 57344.0, 2.0 ** -14)
    scale_x = (0.02 * (1.0 + np.random.rand(1))).astype(np.float32)
    scale_w = (0.02 * (1.0 + np.random.rand(1))).astype(np.float32)
    return {"x": x, "w_mat": w_mat, "scale_x": scale_x, "scale_w": scale_w}

def reference(x, w_mat, scale_x, scale_w):
    acc = (x @ w_mat).astype(np.float32)
    y = (acc * (scale_x * scale_w)).astype(np.float32)
    return np.maximum(y, 0.0).astype(np.float32)
```

```cpp
#include <hip/hip_runtime.h>
#include <rccl/rccl.h>
#include <cstdio>
#include <cstdint>

#define HIP_CHECK(expr) do { hipError_t _e = (expr); if (_e != hipSuccess) { \
    fprintf(stderr, "HIP error %s at %s:%d\n", hipGetErrorString(_e), __FILE__, __LINE__); \
    return; } } while (0)

namespace {

typedef unsigned u32x4 __attribute__((ext_vector_type(4)));
typedef int i32x8 __attribute__((ext_vector_type(8)));
typedef float f32x16 __attribute__((ext_vector_type(16)));
typedef float f32x4 __attribute__((ext_vector_type(4)));

constexpr int NR = 8;
constexpr int NTOT = 16384;
constexpr int KS = 512;
constexpr int NS = 2048;
constexpr unsigned UNIT = 16384;

constexpr unsigned OFF_CTR = 0;
constexpr unsigned OFF_TMO = 4096;
constexpr unsigned OFF_XF = 8192;
constexpr unsigned OFF_WF = 16384;
constexpr unsigned OFF_OF = 24576;
constexpr unsigned OFF_PD = 28672;
constexpr unsigned OFF_DIAG = 65536;
constexpr unsigned OFF_DATA = 1u << 20;
constexpr unsigned X_BYTES = 16u << 20;
constexpr unsigned W_BYTES = 8u << 20;
constexpr unsigned PAR_BYTES = X_BYTES + W_BYTES;
constexpr size_t WS_NEED = (size_t)OFF_DATA + 2 * (size_t)PAR_BYTES;

constexpr int NWG = 256;
constexpr int NTHREADS = 512;
#define PUSH_WAVES_PER_WG 4
#define GATE_IT 5
constexpr int PW_PER_WG = PUSH_WAVES_PER_WG;
constexpr int N_PUSH_WG = 256 / PW_PER_WG;
constexpr unsigned STAGE_BYTES = 3 * UNIT;
constexpr unsigned EPI_PITCH = 528;
constexpr unsigned LDS_MAIN = 3 * STAGE_BYTES;
constexpr unsigned LDS_BYTES = LDS_MAIN + 64;
constexpr unsigned long long DEADLINE_TICKS = 40000000ull;
constexpr int AUX_SYS = 17;

struct Params {
  const uint8_t* x;
  const uint8_t* w;
  const float* sx;
  const float* sw;
  float* out[NR];
  uint8_t* ws[NR];
  int rank;
};

__device__ __forceinline__ __amdgpu_buffer_rsrc_t mk_rsrc(const void* p, unsigned bytes) {
  return __builtin_amdgcn_make_buffer_rsrc((void*)p, 0, bytes, 0x00020000);
}
__device__ __forceinline__ unsigned ld_sys(const unsigned* p) {
  return __hip_atomic_load(p, __ATOMIC_RELAXED, __HIP_MEMORY_SCOPE_SYSTEM);
}
__device__ __forceinline__ void st_sys(unsigned* p, unsigned v) {
  __hip_atomic_store(p, v, __ATOMIC_RELAXED, __HIP_MEMORY_SCOPE_SYSTEM);
}
__device__ __forceinline__ unsigned ld_agent(const unsigned* p) {
  return __hip_atomic_load(p, __ATOMIC_RELAXED, __HIP_MEMORY_SCOPE_AGENT);
}
__device__ __forceinline__ void st_agent(unsigned* p, unsigned v) {
  __hip_atomic_store(p, v, __ATOMIC_RELAXED, __HIP_MEMORY_SCOPE_AGENT);
}
#define WAIT_VM(n) asm volatile("s_waitcnt vmcnt(" #n ")" ::: "memory")
#define WAIT_LGKM0() asm volatile("s_waitcnt lgkmcnt(0)" ::: "memory")
#define RAW_BARRIER() do { WAIT_LGKM0(); __builtin_amdgcn_s_barrier(); } while (0)

__device__ __forceinline__ bool wave_wait_flags(const unsigned* flag, bool active, unsigned target, unsigned* tmo, unsigned call) {
  const unsigned long long t0 = __builtin_amdgcn_s_memrealtime();
  for (;;) {
    const unsigned v = active ? ld_sys(flag) : target;
    if (__all(v >= target)) return true;
    if (ld_agent(tmo) == call || __builtin_amdgcn_s_memrealtime() - t0 > DEADLINE_TICKS) {
      st_agent(tmo, call);
      return false;
    }
    __builtin_amdgcn_s_sleep(2);
  }
}

#define STAMP(slot) do {} while (0)
#define STAMPW(slot) do {} while (0)

__device__ __forceinline__ void load_x_unit(u32x4 (&v)[16], const Params& p, int mu, int kc, int lane) {
  const int mt = mu >> 1;
  const int row0 = 512 * (mt & 7) + 256 * (mt >> 3) + 128 * (mu & 1);
  const auto src = mk_rsrc(p.x, 4096u * KS);
  const unsigned base = (unsigned)(row0 + (lane & 31)) * KS + kc * 128 + (lane >> 5) * 16;
#pragma unroll
  for (int i = 0; i < 16; ++i) {
    const int rb = i >> 2, kb = (i >> 1) & 1, h = i & 1;
    v[i] = __builtin_amdgcn_raw_buffer_load_b128(src, base + rb * 32 * KS + kb * 64 + h * 32, 0, 0);
  }
}
__device__ __forceinline__ void load_w_raw(u32x4 (&r)[16], const Params& p, int dest, int nt, int kc, int lane) {
  const auto src = mk_rsrc(p.w, (unsigned)KS * NTOT);
  const unsigned base = (unsigned)(kc * 128 + (lane >> 3) * 16) * NTOT + dest * NS + nt * 128 + (lane & 7) * 16;
#pragma unroll
  for (int i = 0; i < 16; ++i) r[i] = __builtin_amdgcn_raw_buffer_load_b128(src, base + i * NTOT, 0, 0);
}
__device__ __forceinline__ void transpose16(const u32x4 (&r)[16], u32x4 (&o)[16]) {
#pragma unroll
  for (int d = 0; d < 4; ++d) {
#pragma unroll
    for (int e = 0; e < 4; ++e) {
      const unsigned r0 = r[4 * e][d], r1 = r[4 * e + 1][d], r2 = r[4 * e + 2][d], r3 = r[4 * e + 3][d];
      const unsigned t0 = __builtin_amdgcn_perm(r1, r0, 0x05010400u);
      const unsigned t1 = __builtin_amdgcn_perm(r1, r0, 0x07030602u);
      const unsigned t2 = __builtin_amdgcn_perm(r3, r2, 0x05010400u);
      const unsigned t3 = __builtin_amdgcn_perm(r3, r2, 0x07030602u);
      o[4 * d + 0][e] = __builtin_amdgcn_perm(t2, t0, 0x05040100u);
      o[4 * d + 1][e] = __builtin_amdgcn_perm(t2, t0, 0x07060302u);
      o[4 * d + 2][e] = __builtin_amdgcn_perm(t3, t1, 0x05040100u);
      o[4 * d + 3][e] = __builtin_amdgcn_perm(t3, t1, 0x07060302u);
    }
  }
}

__device__ __forceinline__ void push_wave(const Params& p, int pw, unsigned call, int lane) {
  const unsigned dest = (unsigned)(p.rank + (pw & 7)) & 7u;
  const int slot = pw >> 3;
  uint8_t* dws = p.ws[dest];
  const auto dst = mk_rsrc(dws + OFF_DATA + (call & 1u) * PAR_BYTES, PAR_BYTES);
  unsigned* xf = (unsigned*)(dws + OFF_XF);
  unsigned* wf = (unsigned*)(dws + OFF_WF);
#pragma unroll 1
  for (int it = 0; it < 6; ++it) {
    u32x4 v[16];
    unsigned off;
    unsigned* flag;
    bool is_w = false;
    int kc, mu = 0;
    if (it < 4) { kc = it; is_w = slot >= 16; mu = slot; }
    else { const int u = slot + 32 * (it - 4); kc = u >> 4; mu = 16 + (u & 15); }
    if (is_w) {
      const int nt = slot - 16;
      u32x4 r[16];
      load_w_raw(r, p, (int)dest, nt, kc, lane);
      transpose16(r, v);
      off = X_BYTES + (unsigned)((p.rank * 4 + kc) * 16 + nt) * UNIT;
      flag = wf + (nt * 4 + kc) * 8 + p.rank;
#pragma unroll
      for (int j = 0; j < 16; ++j)
        __builtin_amdgcn_raw_buffer_store_b128(v[j], dst, off + j * 1024 + ((lane ^ (((j >> 1) & 1) * 8)) * 16), 0, AUX_SYS);
    } else {
      load_x_unit(v, p, mu, kc, lane);
      off = (unsigned)((p.rank * 4 + kc) * 32 + mu) * UNIT;
      flag = xf + (((mu >> 1) * 4 + kc) * 8 + p.rank) * 2 + (mu & 1);
#pragma unroll
      for (int j = 0; j < 16; ++j)
        __builtin_amdgcn_raw_buffer_store_b128(v[j], dst, off + j * 1024 + lane * 16, 0, AUX_SYS);
    }
    if (it == GATE_IT && lane == 0)
      __hip_atomic_fetch_add((unsigned*)(p.ws[p.rank] + OFF_PD) + dest * 32, 1u, __ATOMIC_RELAXED, __HIP_MEMORY_SCOPE_AGENT);
    WAIT_VM(0);
    if (lane == 0) st_sys(flag, call);
    STAMPW(it);
  }
}

__global__ void __launch_bounds__(NTHREADS, 2) gemm_rs_kernel(Params p) {
  __shared__ __attribute__((aligned(16))) uint8_t smem[LDS_BYTES];
  const int tid = threadIdx.x;
  const int lane = tid & 63;
  const int wave = __builtin_amdgcn_readfirstlane(tid >> 6);
  const int b = blockIdx.x;
  uint8_t* myws = p.ws[p.rank];
  unsigned* tmo = (unsigned*)(myws + OFF_TMO);
  volatile unsigned* lds_word = (volatile unsigned*)(smem + LDS_MAIN);

  STAMP(0);
  unsigned* ctr = (unsigned*)(myws + OFF_CTR) + b;
  const unsigned call = __builtin_amdgcn_readfirstlane(ld_agent(ctr)) + 1u;

  if (b < N_PUSH_WG && wave < PW_PER_WG) push_wave(p, b * PW_PER_WG + wave, call, lane);
  STAMP(1);

  const int idx = b & 127, xcd = idx & 7, ii = idx >> 3;
  const int mt = ((b < 128) ? 8 : 0) + 4 * (xcd & 1) + (ii & 3);
  const int nt = 4 * (xcd >> 1) + (ii >> 2);
  const int wm = wave & 3, wn = wave >> 2;

  const uint8_t* data = myws + OFF_DATA + (call & 1u) * PAR_BYTES;
  const unsigned xa_off = (unsigned)((wm >> 1) * UNIT + (wm & 1) * 2 * 4096 + lane * 16);
  const unsigned wb_off = 2 * UNIT + (unsigned)(wn * 2 * 4096 + ((lane & 31) >> 3) * 1024 +
                                                ((((lane >> 5) ^ (lane >> 4)) & 1) * 8 + (lane & 7)) * 16);
  f32x16 acc[2][2];
#pragma unroll
  for (int i = 0; i < 2; ++i)
#pragma unroll
    for (int j = 0; j < 2; ++j)
#pragma unroll
      for (int r = 0; r < 16; ++r) acc[i][j][r] = 0.f;

  bool ok = true;
#pragma unroll 1
  for (int kc = 0; kc < 4; ++kc) {
    if (wave == 0) {
      const unsigned* f = lane < 16 ? (const unsigned*)(myws + OFF_XF) + (mt * 4 + kc) * 16 + lane
                                    : (const unsigned*)(myws + OFF_WF) + (nt * 4 + kc) * 8 + (lane & 7);
      const bool good = wave_wait_flags(f, lane < 24, call, tmo, call);
      __builtin_amdgcn_fence(__ATOMIC_ACQUIRE, "");
      WAIT_VM(0);
      if (lane == 0) *lds_word = good ? 1u : 0u;
    }
    RAW_BARRIER();
    ok = (*lds_word != 0u);
    if (!__builtin_amdgcn_readfirstlane(ok)) break;
    STAMP(2 + kc);

    auto stage = [&](int s) {
      const int q = (p.rank + s) & 7;
      const uint8_t* xs = data + (unsigned)((q * 4 + kc) * 32 + 2 * mt) * UNIT;
      const uint8_t* wsrc = data + X_BYTES + (unsigned)((q * 4 + kc) * 16 + nt) * UNIT;
      uint8_t* l = smem + (s % 3) * STAGE_BYTES;
#pragma unroll
      for (int j = 0; j < 4; ++j)
        __builtin_amdgcn_global_load_lds((const unsigned*)(xs + (wave * 4 + j) * 1024 + lane * 16),
                                         (__attribute__((address_space(3))) unsigned*)(l + (wave * 4 + j) * 1024), 16, 0, 0);
#pragma unroll
      for (int j = 0; j < 2; ++j)
        __builtin_amdgcn_global_load_lds((const unsigned*)(wsrc + (wave * 2 + j) * 1024 + lane * 16),
                                         (__attribute__((address_space(3))) unsigned*)(l + 2 * UNIT + (wave * 2 + j) * 1024), 16, 0, 0);
    };
    stage(0);
    stage(1);
#pragma unroll
    for (int s = 0; s < 8; ++s) {
      if (s < 7) WAIT_VM(6); else WAIT_VM(0);
      RAW_BARRIER();
      if (s + 2 < 8) stage(s + 2);
      const uint8_t* l = smem + (s % 3) * STAGE_BYTES;
#pragma unroll
      for (int kb = 0; kb < 2; ++kb) {
        i32x8 xa[2], wb[2];
#pragma unroll
        for (int i = 0; i < 2; ++i) {
          const u32x4 lo = *(const u32x4*)(l + xa_off + i * 4096 + kb * 2048);
          const u32x4 hi = *(const u32x4*)(l + xa_off + i * 4096 + kb * 2048 + 1024);
          xa[i] = i32x8{(int)lo[0], (int)lo[1], (int)lo[2], (int)lo[3], (int)hi[0], (int)hi[1], (int)hi[2], (int)hi[3]};
          const u32x4 wlo = *(const u32x4*)(l + wb_off + i * 4096 + kb * 512);
          const u32x4 whi = *(const u32x4*)(l + wb_off + i * 4096 + kb * 512 + 256);
          wb[i] = i32x8{(int)wlo[0], (int)wlo[1], (int)wlo[2], (int)wlo[3], (int)whi[0], (int)whi[1], (int)whi[2], (int)whi[3]};
        }
#pragma unroll
        for (int i = 0; i < 2; ++i)
#pragma unroll
          for (int j = 0; j < 2; ++j)
            acc[i][j] = __builtin_amdgcn_mfma_scale_f32_32x32x64_f8f6f4(wb[i], xa[j], acc[i][j], 1, 1, 0, 0x7F7F7F7F, 0, 0x7F7F7F7F);
      }
    }
    RAW_BARRIER();
  }
  STAMP(6);

  const bool bad = !ok || (ld_agent(tmo) == call);
  const float scale = p.sx[0] * p.sw[0];
  {
    const float nanv = __builtin_nanf("");
#pragma unroll
    for (int i = 0; i < 2; ++i)
#pragma unroll
      for (int j = 0; j < 2; ++j) {
        const int row = wm * 64 + j * 32 + (lane & 31);
#pragma unroll
        for (int r = 0; r < 4; ++r) {
          f32x4 v;
#pragma unroll
          for (int t = 0; t < 4; ++t) {
            const float y = acc[i][j][r + 4 * t] * scale;
            v[t] = bad ? nanv : ((y <= 0.f) ? 0.f : y);
          }
          const int col = 16 * (r + 4 * (lane >> 5)) + 4 * (wn * 2 + i);
          *(f32x4*)(smem + row * EPI_PITCH + col * 4) = v;
        }
      }
  }
  const unsigned dest = (unsigned)(mt & 7);
  if (wave == 0 && dest != (unsigned)p.rank)
    (void)wave_wait_flags((const unsigned*)(myws + OFF_PD) + dest * 32, lane == 0, call * 32u, tmo, call);
  RAW_BARRIER();
  STAMP(9);
  {
    const auto dst = mk_rsrc(p.out[dest], 512u * NTOT * 4u);
    const unsigned obase = (unsigned)((mt >> 3) * 256) * (NTOT * 4u) + (unsigned)(p.rank * NS + nt * 128) * 4u + (lane & 31) * 16;
#define OUT_WAVES 1
#define OUT_VM 2
#define STR2(x) #x
#define STR(x) STR2(x)
    if (wave < OUT_WAVES) {
#pragma unroll 4
      for (int it = 0; it < 128 / OUT_WAVES; ++it) {
        const int row = (it * OUT_WAVES + wave) * 2 + (lane >> 5);
        const u32x4 v = *(const u32x4*)(smem + row * EPI_PITCH + (lane & 31) * 16);
        __builtin_amdgcn_raw_buffer_store_b128(v, dst, obase + (unsigned)row * (NTOT * 4u), 0, AUX_SYS);
        asm volatile("s_waitcnt vmcnt(" STR(OUT_VM) ")" ::: "memory");
      }
    }
  }
  WAIT_VM(0);
  RAW_BARRIER();
  if (tid == 0) {
    st_sys((unsigned*)(p.ws[dest] + OFF_OF) + p.rank * 32 + (mt >> 3) * 16 + nt, call);
    st_agent(ctr, call);
  }
  STAMP(7);

  if (b == NWG - 1 && wave < 4) {
    const int src = tid >> 5;
    const bool good = wave_wait_flags((const unsigned*)(myws + OFF_OF) + tid, src != p.rank, call, tmo, call);
    __builtin_amdgcn_fence(__ATOMIC_ACQUIRE, "");
    if (!good && lane == 0) {
      p.out[p.rank][p.rank * NS] = __builtin_nanf("");
    }
    STAMP(8);
  }
}

}

extern "C" void kernel_launch(int rank, int nranks, ncclComm_t comm,
                              hipStream_t stream,
                              void* const* d_in, const int* in_sizes, int n_in,
                              void* d_out, int out_size,
                              const int* in_dtypes, int out_dtype,
                              void* const* d_out_all,
                              void* const* d_ws_all, size_t ws_bytes) {
  (void)comm; (void)in_sizes; (void)n_in; (void)d_out; (void)out_size; (void)in_dtypes; (void)out_dtype;
  if (nranks != NR || (unsigned)rank >= (unsigned)NR || ws_bytes < WS_NEED) {
    fprintf(stderr, "kernel_launch: unsupported configuration\n");
    return;
  }
  Params p;
  p.x = (const uint8_t*)d_in[0];
  p.w = (const uint8_t*)d_in[1];
  p.sx = (const float*)d_in[2];
  p.sw = (const float*)d_in[3];
  for (int i = 0; i < NR; ++i) {
    p.out[i] = (float*)d_out_all[i];
    p.ws[i] = (uint8_t*)d_ws_all[i];
  }
  p.rank = rank;
  hipLaunchKernelGGL(gemm_rs_kernel, dim3(NWG), dim3(NTHREADS), 0, stream, p);
  HIP_CHECK(hipGetLastError());
}
```

```cpp
#include <hip/hip_runtime.h>
#include <rccl/rccl.h>
#include <cstdio>
#include <cstdint>

#define HIP_CHECK(expr) do { hipError_t _e = (expr); if (_e != hipSuccess) { \
    fprintf(stderr, "HIP error %s at %s:%d\n", hipGetErrorString(_e), __FILE__, __LINE__); \
    return; } } while (0)

namespace {

typedef unsigned u32x4 __attribute__((ext_vector_type(4)));
typedef int i32x8 __attribute__((ext_vector_type(8)));
typedef int i32x4 __attribute__((ext_vector_type(4)));
typedef float f32x16 __attribute__((ext_vector_type(16)));
typedef float f32x4 __attribute__((ext_vector_type(4)));
typedef __attribute__((address_space(3))) unsigned lds_u32;

constexpr int NR = 8;
constexpr int NTOT = 16384;
constexpr int KS = 512;
constexpr int NS = 2048;
constexpr unsigned UNIT = 16384;

constexpr unsigned OFF_CTR = 0;
constexpr unsigned OFF_TMO = 4096;
constexpr unsigned OFF_XF = 8192;
constexpr unsigned OFF_WF = 16384;
constexpr unsigned OFF_OF = 24576;
constexpr unsigned OFF_PD = 28672;
constexpr unsigned OFF_SF = 32768;
constexpr unsigned OFF_DIAG = 65536;
constexpr unsigned OFF_DATA = 1u << 20;
constexpr unsigned X_BYTES = 16u << 20;
constexpr unsigned W_BYTES = 8u << 20;
constexpr unsigned PAR_BYTES = X_BYTES + W_BYTES;
constexpr unsigned SEG_BYTES = 32 * 528;
constexpr unsigned TILE_BYTES = 8 * SEG_BYTES;
constexpr unsigned OFF_STG = OFF_DATA + 2 * PAR_BYTES;
constexpr unsigned STG_BYTES = 8 * 32 * TILE_BYTES;
constexpr size_t WS_NEED = (size_t)OFF_STG + STG_BYTES;

constexpr int NWG = 256;
constexpr int NTHREADS = 512;
#define PUSH_WAVES_PER_WG 4
#define GATE_N 16
#define GATE_IT 3
constexpr int PW_PER_WG = PUSH_WAVES_PER_WG;
constexpr int N_PUSH_WG = 256 / PW_PER_WG;
constexpr unsigned STAGE_BYTES = 3 * UNIT;
constexpr unsigned EPI_PITCH = 528;
constexpr unsigned LDS_MAIN = 3 * STAGE_BYTES;
constexpr unsigned LDS_BYTES = LDS_MAIN + 64 + 256;
constexpr unsigned long long DEADLINE_TICKS = 40000000ull;
#define AUX_OUT 16
constexpr int AUX_SYS = 17;

struct Params {
  const uint8_t* x;
  const uint8_t* w;
  const float* sx;
  const float* sw;
  float* out[NR];
  uint8_t* ws[NR];
  int rank;
};

__device__ __forceinline__ __amdgpu_buffer_rsrc_t mk_rsrc(const void* p, unsigned bytes) {
  return __builtin_amdgcn_make_buffer_rsrc((void*)p, 0, bytes, 0x00020000);
}
__device__ __forceinline__ unsigned ld_sys(const unsigned* p) {
  return __hip_atomic_load(p, __ATOMIC_RELAXED, __HIP_MEMORY_SCOPE_SYSTEM);
}
__device__ __forceinline__ void st_sys(unsigned* p, unsigned v) {
  __hip_atomic_store(p, v, __ATOMIC_RELAXED, __HIP_MEMORY_SCOPE_SYSTEM);
}
__device__ __forceinline__ unsigned ld_agent(const unsigned* p) {
  return __hip_atomic_load(p, __ATOMIC_RELAXED, __HIP_MEMORY_SCOPE_AGENT);
}
__device__ __forceinline__ void st_agent(unsigned* p, unsigned v) {
  __hip_atomic_store(p, v, __ATOMIC_RELAXED, __HIP_MEMORY_SCOPE_AGENT);
}
#define STR2(x) #x
#define STR(x) STR2(x)
#define PUSH_VM 3
#define PUSH_THROTTLE() asm volatile("s_waitcnt vmcnt(" STR(PUSH_VM) ")" ::: "memory")
#define WAIT_VM(n) asm volatile("s_waitcnt vmcnt(" #n ")" ::: "memory")
#define WAIT_LGKM0() asm volatile("s_waitcnt lgkmcnt(0)" ::: "memory")
#define RAW_BARRIER() do { WAIT_LGKM0(); __builtin_amdgcn_s_barrier(); } while (0)

__device__ __forceinline__ bool wave_wait_flags(const unsigned* flag, bool active, unsigned target, unsigned* tmo, unsigned call) {
  const unsigned long long t0 = __builtin_amdgcn_s_memrealtime();
  for (unsigned spin = 1;; ++spin) {
    const unsigned v = active ? ld_sys(flag) : target;
    if (__all(v >= target)) return true;
    if ((spin & 7u) == 0u &&
        (ld_agent(tmo) == call || __builtin_amdgcn_s_memrealtime() - t0 > DEADLINE_TICKS)) {
      st_agent(tmo, call);
      return false;
    }
    __builtin_amdgcn_s_sleep(2);
  }
}

#define STAMP(slot) do {} while (0)
#define STAMPX(slot) do {} while (0)
#define STAMPW(slot) do {} while (0)

template <int I0, int N>
__device__ __forceinline__ void load_x_unit(u32x4 (&v)[N], const Params& p, int mu, int kc, int lane, int i0) {
  const int mt = mu >> 1;
  const int row0 = 512 * (mt & 7) + 256 * (mt >> 3) + 128 * (mu & 1);
  const auto src = mk_rsrc(p.x, 4096u * KS);
  const unsigned base = (unsigned)(row0 + (i0 >> 2) * 32 + (lane & 31)) * KS + kc * 128 + (lane >> 5) * 16;
#pragma unroll
  for (int i = 0; i < N; ++i) {
    const int rb = i >> 2, kb = (i >> 1) & 1, h = i & 1;
    v[i] = __builtin_amdgcn_raw_buffer_load_b128(src, base + rb * 32 * KS + kb * 64 + h * 32, 0, 0);
  }
}
__device__ __forceinline__ void load_w_raw(u32x4 (&r)[16], const Params& p, int dest, int nt, int kc, int lane) {
  const auto src = mk_rsrc(p.w, (unsigned)KS * NTOT);
  const unsigned base = (unsigned)(kc * 128 + (lane >> 3) * 16) * NTOT + dest * NS + nt * 128 + (lane & 7) * 16;
#pragma unroll
  for (int i = 0; i < 16; ++i) r[i] = __builtin_amdgcn_raw_buffer_load_b128(src, base + i * NTOT, 0, 0);
}
__device__ __forceinline__ void transpose16(const u32x4 (&r)[16], u32x4 (&o)[16]) {
#pragma unroll
  for (int d = 0; d < 4; ++d) {
#pragma unroll
    for (int e = 0; e < 4; ++e) {
      const unsigned r0 = r[4 * e][d], r1 = r[4 * e + 1][d], r2 = r[4 * e + 2][d], r3 = r[4 * e + 3][d];
      const unsigned t0 = __builtin_amdgcn_perm(r1, r0, 0x05010400u);
      const unsigned t1 = __builtin_amdgcn_perm(r1, r0, 0x07030602u);
      const unsigned t2 = __builtin_amdgcn_perm(r3, r2, 0x05010400u);
      const unsigned t3 = __builtin_amdgcn_perm(r3, r2, 0x07030602u);
      o[4 * d + 0][e] = __builtin_amdgcn_perm(t2, t0, 0x05040100u);
      o[4 * d + 1][e] = __builtin_amdgcn_perm(t2, t0, 0x07060302u);
      o[4 * d + 2][e] = __builtin_amdgcn_perm(t3, t1, 0x05040100u);
      o[4 * d + 3][e] = __builtin_amdgcn_perm(t3, t1, 0x07060302u);
    }
  }
}

constexpr unsigned ITEMS = 256;
__device__ __forceinline__ void push_wave(const Params& p, int pw, unsigned call, int lane) {
  const unsigned dest = (unsigned)(p.rank + (pw & 7)) & 7u;
  uint8_t* dws = p.ws[dest];
  const auto dst = mk_rsrc(dws + OFF_DATA + (call & 1u) * PAR_BYTES, PAR_BYTES);
  unsigned* xf = (unsigned*)(dws + OFF_XF);
  unsigned* wf = (unsigned*)(dws + OFF_WF);
  unsigned* gate = (unsigned*)(p.ws[p.rank] + OFF_PD) + dest * 32;
#define DRAW() do {} while (0)
  unsigned it = pw >> 3;
#pragma unroll 1
  while (it < ITEMS) {
    unsigned nxt = 0;
    const int kc = (it >> 5) & 3, slot = it & 31;
    if (it >= 128) {
      u32x4 v[8];
      const int mu = 16 + (slot >> 1), sub = slot & 1;
      load_x_unit<0, 8>(v, p, mu, kc, lane, sub * 8);
      const unsigned off = (unsigned)((p.rank * 4 + kc) * 32 + mu) * UNIT + sub * 8192;
#pragma unroll
      for (int j = 0; j < 8; ++j) {
        __builtin_amdgcn_raw_buffer_store_b128(v[j], dst, off + j * 1024 + lane * 16, 0, AUX_SYS);
        PUSH_THROTTLE();
      }
      DRAW();
      if (kc == GATE_IT && lane == 0)
        __hip_atomic_fetch_add(gate, 1u, __ATOMIC_RELAXED, __HIP_MEMORY_SCOPE_AGENT);
      WAIT_VM(0);
      if (lane == 0) st_sys(xf + (((mu >> 1) * 4 + kc) * 8 + p.rank) * 4 + (mu & 1) * 2 + sub, call);
    } else if (slot >= 16) {
      const int nt = slot - 16;
      u32x4 r[16], v[16];
      load_w_raw(r, p, (int)dest, nt, kc, lane);
      transpose16(r, v);
      const unsigned off = X_BYTES + (unsigned)((p.rank * 4 + kc) * 16 + nt) * UNIT;
#pragma unroll
      for (int j = 0; j < 16; ++j) {
        __builtin_amdgcn_raw_buffer_store_b128(v[j], dst, off + j * 1024 + ((lane ^ (((j >> 1) & 1) * 8)) * 16), 0, AUX_SYS);
        PUSH_THROTTLE();
      }
      DRAW();
      WAIT_VM(0);
      if (lane == 0) st_sys(wf + (nt * 4 + kc) * 8 + p.rank, call);
    } else {
      u32x4 v[16];
      const int mu = slot;
      load_x_unit<0, 16>(v, p, mu, kc, lane, 0);
      const unsigned off = (unsigned)((p.rank * 4 + kc) * 32 + mu) * UNIT;
#pragma unroll
      for (int j = 0; j < 16; ++j) {
        __builtin_amdgcn_raw_buffer_store_b128(v[j], dst, off + j * 1024 + lane * 16, 0, AUX_SYS);
        PUSH_THROTTLE();
      }
      DRAW();
      WAIT_VM(0);
      if (lane < 2) st_sys(xf + (((mu >> 1) * 4 + kc) * 8 + p.rank) * 4 + (mu & 1) * 2 + lane, call);
    }
    STAMPW(it >> 5);
    (void)nxt;
    it += 32;
  }
}

template <int WIN>
__device__ __forceinline__ void send_tile(const uint8_t* smem, const volatile lds_u32* seg_len, __amdgpu_buffer_rsrc_t dst,
                                          unsigned* sf, unsigned call, int lane) {
#pragma unroll
  for (int h = 0; h < 4; ++h) {
    const u32x4 v = *(const u32x4*)(smem + (h * 2 + (lane >> 5)) * SEG_BYTES + (lane & 31) * 16);
    __builtin_amdgcn_raw_buffer_store_b128(v, dst, h * 1024 + lane * 16, 0, AUX_SYS);
    asm volatile("s_waitcnt vmcnt(%0)" :: "i"(WIN) : "memory");
  }
  int published = 0;
  int issued = 0;
  int end_of[9];
  end_of[0] = 0;
#pragma unroll
  for (int sg = 0; sg < 8; ++sg) {
    const unsigned nbytes = __builtin_amdgcn_readfirstlane(seg_len[sg]);
    const int nch = (int)((nbytes + 1023u) >> 10);
#pragma unroll 1
    for (int c = 0; c < nch; ++c) {
      const unsigned o = (unsigned)c * 1024u + lane * 16u;
      if (o < nbytes) {
        const u32x4 v = *(const u32x4*)(smem + sg * SEG_BYTES + 512 + o);
        __builtin_amdgcn_raw_buffer_store_b128(v, dst, 4096u + sg * 16384u + o, 0, AUX_SYS);
      }
      asm volatile("s_waitcnt vmcnt(%0)" :: "i"(WIN) : "memory");
      ++issued;
      if (published == sg && issued >= end_of[sg] + WIN) {
        if (lane == 0) st_sys(sf + sg, call);
        published = sg + 1;
      }
    }
    if (published == sg) {
      WAIT_VM(0);
      if (lane == 0) st_sys(sf + sg, call);
      published = sg + 1;
    }
    end_of[sg + 1] = issued;
  }
  WAIT_VM(0);
  if (lane == 0) st_sys(sf + 8, call);
}

__global__ void __launch_bounds__(NTHREADS, 2) gemm_rs_kernel(Params p) {
  __shared__ __attribute__((aligned(16))) uint8_t smem[LDS_BYTES];
  const int tid = threadIdx.x;
  const int lane = tid & 63;
  const int wave = __builtin_amdgcn_readfirstlane(tid >> 6);
  const int b = blockIdx.x;
  uint8_t* myws = p.ws[p.rank];
  unsigned* tmo = (unsigned*)(myws + OFF_TMO);
  volatile lds_u32* lds_word = (volatile lds_u32*)(smem + LDS_MAIN);
  volatile lds_u32* lds_flags = (volatile lds_u32*)(smem + LDS_MAIN + 64);

  STAMP(0);
  unsigned* ctr = (unsigned*)(myws + OFF_CTR) + b;
  const unsigned call = __builtin_amdgcn_readfirstlane(ld_agent(ctr)) + 1u;

  const float scale = p.sx[0] * p.sw[0];

  if (b < N_PUSH_WG && wave < PW_PER_WG) push_wave(p, b * PW_PER_WG + wave, call, lane);
  STAMP(1);

  const int idx = b & 127, xcd = idx & 7, ii = idx >> 3;
  const int mt = ((b < 128) ? 8 : 0) + 4 * (xcd & 1) + (ii & 3);
  const int nt = 4 * (xcd >> 1) + (ii >> 2);
  const int wm = wave & 3, wn = wave >> 2;

  const uint8_t* data = myws + OFF_DATA + (call & 1u) * PAR_BYTES;
  const unsigned xa_off = (unsigned)((wm >> 1) * UNIT + (wm & 1) * 2 * 4096 + lane * 16);
  const unsigned wb_off = 2 * UNIT + (unsigned)(wn * 2 * 4096 + ((lane & 31) >> 3) * 1024 +
                                                ((((lane >> 5) ^ (lane >> 4)) & 1) * 8 + (lane & 7)) * 16);
  f32x16 acc[2][2];
#pragma unroll
  for (int i = 0; i < 2; ++i)
#pragma unroll
    for (int j = 0; j < 2; ++j)
#pragma unroll
      for (int r = 0; r < 16; ++r) acc[i][j][r] = 0.f;

  bool ok = true;
  const unsigned* fl0 = lane < 32 ? (const unsigned*)(myws + OFF_XF) + mt * 128 + lane
                                  : (const unsigned*)(myws + OFF_WF) + nt * 32 + (lane & 7);
  const int fl_stride = lane < 32 ? 32 : 8;
  const uint8_t* xbase = data + (unsigned)(2 * mt) * UNIT + wave * 4096 + lane * 16;
  const uint8_t* wbase = data + X_BYTES + (unsigned)nt * UNIT + wave * 2048 + lane * 16;
  auto glds1 = [&](int s, unsigned ring, int j) {
    const unsigned qk = (unsigned)(((p.rank + s) & 7) * 4 + (s >> 3));
    uint8_t* l = smem + ring;
    if (j < 4) __builtin_amdgcn_global_load_lds((const unsigned*)(xbase + qk * 32 * UNIT + j * 1024), (lds_u32*)(l + (wave * 4 + j) * 1024), 16, 0, 0);
    else __builtin_amdgcn_global_load_lds((const unsigned*)(wbase + qk * 16 * UNIT + (j - 4) * 1024), (lds_u32*)(l + 2 * UNIT + (wave * 2 + j - 4) * 1024), 16, 0, 0);
  };
  if (wave == 0) {
    const bool good = wave_wait_flags(fl0, true, call, tmo, call);
    __builtin_amdgcn_fence(__ATOMIC_ACQUIRE, "");
    WAIT_VM(0);
    if (lane == 0) *lds_word = good ? 1u : 0u;
  }
  RAW_BARRIER();
  ok = __builtin_amdgcn_readfirstlane(*lds_word) != 0u;
  STAMP(2);
  if (ok) {
    struct Frag { i32x4 r[8]; };
    auto rd = [&](Frag& f, unsigned ring, int kb, int i) {
      const unsigned o = (i < 2) ? wb_off + kb * 512 + i * 256
                       : (i < 6) ? xa_off + ((i - 2) >> 1) * 4096 + kb * 2048 + ((i - 2) & 1) * 1024
                                 : wb_off + 4096 + kb * 512 + (i - 6) * 256;
      f.r[i] = *(const i32x4*)(smem + ring + o);
    };
#define SCHED() __builtin_amdgcn_sched_barrier(0)
#define CAT(a, b) __builtin_shufflevector(a, b, 0, 1, 2, 3, 4, 5, 6, 7)
#define MA(f, i, j) acc[i][j] = __builtin_amdgcn_mfma_scale_f32_32x32x64_f8f6f4(                  \
      CAT(f.r[(i) * 6], f.r[(i) * 6 + 1]), CAT(f.r[2 + 2 * (j)], f.r[3 + 2 * (j)]), acc[i][j], 1, 1, 0, 0x7F7F7F7F, 0, 0x7F7F7F7F)
#pragma unroll
    for (int s = 0; s < 3; ++s)
#pragma unroll
      for (int j = 0; j < 6; ++j) glds1(s, s * STAGE_BYTES, j);
    WAIT_VM(12);
    RAW_BARRIER();
    Frag f0, f1;
#pragma unroll
    for (int i = 0; i < 8; ++i) rd(f0, 0, 0, i);
    __builtin_amdgcn_s_waitcnt(0xC07F);
    unsigned ring = 0, ringp = 2 * STAGE_BYTES;
#define STEP(WAITN, G2, G1) do {                                                         \
      const unsigned ring1 = ring == 2 * STAGE_BYTES ? 0u : ring + STAGE_BYTES;          \
      MA(f0, 0, 0); SCHED(); rd(f1, ring, 1, 0); rd(f1, ring, 1, 1); rd(f1, ring, 1, 2); rd(f1, ring, 1, 3); SCHED(); \
      MA(f0, 0, 1); SCHED(); rd(f1, ring, 1, 4); rd(f1, ring, 1, 5); rd(f1, ring, 1, 6); rd(f1, ring, 1, 7); SCHED(); \
      MA(f0, 1, 0); SCHED(); if (G2) { glds1(s + 2, ringp, 3); glds1(s + 2, ringp, 4); glds1(s + 2, ringp, 5); } SCHED(); \
      MA(f0, 1, 1); SCHED();                                                             \
      WAIT_VM(WAITN);                                                                    \
      RAW_BARRIER();            \
      MA(f1, 0, 0); SCHED(); rd(f0, ring1, 0, 0); rd(f0, ring1, 0, 1); rd(f0, ring1, 0, 2); rd(f0, ring1, 0, 3); SCHED(); \
      MA(f1, 0, 1); SCHED(); rd(f0, ring1, 0, 4); rd(f0, ring1, 0, 5); rd(f0, ring1, 0, 6); rd(f0, ring1, 0, 7); SCHED(); \
      MA(f1, 1, 0); SCHED(); if (G1) { glds1(s + 3, ring, 0); glds1(s + 3, ring, 1); glds1(s + 3, ring, 2); } SCHED(); \
      MA(f1, 1, 1);                                                                      \
      __builtin_amdgcn_s_waitcnt(0xC07F);                                                \
      SCHED();                                                                           \
      ringp = ring; ring = ring1;                                                        \
    } while (0)
    int s = 0;
    STEP(6, false, true);
    ++s;
#pragma unroll 1
    for (; s < 29; ++s) {
      if ((s & 7) == 3 && s < 27 && wave == 0)
        __builtin_amdgcn_global_load_lds(fl0 + ((s + 5) >> 3) * fl_stride, (lds_u32*)lds_flags, 4, 0, AUX_SYS);
      if ((s & 7) == 5) {
        if (!__all(lds_flags[lane] >= call)) {
          if (wave == 0) {
            const bool good = wave_wait_flags(fl0 + ((s + 3) >> 3) * fl_stride, true, call, tmo, call);
            if (!good && lane == 0) *lds_word = 0u;
          }
          RAW_BARRIER();
        }
        STAMP(2 + ((s + 3) >> 3));
      }
      STEP(6, true, true);
    }
    STEP(6, true, false);
    ++s;
    STEP(0, false, false);
#pragma unroll
    for (int i = 0; i < 8; ++i) rd(f1, ring, 1, i);
    MA(f0, 0, 0); MA(f0, 0, 1); MA(f0, 1, 0); MA(f0, 1, 1);
    MA(f1, 0, 0); MA(f1, 0, 1); MA(f1, 1, 0); MA(f1, 1, 1);
  }
  WAIT_VM(0);
  RAW_BARRIER();
  STAMP(6);

  const bool bad = *lds_word == 0u;
  {
    const float nanv = __builtin_nanf("");
#pragma unroll
    for (int i = 0; i < 2; ++i)
#pragma unroll
      for (int j = 0; j < 2; ++j) {
        const int row = wm * 64 + j * 32 + (lane & 31);
#pragma unroll
        for (int r = 0; r < 4; ++r) {
          f32x4 v;
#pragma unroll
          for (int t = 0; t < 4; ++t) {
            const float y = acc[i][j][r + 4 * t] * scale;
            v[t] = bad ? nanv : ((y <= 0.f) ? 0.f : y);
          }
          const int col = 16 * (r + 4 * (lane >> 5)) + 4 * (wn * 2 + i);
          *(f32x4*)(smem + row * EPI_PITCH + col * 4) = v;
        }
      }
  }
  const unsigned dest = (unsigned)(mt & 7);
  const int tile = (mt >> 3) * 16 + nt;
  RAW_BARRIER();
  const unsigned obase = (unsigned)((mt >> 3) * 256) * (NTOT * 4u) + (unsigned)(nt * 128) * 4u + (lane & 31) * 16;
#define OUT_VM 3
#define OUT_VM2 4
#define OUT_VM3 6
#define OUT_THROTTLE() asm volatile("s_waitcnt vmcnt(" STR(OUT_VM) ")" ::: "memory")
  if (dest == (unsigned)p.rank) {
    const auto dst = mk_rsrc(p.out[p.rank], 512u * NTOT * 4u);
#pragma unroll 4
    for (int it = 0; it < 16; ++it) {
      const int row = wave * 32 + it * 2 + (lane >> 5);
      const u32x4 v = *(const u32x4*)(smem + row * EPI_PITCH + (lane & 31) * 16);
      __builtin_amdgcn_raw_buffer_store_b128(v, dst, obase + (unsigned)(p.rank * NS * 4) + (unsigned)row * (NTOT * 4u), 0, AUX_OUT);
    }
    if (tid == 0) st_agent(ctr, call);
    STAMP(7);
    return;
  }
  {
    uint8_t* seg = smem + wave * SEG_BYTES;
    u32x4 v[16];
#pragma unroll
    for (int blk = 0; blk < 16; ++blk)
      v[blk] = *(const u32x4*)(seg + (blk * 2 + (lane >> 5)) * EPI_PITCH + (lane & 31) * 16);
    WAIT_LGKM0();
    __builtin_amdgcn_sched_barrier(0);
    unsigned base = 512;
    unsigned blo = 0, bhi = 0;
#pragma unroll
    for (int blk = 0; blk < 16; ++blk)
#pragma unroll
      for (int t = 0; t < 4; ++t) {
        const int e = blk * 4 + t;
        const unsigned val = v[blk][t];
        const bool nz = val != 0u;
        const unsigned long long m = __ballot(nz);
        const unsigned pos = __builtin_amdgcn_mbcnt_hi((unsigned)(m >> 32), __builtin_amdgcn_mbcnt_lo((unsigned)m, 0u));
        *(unsigned*)(seg + (nz ? base + pos * 4 : lane * 8u)) = val;
        base += (unsigned)__popcll(m) * 4u;
        if (e < 32) blo |= nz ? (1u << (e & 31)) : 0u; else bhi |= nz ? (1u << (e & 31)) : 0u;
      }
    *(uint2*)(seg + lane * 8) = make_uint2(blo, bhi);
    if (lane == 0) lds_word[1 + wave] = base - 512u;
    if (tid == 0) lds_word[9] = 0u;
  }
  RAW_BARRIER();
  STAMP(10);
  if (wave == 0) {
    (void)wave_wait_flags((const unsigned*)(myws + OFF_PD) + dest * 32, lane == 0, (call - 1u) * 32u + GATE_N, tmo, call);
    STAMP(9);
    const auto dst = mk_rsrc(p.ws[dest] + OFF_STG + (unsigned)(p.rank * 32 + tile) * TILE_BYTES, TILE_BYTES);
    unsigned* sf = (unsigned*)(p.ws[dest] + OFF_SF) + (p.rank * 32 + tile) * 16;
    if (b >= 128) send_tile<OUT_VM>(smem, lds_word + 1, dst, sf, call, lane);
    else if (b >= N_PUSH_WG) send_tile<OUT_VM2>(smem, lds_word + 1, dst, sf, call, lane);
    else send_tile<OUT_VM3>(smem, lds_word + 1, dst, sf, call, lane);
    if (lane == 0) st_agent(ctr, call);
    STAMP(7);
  } else if (wave == 1) {
    const unsigned* sf = (const unsigned*)(myws + OFF_SF) + (dest * 32 + tile) * 16;
    const unsigned long long t0 = __builtin_amdgcn_s_memrealtime();
    int have = 0;
    for (unsigned spin = 1;; ++spin) {
      const unsigned v = lane < 9 ? ld_sys(sf + lane) : 0u;
      const int n = __builtin_ctzll(~__ballot(v >= call));
      if (n > have) {
        have = n;
        if (lane == 0) lds_word[9] = (unsigned)n;
        if (n >= 9) break;
      }
      if ((spin & 7u) == 0u &&
          (ld_agent(tmo) == call || __builtin_amdgcn_s_memrealtime() - t0 > DEADLINE_TICKS)) {
        st_agent(tmo, call);
        if (lane == 0) lds_word[9] = 0x100u;
        break;
      }
      __builtin_amdgcn_s_sleep(1);
    }
  } else if (wave >= 4) {
    const int rw = wave - 4;
    const auto src = mk_rsrc(myws + OFF_STG + (unsigned)(dest * 32 + tile) * TILE_BYTES, TILE_BYTES);
    const auto dst = mk_rsrc(p.out[p.rank], 512u * NTOT * 4u);
    const unsigned ob = obase + dest * (NS * 4u);
    auto wait_part = [&](unsigned need) -> bool {
      unsigned v;
      while ((v = __builtin_amdgcn_readfirstlane(lds_word[9])) < need) __builtin_amdgcn_s_sleep(1);
      asm volatile("" ::: "memory");
      return v < 0x100u;
    };
    (void)wait_part(1);
    unsigned mine[8], below[8];
#pragma unroll
    for (int sg = 0; sg < 8; ++sg) {
      const auto mm = __builtin_amdgcn_raw_buffer_load_b64(src, sg * 512 + lane * 8, 0, 0);
      const unsigned long long bits = ((unsigned long long)mm[1] << 32) | mm[0];
      mine[sg] = (unsigned)(bits >> (16 * rw)) & 0xFFFFu;
      below[sg] = (unsigned)__popcll(bits & ((1ull << (16 * rw)) - 1ull));
    }
#pragma unroll
    for (int sg = 0; sg < 8; ++sg) {
      unsigned base = 0;
#pragma unroll
      for (int bt = 0; bt < 6; ++bt) base += (unsigned)__popcll(__ballot((below[sg] >> bt) & 1u)) << bt;
      base = 4096u + sg * 16384u + base * 4u;
      unsigned off[16];
#pragma unroll
      for (int e = 0; e < 16; ++e) {
        const bool bit = (mine[sg] >> e) & 1u;
        const unsigned long long m = __ballot(bit);
        const unsigned pos = __builtin_amdgcn_mbcnt_hi((unsigned)(m >> 32), __builtin_amdgcn_mbcnt_lo((unsigned)m, 0u));
        off[e] = bit ? base + pos * 4u : 0xFFFFFFF0u;
        base += (unsigned)__popcll(m) * 4u;
      }
      const bool good = wait_part(2 + sg);
      u32x4 v[4];
#pragma unroll
      for (int e = 0; e < 16; ++e) v[e >> 2][e & 3] = __builtin_amdgcn_raw_buffer_load_b32(src, off[e], 0, 0);
#pragma unroll
      for (int bb = 0; bb < 4; ++bb) {
        const int row = sg * 32 + (rw * 4 + bb) * 2 + (lane >> 5);
        if (!good) v[bb] = u32x4{0x7FC00000u, 0x7FC00000u, 0x7FC00000u, 0x7FC00000u};
        __builtin_amdgcn_raw_buffer_store_b128(v[bb], dst, ob + (unsigned)row * (NTOT * 4u), 0, AUX_OUT);
      }
    }
    if (wave == 4) { if (lane == 0) { STAMPX(8); } }
  }
}

}

extern "C" void kernel_launch(int rank, int nranks, ncclComm_t comm,
                              hipStream_t stream,
                              void* const* d_in, const int* in_sizes, int n_in,
                              void* d_out, int out_size,
                              const int* in_dtypes, int out_dtype,
                              void* const* d_out_all,
                              void* const* d_ws_all, size_t ws_bytes) {
  (void)comm; (void)in_sizes; (void)n_in; (void)d_out; (void)out_size; (void)in_dtypes; (void)out_dtype;
  if (nranks != NR || (unsigned)rank >= (unsigned)NR || ws_bytes < WS_NEED) {
    fprintf(stderr, "kernel_launch: unsupported configuration\n");
    return;
  }
  Params p;
  p.x = (const uint8_t*)d_in[0];
  p.w = (const uint8_t*)d_in[1];
  p.sx = (const float*)d_in[2];
  p.sw = (const float*)d_in[3];
  for (int i = 0; i < NR; ++i) {
    p.out[i] = (float*)d_out_all[i];
    p.ws[i] = (uint8_t*)d_ws_all[i];
  }
  p.rank = rank;
  hipLaunchKernelGGL(gemm_rs_kernel, dim3(NWG), dim3(NTHREADS), 0, stream, p);
  HIP_CHECK(hipGetLastError());
}
```

```cpp
#include <hip/hip_runtime.h>
#include <rccl/rccl.h>
#include <cstdio>
#include <cstdint>

#define HIP_CHECK(expr) do { hipError_t _e = (expr); if (_e != hipSuccess) { \
    fprintf(stderr, "HIP error %s at %s:%d\n", hipGetErrorString(_e), __FILE__, __LINE__); \
    return; } } while (0)

namespace {

typedef unsigned u32x4 __attribute__((ext_vector_type(4)));
typedef int i32x8 __attribute__((ext_vector_type(8)));
typedef int i32x4 __attribute__((ext_vector_type(4)));
typedef float f32x16 __attribute__((ext_vector_type(16)));
typedef float f32x4 __attribute__((ext_vector_type(4)));
typedef __attribute__((address_space(3))) unsigned lds_u32;

constexpr int NR = 8;
constexpr int NTOT = 16384;
constexpr int KS = 512;
constexpr int NS = 2048;
constexpr unsigned UNIT = 16384;

constexpr unsigned OFF_CTR = 0;
constexpr unsigned OFF_TMO = 4096;
constexpr unsigned OFF_XF = 8192;
constexpr unsigned OFF_WF = 16384;
constexpr unsigned OFF_TK = 24576;
constexpr unsigned OFF_H2 = 26624;
constexpr unsigned OFF_PD = 28672;
constexpr unsigned OFF_HD = 30720;
constexpr unsigned OFF_SF = 32768;
constexpr unsigned OFF_DIAG = 65536;
constexpr unsigned OFF_DATA = 1u << 20;
constexpr unsigned X_BYTES = 16u << 20;
constexpr unsigned W_BYTES = 8u << 20;
constexpr unsigned PAR_BYTES = X_BYTES + W_BYTES;
constexpr unsigned ITEM_BYTES = 16 * 528;
constexpr unsigned TILE_BYTES = 16 * ITEM_BYTES;
constexpr unsigned OFF_STG = OFF_DATA + 2 * PAR_BYTES;
constexpr unsigned STG_BYTES = 8 * 32 * TILE_BYTES;
constexpr size_t WS_NEED = (size_t)OFF_STG + STG_BYTES;

constexpr int NWG = 256;
constexpr int NTHREADS = 512;
#define PUSH_WAVES_PER_WG 4
#define GATE_N 4
#define GATE_IT 3
constexpr int PW_PER_WG = PUSH_WAVES_PER_WG;
constexpr int N_PUSH_WG = 256 / PW_PER_WG;
constexpr unsigned STAGE_BYTES = 3 * UNIT;
constexpr unsigned EPI_PITCH = 528;
constexpr unsigned LDS_MAIN = 3 * STAGE_BYTES;
constexpr unsigned LDS_BYTES = LDS_MAIN + 512;
constexpr unsigned long long DEADLINE_TICKS = 40000000ull;
#define AUX_OUT 16
constexpr int AUX_SYS = 17;

struct Params {
  const uint8_t* x;
  const uint8_t* w;
  const float* sx;
  const float* sw;
  float* out[NR];
  uint8_t* ws[NR];
  int rank;
};

__device__ __forceinline__ __amdgpu_buffer_rsrc_t mk_rsrc(const void* p, unsigned bytes) {
  return __builtin_amdgcn_make_buffer_rsrc((void*)p, 0, bytes, 0x00020000);
}
__device__ __forceinline__ unsigned ld_sys(const unsigned* p) {
  return __hip_atomic_load(p, __ATOMIC_RELAXED, __HIP_MEMORY_SCOPE_SYSTEM);
}
__device__ __forceinline__ void st_sys(unsigned* p, unsigned v) {
  __hip_atomic_store(p, v, __ATOMIC_RELAXED, __HIP_MEMORY_SCOPE_SYSTEM);
}
__device__ __forceinline__ unsigned ld_agent(const unsigned* p) {
  return __hip_atomic_load(p, __ATOMIC_RELAXED, __HIP_MEMORY_SCOPE_AGENT);
}
__device__ __forceinline__ void st_agent(unsigned* p, unsigned v) {
  __hip_atomic_store(p, v, __ATOMIC_RELAXED, __HIP_MEMORY_SCOPE_AGENT);
}
#define STR2(x) #x
#define STR(x) STR2(x)
#define PUSH_VM 1
#define PULL_VM2 3
#define PULL_FAST 480
#define PULL_AHEAD 2
#define H_STOP 200
#define FREE_AHEAD 8
#define PUSH_VM2 3
#define WAIT_VM(n) asm volatile("s_waitcnt vmcnt(" #n ")" ::: "memory")
#define WAIT_LGKM0() asm volatile("s_waitcnt lgkmcnt(0)" ::: "memory")
#define RAW_BARRIER() do { WAIT_LGKM0(); __builtin_amdgcn_s_barrier(); } while (0)

__device__ __forceinline__ bool wave_wait_flags(const unsigned* flag, bool active, unsigned target, unsigned* tmo, unsigned call) {
  const unsigned long long t0 = __builtin_amdgcn_s_memrealtime();
  for (unsigned spin = 1;; ++spin) {
    const unsigned v = active ? ld_sys(flag) : target;
    if (__all(v >= target)) return true;
    if ((spin & 7u) == 0u &&
        (ld_agent(tmo) == call || __builtin_amdgcn_s_memrealtime() - t0 > DEADLINE_TICKS)) {
      st_agent(tmo, call);
      return false;
    }
    __builtin_amdgcn_s_sleep(2);
  }
}

#define LOGB(wsbase, wv, slot, dest, bytes) do {} while (0)
#define STAMP(slot) do {} while (0)
#define STAMPX(slot) do {} while (0)
#define STAMPW(slot) do {} while (0)

template <int N>
__device__ __forceinline__ void load_x_unit(u32x4 (&v)[N], const Params& p, int mu, int kc, int lane, int i0) {
  const int mt = mu >> 1;
  const int row0 = 512 * (mt & 7) + 256 * (mt >> 3) + 128 * (mu & 1);
  const auto src = mk_rsrc(p.x, 4096u * KS);
  const unsigned base = (unsigned)(row0 + (i0 >> 2) * 32 + (lane & 31)) * KS + kc * 128 + (lane >> 5) * 16;
#pragma unroll
  for (int i = 0; i < N; ++i) {
    const int rb = i >> 2, kb = (i >> 1) & 1, h = i & 1;
    v[i] = __builtin_amdgcn_raw_buffer_load_b128(src, base + rb * 32 * KS + kb * 64 + h * 32, 0, 0);
  }
}
__device__ __forceinline__ void load_w_raw(u32x4 (&r)[16], const Params& p, int dest, int nt, int kc, int lane) {
  const auto src = mk_rsrc(p.w, (unsigned)KS * NTOT);
  const unsigned base = (unsigned)(kc * 128 + (lane >> 3) * 16) * NTOT + dest * NS + nt * 128 + (lane & 7) * 16;
#pragma unroll
  for (int i = 0; i < 16; ++i) r[i] = __builtin_amdgcn_raw_buffer_load_b128(src, base + i * NTOT, 0, 0);
}
__device__ __forceinline__ void transpose16(const u32x4 (&r)[16], u32x4 (&o)[16]) {
#pragma unroll
  for (int d = 0; d < 4; ++d) {
#pragma unroll
    for (int e = 0; e < 4; ++e) {
      const unsigned r0 = r[4 * e][d], r1 = r[4 * e + 1][d], r2 = r[4 * e + 2][d], r3 = r[4 * e + 3][d];
      const unsigned t0 = __builtin_amdgcn_perm(r1, r0, 0x05010400u);
      const unsigned t1 = __builtin_amdgcn_perm(r1, r0, 0x07030602u);
      const unsigned t2 = __builtin_amdgcn_perm(r3, r2, 0x05010400u);
      const unsigned t3 = __builtin_amdgcn_perm(r3, r2, 0x07030602u);
      o[4 * d + 0][e] = __builtin_amdgcn_perm(t2, t0, 0x05040100u);
      o[4 * d + 1][e] = __builtin_amdgcn_perm(t2, t0, 0x07060302u);
      o[4 * d + 2][e] = __builtin_amdgcn_perm(t3, t1, 0x05040100u);
      o[4 * d + 3][e] = __builtin_amdgcn_perm(t3, t1, 0x07060302u);
    }
  }
}

template <int WIN>
__device__ __forceinline__ void push_phase1(const Params& p, unsigned dest, int slot, unsigned call, int lane) {
  uint8_t* dws = p.ws[dest];
  const auto dst = mk_rsrc(dws + OFF_DATA + (call & 1u) * PAR_BYTES, PAR_BYTES);
#pragma unroll 1
  for (int kc = 0; kc < 4; ++kc) {
    u32x4 v[16];
    if (slot >= 16) {
      const int nt = slot - 16;
      u32x4 r[16];
      load_w_raw(r, p, (int)dest, nt, kc, lane);
      transpose16(r, v);
      const unsigned off = X_BYTES + (unsigned)((p.rank * 4 + kc) * 16 + nt) * UNIT;
#pragma unroll
      for (int j = 0; j < 16; ++j) {
        __builtin_amdgcn_raw_buffer_store_b128(v[j], dst, off + j * 1024 + ((lane ^ (((j >> 1) & 1) * 8)) * 16), 0, AUX_SYS);
        asm volatile("s_waitcnt vmcnt(%0)" :: "i"(WIN) : "memory");
      }
      WAIT_VM(0);
      if (lane == 0) st_sys((unsigned*)(dws + OFF_WF) + (nt * 4 + kc) * 8 + p.rank, call);
    } else {
      const int mu = slot;
      load_x_unit<16>(v, p, mu, kc, lane, 0);
      const unsigned off = (unsigned)((p.rank * 4 + kc) * 32 + mu) * UNIT;
#pragma unroll
      for (int j = 0; j < 16; ++j) {
        __builtin_amdgcn_raw_buffer_store_b128(v[j], dst, off + j * 1024 + lane * 16, 0, AUX_SYS);
        asm volatile("s_waitcnt vmcnt(%0)" :: "i"(WIN) : "memory");
      }
      WAIT_VM(0);
      if (lane < 2) st_sys((unsigned*)(dws + OFF_XF) + (((mu >> 1) * 4 + kc) * 8 + p.rank) * 4 + (mu & 1) * 2 + lane, call);
    }
    STAMPW(kc);
  }
}
template <int WIN>
__device__ __forceinline__ void push_phase2(const Params& p, unsigned dest, int slot, int kc0, int kc1, unsigned call, int lane) {
  uint8_t* dws = p.ws[dest];
  const auto dst = mk_rsrc(dws + OFF_DATA + (call & 1u) * PAR_BYTES, PAR_BYTES);
  const int mu = 16 + (slot >> 1), sub = slot & 1;
#pragma unroll 1
  for (int kc = kc0; kc < kc1; ++kc) {
    u32x4 v[8];
    load_x_unit<8>(v, p, mu, kc, lane, sub * 8);
    const unsigned off = (unsigned)((p.rank * 4 + kc) * 32 + mu) * UNIT + sub * 8192;
#pragma unroll
    for (int j = 0; j < 8; ++j) {
      __builtin_amdgcn_raw_buffer_store_b128(v[j], dst, off + j * 1024 + lane * 16, 0, AUX_SYS);
      asm volatile("s_waitcnt vmcnt(%0)" :: "i"(WIN) : "memory");
    }
    if (kc == GATE_IT && lane == 0)
      __hip_atomic_fetch_add((unsigned*)(p.ws[p.rank] + OFF_PD) + dest * 32, 1u, __ATOMIC_RELAXED, __HIP_MEMORY_SCOPE_AGENT);
    WAIT_VM(0);
    if (lane == 0) st_sys((unsigned*)(dws + OFF_XF) + (((mu >> 1) * 4 + kc) * 8 + p.rank) * 4 + (mu & 1) * 2 + sub, call);
    STAMPW(4 + kc);
  }
}

constexpr unsigned ITEMS = 256;
__device__ __forceinline__ void lds_wait_ge(const volatile lds_u32* w, unsigned v) {
  while (__builtin_amdgcn_readfirstlane(*w) < v) __builtin_amdgcn_s_sleep(1);
  asm volatile("" ::: "memory");
}
template <int N>
__device__ __forceinline__ void glds_x(const Params& p, uint8_t* l, int mu, int kc, int lane, int i0) {
  const int mt = mu >> 1;
  const int row0 = 512 * (mt & 7) + 256 * (mt >> 3) + 128 * (mu & 1);
  const uint8_t* src = p.x + (unsigned)(row0 + i0 * 8 + (lane >> 3)) * KS + kc * 128 + (((lane ^ (lane >> 3)) & 7) * 16);
#pragma unroll
  for (int i = 0; i < N; ++i)
    __builtin_amdgcn_global_load_lds((const unsigned*)(src + i * 8 * KS), (lds_u32*)(l + i * 1024), 16, 0, 0);
}
__device__ __forceinline__ unsigned push_loader(const Params& p, uint8_t* ring, volatile lds_u32* ctl, unsigned dest, unsigned it,
                                                unsigned stop, const unsigned* ctr, int lane) {
  unsigned call = 0;
  unsigned* head = nullptr;
#pragma unroll 1
  for (unsigned k = 0;; ++k) {
    uint8_t* l = ring + (k & 1) * UNIT;
    if (k > 0) {
      lds_wait_ge(ctl + 1, k);
      if (it >= stop) it = ITEMS;
      else {
        unsigned t = 0;
        if (lane == 0) t = __hip_atomic_fetch_add(head, 1u, __ATOMIC_RELAXED, __HIP_MEMORY_SCOPE_AGENT);
        it = __builtin_amdgcn_readfirstlane(t) + 32u;
      }
    }
    const int kc = (it >> 5) & 3, slot = it & 31;
    if (it >= ITEMS) {
    } else if (it < 128 && slot >= 16) {
      u32x4 r[16], v[16];
      load_w_raw(r, p, (int)dest, slot - 16, kc, lane);
      if (k == 0) __builtin_amdgcn_s_barrier();
      transpose16(r, v);
#pragma unroll
      for (int j = 0; j < 16; ++j) *(u32x4*)(l + j * 1024 + ((lane ^ (((j >> 1) & 1) * 8)) * 16)) = v[j];
    } else {
      if (it < 128) glds_x<16>(p, l, slot, kc, lane, 0);
      else glds_x<8>(p, l, 16 + (slot >> 1), kc, lane, (slot & 1) * 8);
      if (k == 0) __builtin_amdgcn_s_barrier();
      WAIT_VM(0);
    }
    if (lane == 0) ctl[2 + (k & 1)] = it;
    WAIT_LGKM0();
    if (lane == 0) ctl[0] = k + 1;
    if (k == 0 && threadIdx.x == 256) { STAMPX(12); }
    if (k == 0) {
      call = __builtin_amdgcn_readfirstlane(ld_agent(ctr)) + 1u;
      head = (unsigned*)(p.ws[p.rank] + OFF_HD) + ((call & 1u) * 8 + dest) * 32;
    }
    if (it >= ITEMS) return call;
  }
}
template <int NCH, int WIN>
__device__ __forceinline__ void stream_item(const uint8_t* l, bool raw_x, __amdgpu_buffer_rsrc_t dst, unsigned off, unsigned* prev_flag,
                                            int prev_n, unsigned call, int lane, volatile lds_u32* freed, unsigned freed_val) {
  u32x4 v[NCH];
  unsigned a4[4];
#pragma unroll
  for (int q = 0; q < 4; ++q)
    a4[q] = raw_x ? (unsigned)((lane & 31) * 128 + (((q * 2 + (lane >> 5)) ^ lane) & 7) * 16) : (unsigned)(q * 1024 + lane * 16);
#pragma unroll
  for (int j = 0; j < NCH; ++j) v[j] = *(const u32x4*)(l + (j >> 2) * 4096 + a4[j & 3]);
#pragma unroll
  for (int j = 0; j < NCH; ++j) {
    __builtin_amdgcn_raw_buffer_store_b128(v[j], dst, off + j * 1024 + lane * 16, 0, AUX_SYS);
    asm volatile("s_waitcnt vmcnt(%0)" :: "i"(WIN) : "memory");
    if (j + 1 == WIN && lane < prev_n) st_sys(prev_flag + lane, call);
    if (j == NCH - FREE_AHEAD && lane == 0) *freed = freed_val;
  }
}
__device__ __forceinline__ unsigned* xflag(uint8_t* dws, int rank, int mu, int kc, int sub) {
  return (unsigned*)(dws + OFF_XF) + (((mu >> 1) * 4 + kc) * 8 + rank) * 4 + (mu & 1) * 2 + sub;
}
__device__ __forceinline__ void push_storer(const Params& p, const uint8_t* ring, volatile lds_u32* ctl, unsigned dest,
                                            unsigned call, int lane) {
  uint8_t* dws = p.ws[dest];
  const auto dst = mk_rsrc(dws + OFF_DATA + (call & 1u) * PAR_BYTES, PAR_BYTES);
  unsigned* pf = nullptr;
  int pn = 0;
#pragma unroll 1
  for (unsigned k = 0;; ++k) {
    if (k == 0) { STAMP(11); }
    lds_wait_ge(ctl, k + 1);
    if (k == 0) { STAMP(13); }
    const unsigned it = __builtin_amdgcn_readfirstlane(ctl[2 + (k & 1)]);
    if (it >= ITEMS) break;
    const int kc = (it >> 5) & 3, slot = it & 31;
    if (it < 128) {
      const bool is_w = slot >= 16;
      const unsigned off = is_w ? X_BYTES + (unsigned)((p.rank * 4 + kc) * 16 + slot - 16) * UNIT
                                : (unsigned)((p.rank * 4 + kc) * 32 + slot) * UNIT;
      stream_item<16, PUSH_VM>(ring + (k & 1) * UNIT, !is_w, dst, off, pf, pn, call, lane, ctl + 1, k + 1);
      pf = is_w ? (unsigned*)(dws + OFF_WF) + ((slot - 16) * 4 + kc) * 8 + p.rank : xflag(dws, p.rank, slot, kc, 0);
      pn = is_w ? 1 : 2;
    } else {
      const int mu = 16 + (slot >> 1), sub = slot & 1;
      const unsigned off = (unsigned)((p.rank * 4 + kc) * 32 + mu) * UNIT + sub * 8192;
      if (it < H_STOP + 16) stream_item<8, PUSH_VM>(ring + (k & 1) * UNIT, true, dst, off, pf, pn, call, lane, ctl + 1, k + 1);
      else stream_item<8, PUSH_VM2>(ring + (k & 1) * UNIT, true, dst, off, pf, pn, call, lane, ctl + 1, k + 1);
      if (kc == GATE_IT && lane == 0)
        __hip_atomic_fetch_add((unsigned*)(p.ws[p.rank] + OFF_PD) + dest * 32, 1u, __ATOMIC_RELAXED, __HIP_MEMORY_SCOPE_AGENT);
      pf = xflag(dws, p.rank, mu, kc, sub);
      pn = 1;
    }
    STAMPW(it >> 5);
    LOGB(p.ws[p.rank], threadIdx.x >> 6, k, dest, it < 128 ? 16384 : 8192);
  }
  WAIT_VM(0);
  if (lane < pn) st_sys(pf + lane, call);
}

__global__ void __launch_bounds__(NTHREADS, 2) gemm_rs_kernel(Params p) {
  __shared__ __attribute__((aligned(16))) uint8_t smem[LDS_BYTES];
  const int tid = threadIdx.x;
  const int lane = tid & 63;
  const int wave = __builtin_amdgcn_readfirstlane(tid >> 6);
  const int b = blockIdx.x;
  uint8_t* myws = p.ws[p.rank];
  unsigned* tmo = (unsigned*)(myws + OFF_TMO);
  volatile lds_u32* lds_word = (volatile lds_u32*)(smem + LDS_MAIN);
  volatile lds_u32* lds_flags = lds_word + 64;

  STAMP(0);
  unsigned* ctr = (unsigned*)(myws + OFF_CTR) + b;
  unsigned call = 0;
  if (b >= 56) call = __builtin_amdgcn_readfirstlane(ld_agent(ctr)) + 1u;

  const float scale = p.sx[0] * p.sw[0];

  if (b < 56) {
    const int pr = wave & 3, pw = b * 4 + pr;
    volatile lds_u32* ctl = lds_word + 32 + pr * 4;
    const unsigned dest = (unsigned)(p.rank + 1 + pw % 7) & 7u;
    uint8_t* ring = smem + pr * 2 * UNIT;
    if (wave < 4) {
      const unsigned craw = ld_agent(ctr);
      if (lane < 4) ctl[lane] = 0u;
      RAW_BARRIER();
      call = __builtin_amdgcn_readfirstlane(craw) + 1u;
      push_storer(p, ring, ctl, dest, call, lane);
    } else {
      call = push_loader(p, ring, ctl, dest, (unsigned)(pw / 7), b < 32 ? ITEMS : H_STOP, ctr, lane);
    }
  } else if (b < 64) {
    const int slot = (b - 56) * 4 + (wave & 3);
    if (wave < 4) push_phase1<12>(p, (unsigned)p.rank, slot, call, lane);
    else push_phase2<12>(p, (unsigned)p.rank, slot, 0, 4, call, lane);
    if (b == 56 && tid < 8) st_agent((unsigned*)(myws + OFF_HD) + (((call + 1u) & 1u) * 8 + tid) * 32, 0u);
    if (b == 57 && tid < 8) st_agent((unsigned*)(myws + OFF_H2) + (((call + 1u) & 1u) * 8 + tid) * 32, 0u);
  }
  STAMP(1);

  int mt, nt;
  {
    const int xcd = b & 7, ii = b >> 3;
    int half, mrel;
    if (ii < 4) { half = ii >> 1; mrel = 0; nt = 2 * xcd + (ii & 1); }
    else {
      half = ii < 18 ? 1 : 0;
      const int j = ii < 18 ? ii - 4 : ii - 18, o = xcd & 1, g = xcd >> 1;
      (void)o; (void)g;
      mrel = 1 + (j + xcd) % 7; nt = 2 * xcd + j / 7;
    }
    mt = half * 8 + ((p.rank + mrel) & 7);
  }
  const int wm = wave & 3, wn = wave >> 2;

  const uint8_t* data = myws + OFF_DATA + (call & 1u) * PAR_BYTES;
  const unsigned xa_off = (unsigned)((wm >> 1) * UNIT + (wm & 1) * 2 * 4096 + lane * 16);
  const unsigned wb_off = 2 * UNIT + (unsigned)(wn * 2 * 4096 + ((lane & 31) >> 3) * 1024 +
                                                ((((lane >> 5) ^ (lane >> 4)) & 1) * 8 + (lane & 7)) * 16);
  f32x16 acc[2][2];
#pragma unroll
  for (int i = 0; i < 2; ++i)
#pragma unroll
    for (int j = 0; j < 2; ++j)
#pragma unroll
      for (int r = 0; r < 16; ++r) acc[i][j][r] = 0.f;

  bool ok = true;
  const unsigned* fl0 = lane < 32 ? (const unsigned*)(myws + OFF_XF) + mt * 128 + lane
                                  : (const unsigned*)(myws + OFF_WF) + nt * 32 + (lane & 7);
  const int fl_stride = lane < 32 ? 32 : 8;
  const uint8_t* xbase = data + (unsigned)(2 * mt) * UNIT + wave * 4096 + lane * 16;
  const uint8_t* wbase = data + X_BYTES + (unsigned)nt * UNIT + wave * 2048 + lane * 16;
  auto glds1 = [&](int s, unsigned ring, int j) {
    const unsigned qk = (unsigned)(((p.rank + s) & 7) * 4 + (s >> 3));
    uint8_t* l = smem + ring;
    if (j < 4) __builtin_amdgcn_global_load_lds((const unsigned*)(xbase + qk * 32 * UNIT + j * 1024), (lds_u32*)(l + (wave * 4 + j) * 1024), 16, 0, 0);
    else __builtin_amdgcn_global_load_lds((const unsigned*)(wbase + qk * 16 * UNIT + (j - 4) * 1024), (lds_u32*)(l + 2 * UNIT + (wave * 2 + j - 4) * 1024), 16, 0, 0);
  };
  if (wave == 0) {
    const bool good = wave_wait_flags(fl0, true, call, tmo, call);
    __builtin_amdgcn_fence(__ATOMIC_ACQUIRE, "");
    WAIT_VM(0);
    if (lane == 0) *lds_word = good ? 1u : 0u;
  }
  RAW_BARRIER();
  ok = __builtin_amdgcn_readfirstlane(*lds_word) != 0u;
  STAMP(2);
  if (ok) {
    struct Frag { i32x4 r[8]; };
    auto rd = [&](Frag& f, unsigned ring, int kb, int i) {
      const unsigned o = (i < 2) ? wb_off + kb * 512 + i * 256
                       : (i < 6) ? xa_off + ((i - 2) >> 1) * 4096 + kb * 2048 + ((i - 2) & 1) * 1024
                                 : wb_off + 4096 + kb * 512 + (i - 6) * 256;
      f.r[i] = *(const i32x4*)(smem + ring + o);
    };
#define SCHED() __builtin_amdgcn_sched_barrier(0)
#define CAT(a, b) __builtin_shufflevector(a, b, 0, 1, 2, 3, 4, 5, 6, 7)
#define MA(f, i, j) acc[i][j] = __builtin_amdgcn_mfma_scale_f32_32x32x64_f8f6f4(                  \
      CAT(f.r[(i) * 6], f.r[(i) * 6 + 1]), CAT(f.r[2 + 2 * (j)], f.r[3 + 2 * (j)]), acc[i][j], 1, 1, 0, 0x7F7F7F7F, 0, 0x7F7F7F7F)
#pragma unroll
    for (int s = 0; s < 3; ++s)
#pragma unroll
      for (int j = 0; j < 6; ++j) glds1(s, s * STAGE_BYTES, j);
    WAIT_VM(12);
    RAW_BARRIER();
    Frag f0, f1;
#pragma unroll
    for (int i = 0; i < 8; ++i) rd(f0, 0, 0, i);
    __builtin_amdgcn_s_waitcnt(0xC07F);
    unsigned ring = 0, ringp = 2 * STAGE_BYTES;
#define STEP(WAITN, G2, G1) do {                                                         \
      const unsigned ring1 = ring == 2 * STAGE_BYTES ? 0u : ring + STAGE_BYTES;          \
      MA(f0, 0, 0); SCHED(); rd(f1, ring, 1, 0); rd(f1, ring, 1, 1); rd(f1, ring, 1, 2); rd(f1, ring, 1, 3); SCHED(); \
      MA(f0, 0, 1); SCHED(); rd(f1, ring, 1, 4); rd(f1, ring, 1, 5); rd(f1, ring, 1, 6); rd(f1, ring, 1, 7); SCHED(); \
      MA(f0, 1, 0); SCHED(); if (G2) { glds1(s + 2, ringp, 3); glds1(s + 2, ringp, 4); glds1(s + 2, ringp, 5); } SCHED(); \
      MA(f0, 1, 1); SCHED();                                                             \
      WAIT_VM(WAITN);                                                                    \
      RAW_BARRIER();            \
      MA(f1, 0, 0); SCHED(); rd(f0, ring1, 0, 0); rd(f0, ring1, 0, 1); rd(f0, ring1, 0, 2); rd(f0, ring1, 0, 3); SCHED(); \
      MA(f1, 0, 1); SCHED(); rd(f0, ring1, 0, 4); rd(f0, ring1, 0, 5); rd(f0, ring1, 0, 6); rd(f0, ring1, 0, 7); SCHED(); \
      MA(f1, 1, 0); SCHED(); if (G1) { glds1(s + 3, ring, 0); glds1(s + 3, ring, 1); glds1(s + 3, ring, 2); } SCHED(); \
      MA(f1, 1, 1);                                                                      \
      __builtin_amdgcn_s_waitcnt(0xC07F);                                                \
      SCHED();                                                                           \
      ringp = ring; ring = ring1;                                                        \
    } while (0)
    int s = 0;
    STEP(6, false, true);
    ++s;
#pragma unroll 1
    for (; s < 29; ++s) {
      if ((s & 7) == 3 && s < 27 && wave == 0)
        __builtin_amdgcn_global_load_lds(fl0 + ((s + 5) >> 3) * fl_stride, (lds_u32*)lds_flags, 4, 0, AUX_SYS);
      if ((s & 7) == 5) {
        if (!__all(lds_flags[lane] >= call)) {
          if (wave == 0) {
            const bool good = wave_wait_flags(fl0 + ((s + 3) >> 3) * fl_stride, true, call, tmo, call);
            if (!good && lane == 0) *lds_word = 0u;
          }
          RAW_BARRIER();
        }
        STAMP(2 + ((s + 3) >> 3));
      }
      STEP(6, true, true);
    }
    STEP(6, true, false);
    ++s;
    STEP(0, false, false);
#pragma unroll
    for (int i = 0; i < 8; ++i) rd(f1, ring, 1, i);
    MA(f0, 0, 0); MA(f0, 0, 1); MA(f0, 1, 0); MA(f0, 1, 1);
    MA(f1, 0, 0); MA(f1, 0, 1); MA(f1, 1, 0); MA(f1, 1, 1);
  }
  WAIT_VM(0);
  RAW_BARRIER();
  STAMP(6);

  const bool bad = *lds_word == 0u;
  {
    const float nanv = __builtin_nanf("");
#pragma unroll
    for (int i = 0; i < 2; ++i)
#pragma unroll
      for (int j = 0; j < 2; ++j) {
        const int row = wm * 64 + j * 32 + (lane & 31);
#pragma unroll
        for (int r = 0; r < 4; ++r) {
          f32x4 v;
#pragma unroll
          for (int t = 0; t < 4; ++t) {
            const float y = acc[i][j][r + 4 * t] * scale;
            v[t] = bad ? nanv : ((y <= 0.f) ? 0.f : y);
          }
          const int col = 16 * (r + 4 * (lane >> 5)) + 4 * (wn * 2 + i);
          *(f32x4*)(smem + row * EPI_PITCH + col * 4) = v;
        }
      }
  }
  const unsigned dest = (unsigned)(mt & 7);
  const int tile = (mt >> 3) * 16 + nt;
  RAW_BARRIER();
  STAMP(14);
  const unsigned obase = (unsigned)((mt >> 3) * 256) * (NTOT * 4u) + (unsigned)(nt * 128) * 4u + (lane & 31) * 16;
  if (dest == (unsigned)p.rank) {
    const auto dst = mk_rsrc(p.out[p.rank], 512u * NTOT * 4u);
#pragma unroll 4
    for (int it = 0; it < 16; ++it) {
      const int row = wave * 32 + it * 2 + (lane >> 5);
      const u32x4 v = *(const u32x4*)(smem + row * EPI_PITCH + (lane & 31) * 16);
      __builtin_amdgcn_raw_buffer_store_b128(v, dst, obase + (unsigned)(p.rank * NS * 4) + (unsigned)row * (NTOT * 4u), 0, AUX_OUT);
    }
    if (tid == 0) st_agent(ctr, call);
    STAMP(7);
    return;
  }
  {
    uint8_t* seg = smem + wave * 2 * ITEM_BYTES;
    u32x4 v[16];
#pragma unroll
    for (int blk = 0; blk < 16; ++blk)
      v[blk] = *(const u32x4*)(seg + (blk * 2 + (lane >> 5)) * EPI_PITCH + (lane & 31) * 16);
    WAIT_LGKM0();
    __builtin_amdgcn_sched_barrier(0);
    const auto stg = mk_rsrc(myws + OFF_STG + (dest * 32 + tile) * TILE_BYTES, TILE_BYTES);
#pragma unroll
    for (int hf = 0; hf < 2; ++hf) {
      uint8_t* it = seg + hf * ITEM_BYTES;
      unsigned base = 256;
      unsigned bits = 0;
#pragma unroll
      for (int e = 0; e < 32; ++e) {
        const unsigned val = v[hf * 8 + (e >> 2)][e & 3];
        const bool nz = val != 0u;
        const unsigned long long m = __ballot(nz);
        const unsigned pos = __builtin_amdgcn_mbcnt_hi((unsigned)(m >> 32), __builtin_amdgcn_mbcnt_lo((unsigned)m, 0u));
        *(unsigned*)(it + (nz ? base + pos * 4 : lane * 4u)) = val;
        base += (unsigned)__popcll(m) * 4u;
        bits |= nz ? (1u << e) : 0u;
      }
      *(unsigned*)(it + lane * 4) = bits;
      if (lane == 0) lds_word[16 + wave * 2 + hf] = (base - 256u) >> 2;
      const unsigned go = (unsigned)(wave * 2 + hf) * ITEM_BYTES;
#pragma unroll 3
      for (unsigned o = lane * 16u; o < base; o += 1024u)
        __builtin_amdgcn_raw_buffer_store_b128(*(const u32x4*)(it + o), stg, go + o, 0, AUX_SYS);
    }
    if (tid < 16) lds_word[32 + tid] = 0u;
  }
  WAIT_VM(0);
  RAW_BARRIER();
  STAMP(10);
  const unsigned tag = (call & 0x7FFFu) << 17;
  if (wave == 7) {
    (void)wave_wait_flags((const unsigned*)(myws + OFF_PD) + dest * 32, lane == 0, (call - 1u) * 32u + GATE_N, tmo, call);
    unsigned t = 0;
    if (lane == 0) t = __hip_atomic_fetch_add((unsigned*)(myws + OFF_TK) + dest * 32, 1u, __ATOMIC_RELAXED, __HIP_MEMORY_SCOPE_AGENT);
    t = (__builtin_amdgcn_readfirstlane(t) - (call - 1u) * 32u) & 31u;
    if (lane < 16) st_sys((unsigned*)(p.ws[dest] + OFF_SF) + (p.rank * 32 + t) * 16 + lane, tag | ((unsigned)tile << 12) | lds_word[16 + lane]);
    if (lane == 0) { st_agent(ctr, call); STAMPX(9); }
  }
  if (b < 144) return;

  {
    const int eng = wave < 4 ? (wave & 1) : ((wave - 4) >> 1);
    const int g = (b - 144) * 2 + eng;
    const unsigned src = (unsigned)(p.rank + 1 + g % 7) & 7u;
    volatile lds_u32* c = lds_word + 32 + eng * 8;
    uint8_t* ring = smem + eng * 65536;
    constexpr unsigned END = 0xFFFFFFFFu;
    if (wave < 2) {
      const uint8_t* rstg = p.ws[src] + OFF_STG + (unsigned)(p.rank * 32) * TILE_BYTES + lane * 16;
#pragma unroll 1
      for (unsigned k = 0;; ++k) {
        if (__builtin_amdgcn_readfirstlane(c[0]) <= k) {
          WAIT_VM(0);
          if (lane == 0) c[1] = k;
          lds_wait_ge(c, k + 1);
        }
        const unsigned d = __builtin_amdgcn_readfirstlane(c[4 + (k & 3)]);
        if (d == END) { WAIT_VM(0); if (lane == 0) c[1] = k; break; }
        const unsigned nbytes = 256u + (d & 0xFFFu) * 4u;
        const uint8_t* gsrc = rstg + ((d >> 12) & 511u) * ITEM_BYTES;
        uint8_t* l = ring + (k & 3) * ITEM_BYTES;
        const int nch = (int)((nbytes + 1023u) >> 10);
        const bool fast = (d >> 30) != 0u;
#pragma unroll 1
        for (int ch = 0; ch < nch; ++ch) {
          if ((unsigned)ch * 1024u + lane * 16u < nbytes)
            __builtin_amdgcn_global_load_lds((const unsigned*)(gsrc + ch * 1024), (lds_u32*)(l + ch * 1024), 16, 0, 17);
          if (ch == 0 || !fast) WAIT_VM(1);
          else asm volatile("s_waitcnt vmcnt(%0)" :: "i"(PULL_VM2) : "memory");
          if (ch == 0 && lane == 0) c[1] = k;
        }
        LOGB(myws, wave, k, src, nbytes);
      }
      STAMP(7);
    } else if (wave < 4) {
      unsigned* head = (unsigned*)(myws + OFF_H2) + ((call & 1u) * 8 + src) * 32;
      const unsigned* q = (const unsigned*)(myws + OFF_SF) + src * 512;
      unsigned i = (unsigned)(g / 7);
#pragma unroll 1
      for (unsigned k = 0;; ++k) {
        for (;;) {
          const unsigned u0 = c[2], u1 = c[3], ld = c[1];
          const unsigned lim = __builtin_amdgcn_readfirstlane(min(min(u0, u1) + 4u, ld + PULL_AHEAD));
          if (k < lim) break;
          __builtin_amdgcn_s_sleep(1);
        }
        if (k > 0) {
          unsigned t = 0;
          if (lane == 0) t = __hip_atomic_fetch_add(head, 1u, __ATOMIC_RELAXED, __HIP_MEMORY_SCOPE_AGENT);
          i = __builtin_amdgcn_readfirstlane(t) + 32u;
        }
        unsigned d = END;
        if (i < 512u) {
          const unsigned long long t0 = __builtin_amdgcn_s_memrealtime();
          for (unsigned spin = 1;; ++spin) {
            const unsigned w = __builtin_amdgcn_readfirstlane(ld_sys(q + i));
            if ((w & 0xFFFE0000u) == tag) { d = ((((w >> 12) & 31u) * 16u + (i & 15u)) << 12) | min(w & 0xFFFu, 2048u) | (i >= PULL_FAST ? 1u << 30 : 0u); break; }
            if ((spin & 7u) == 0u &&
                (ld_agent(tmo) == call || __builtin_amdgcn_s_memrealtime() - t0 > DEADLINE_TICKS)) {
              st_agent(tmo, call);
              if (lane == 0) p.out[p.rank][p.rank * NS] = __builtin_nanf("");
              break;
            }
            __builtin_amdgcn_s_sleep(1);
          }
        }
        if (lane == 0) c[4 + (k & 3)] = d;
        WAIT_LGKM0();
        if (lane == 0) c[0] = k + 1;
        if (d == END) break;
      }
    } else {
      const int u = wave & 1;
      const auto dst = mk_rsrc(p.out[p.rank], 512u * NTOT * 4u);
#pragma unroll 1
      for (unsigned k = 0;; ++k) {
        lds_wait_ge(c, k + 1);
        const unsigned d = __builtin_amdgcn_readfirstlane(c[4 + (k & 3)]);
        if (d == END) break;
        lds_wait_ge(c + 1, k + 1);
        const uint8_t* l = ring + (k & 3) * ITEM_BYTES;
        const unsigned bits = *(const unsigned*)(l + lane * 4);
        unsigned base = 256;
        if (u) {
          const unsigned below = __popc(bits & 0xFFFFu);
#pragma unroll
          for (int bt = 0; bt < 5; ++bt) base += (unsigned)__popcll(__ballot((below >> bt) & 1u)) << (bt + 2);
        }
        const unsigned mine = bits >> (16 * u);
        u32x4 v[4];
#pragma unroll
        for (int e = 0; e < 16; ++e) {
          const bool bit = (mine >> e) & 1u;
          const unsigned long long m = __ballot(bit);
          const unsigned pos = __builtin_amdgcn_mbcnt_hi((unsigned)(m >> 32), __builtin_amdgcn_mbcnt_lo((unsigned)m, 0u));
          const unsigned x = *(const unsigned*)(l + base + pos * 4u);
          v[e >> 2][e & 3] = bit ? x : 0u;
          base += (unsigned)__popcll(m) * 4u;
        }
        WAIT_LGKM0();
        if (lane == 0) c[2 + u] = k + 1;
        const unsigned tq = (d >> 12) & 511u, tl = tq >> 4;
        const unsigned ob = ((tl >> 4) * 256u + (tq & 15u) * 16u + u * 8u + (lane >> 5)) * (NTOT * 4u) +
                            (src * NS + (tl & 15u) * 128u) * 4u + (lane & 31) * 16u;
#pragma unroll
        for (int bb = 0; bb < 4; ++bb)
          __builtin_amdgcn_raw_buffer_store_b128(v[bb], dst, ob + bb * 2u * (NTOT * 4u), 0, AUX_OUT);
      }
      if (tid == 256) { STAMPX(8); }
    }
  }
}

}

extern "C" void kernel_launch(int rank, int nranks, ncclComm_t comm,
                              hipStream_t stream,
                              void* const* d_in, const int* in_sizes, int n_in,
                              void* d_out, int out_size,
                              const int* in_dtypes, int out_dtype,
                              void* const* d_out_all,
                              void* const* d_ws_all, size_t ws_bytes) {
  (void)comm; (void)in_sizes; (void)n_in; (void)d_out; (void)out_size; (void)in_dtypes; (void)out_dtype;
  if (nranks != NR || (unsigned)rank >= (unsigned)NR || ws_bytes < WS_NEED) {
    fprintf(stderr, "kernel_launch: unsupported configuration\n");
    return;
  }
  Params p;
  p.x = (const uint8_t*)d_in[0];
  p.w = (const uint8_t*)d_in[1];
  p.sx = (const float*)d_in[2];
  p.sw = (const float*)d_in[3];
  for (int i = 0; i < NR; ++i) {
    p.out[i] = (float*)d_out_all[i];
    p.ws[i] = (uint8_t*)d_ws_all[i];
  }
  p.rank = rank;
  hipLaunchKernelGGL(gemm_rs_kernel, dim3(NWG), dim3(NTHREADS), 0, stream, p);
  HIP_CHECK(hipGetLastError());
}
```

```cpp
#include <hip/hip_runtime.h>
#include <rccl/rccl.h>
#include <cstdio>
#include <cstdint>

#define HIP_CHECK(expr) do { hipError_t _e = (expr); if (_e != hipSuccess) { \
    fprintf(stderr, "HIP error %s at %s:%d\n", hipGetErrorString(_e), __FILE__, __LINE__); \
    return; } } while (0)

namespace {

typedef unsigned u32x4 __attribute__((ext_vector_type(4)));
typedef int i32x4 __attribute__((ext_vector_type(4)));
typedef float f32x16 __attribute__((ext_vector_type(16)));
typedef float f32x4 __attribute__((ext_vector_type(4)));
typedef __attribute__((address_space(3))) unsigned lds_u32;

constexpr int NR = 8;
constexpr int NTOT = 16384;
constexpr int KS = 512;
constexpr int NS = 2048;
constexpr unsigned UNIT = 16384;

constexpr unsigned OFF_CTR = 0;
constexpr unsigned OFF_TMO = 4096;
constexpr unsigned OFF_XF = 8192;
constexpr unsigned OFF_WF = 16384;
constexpr unsigned OFF_TK = 24576;
constexpr unsigned OFF_H2 = 26624;
constexpr unsigned OFF_PD = 28672;
constexpr unsigned OFF_HD = 30720;
constexpr unsigned OFF_SF = 32768;
constexpr unsigned OFF_DIAG = 65536;
constexpr unsigned OFF_DATA = 1u << 20;
constexpr unsigned X_BYTES = 16u << 20;
constexpr unsigned W_BYTES = 8u << 20;
constexpr unsigned PAR_BYTES = X_BYTES + W_BYTES;
constexpr unsigned ITEM_BYTES = 16 * 528;
constexpr unsigned TILE_BYTES = 16 * ITEM_BYTES;
constexpr unsigned OFF_STG = OFF_DATA + 2 * PAR_BYTES;
constexpr unsigned STG_BYTES = 8 * 32 * TILE_BYTES;
constexpr size_t WS_NEED = (size_t)OFF_STG + STG_BYTES;

constexpr int NWG = 256;
constexpr int NTHREADS = 512;
constexpr unsigned STAGE_BYTES = 3 * UNIT;
constexpr unsigned EPI_PITCH = 528;
constexpr unsigned LDS_MAIN = 3 * STAGE_BYTES;
constexpr unsigned LDS_BYTES = LDS_MAIN + 512;
constexpr unsigned long long DEADLINE_TICKS = 40000000ull;
constexpr int AUX_SYS = 17;
constexpr int AUX_OUT = 16;

#define PUSH_VM 1
#define PUSH_VM2 3
#define H_STOP 200
#define FREE_AHEAD 8
#define GATE_IT 2
#define GATE_N 28
#define PULL_VM2 5
#define PULL_FAST 496
#define PULL_AHEAD 3

struct Params {
  const uint8_t* x;
  const uint8_t* w;
  uint8_t* ws[NR];
  float* out;
  const float* sx;
  const float* sw;
  int rank;
};

__device__ __forceinline__ __amdgpu_buffer_rsrc_t mk_rsrc(const void* p, unsigned bytes) {
  return __builtin_amdgcn_make_buffer_rsrc((void*)p, 0, bytes, 0x00020000);
}
__device__ __forceinline__ unsigned ld_sys(const unsigned* p) {
  return __hip_atomic_load(p, __ATOMIC_RELAXED, __HIP_MEMORY_SCOPE_SYSTEM);
}
__device__ __forceinline__ void st_sys(unsigned* p, unsigned v) {
  __hip_atomic_store(p, v, __ATOMIC_RELAXED, __HIP_MEMORY_SCOPE_SYSTEM);
}
__device__ __forceinline__ unsigned ld_agent(const unsigned* p) {
  return __hip_atomic_load(p, __ATOMIC_RELAXED, __HIP_MEMORY_SCOPE_AGENT);
}
__device__ __forceinline__ void st_agent(unsigned* p, unsigned v) {
  __hip_atomic_store(p, v, __ATOMIC_RELAXED, __HIP_MEMORY_SCOPE_AGENT);
}
__device__ __forceinline__ unsigned add_agent(unsigned* p) {
  return __hip_atomic_fetch_add(p, 1u, __ATOMIC_RELAXED, __HIP_MEMORY_SCOPE_AGENT);
}
#define WAIT_VM(n) asm volatile("s_waitcnt vmcnt(%0)" :: "i"(n) : "memory")
#define WAIT_LGKM0() asm volatile("s_waitcnt lgkmcnt(0)" ::: "memory")
#define RAW_BARRIER() do { WAIT_LGKM0(); __builtin_amdgcn_s_barrier(); } while (0)
#define SCHED() __builtin_amdgcn_sched_barrier(0)
#define RFL(v) __builtin_amdgcn_readfirstlane(v)

__device__ __forceinline__ bool give_up(unsigned spin, unsigned long long t0, unsigned* tmo, unsigned call) {
  if ((spin & 7u) != 0u) return false;
  if (ld_agent(tmo) != call && __builtin_amdgcn_s_memrealtime() - t0 <= DEADLINE_TICKS) return false;
  st_agent(tmo, call);
  return true;
}
__device__ __forceinline__ bool wave_wait_flags(const unsigned* flag, bool active, unsigned target, unsigned* tmo, unsigned call) {
  const unsigned long long t0 = __builtin_amdgcn_s_memrealtime();
  for (unsigned spin = 1;; ++spin) {
    const unsigned v = active ? ld_sys(flag) : target;
    if (__all(v >= target)) return true;
    if (give_up(spin, t0, tmo, call)) return false;
    __builtin_amdgcn_s_sleep(2);
  }
}
__device__ __forceinline__ void lds_wait_ge(const volatile lds_u32* w, unsigned v) {
  while (RFL(*w) < v) __builtin_amdgcn_s_sleep(1);
  asm volatile("" ::: "memory");
}
__device__ __forceinline__ void arm_counter(unsigned* head, unsigned call, int lane) {
  if (lane == 0) (void)__hip_atomic_fetch_max(head, call << 10, __ATOMIC_RELAXED, __HIP_MEMORY_SCOPE_AGENT);
  WAIT_VM(0);
}
__device__ __forceinline__ unsigned draw(unsigned* head, int lane) {
  unsigned t = 0;
  if (lane == 0) t = add_agent(head);
  return RFL(t) & 1023u;
}

#define STAMPX(slot) do {} while (0)
#define STAMP(slot) do {} while (0)
#define STAMPW(slot) do {} while (0)
#define LOGB(slot, dest, bytes) do {} while (0)

__device__ __forceinline__ int x_row0(int mu) {
  const int mt = mu >> 1;
  return 512 * (mt & 7) + 256 * (mt >> 3) + 128 * (mu & 1);
}
__device__ __forceinline__ unsigned* xflag(uint8_t* dws, int rank, int mu, int kc, int sub) {
  return (unsigned*)(dws + OFF_XF) + (((mu >> 1) * 4 + kc) * 8 + rank) * 4 + (mu & 1) * 2 + sub;
}
__device__ __forceinline__ unsigned* wflag(uint8_t* dws, int rank, int nt, int kc) {
  return (unsigned*)(dws + OFF_WF) + (nt * 4 + kc) * 8 + rank;
}
template <int N>
__device__ __forceinline__ void glds_x(const Params& p, uint8_t* l, int mu, int kc, int lane, int i0) {
  const uint8_t* src = p.x + (unsigned)(x_row0(mu) + i0 * 8 + (lane >> 3)) * KS + kc * 128 + (((lane ^ (lane >> 3)) & 7) * 16);
#pragma unroll
  for (int i = 0; i < N; ++i)
    __builtin_amdgcn_global_load_lds((const unsigned*)(src + i * 8 * KS), (lds_u32*)(l + i * 1024), 16, 0, 0);
}
__device__ __forceinline__ void load_w_raw(u32x4 (&r)[16], const Params& p, unsigned dest, int nt, int kc, int lane) {
  const auto src = mk_rsrc(p.w, (unsigned)KS * NTOT);
  const unsigned base = (unsigned)(kc * 128 + (lane >> 3) * 16) * NTOT + dest * NS + nt * 128 + (lane & 7) * 16;
#pragma unroll
  for (int i = 0; i < 16; ++i) r[i] = __builtin_amdgcn_raw_buffer_load_b128(src, base + i * NTOT, 0, 0);
}
__device__ __forceinline__ void transpose_store(const u32x4 (&r)[16], uint8_t* l, int lane) {
#pragma unroll
  for (int d = 0; d < 4; ++d) {
    u32x4 o[4];
#pragma unroll
    for (int e = 0; e < 4; ++e) {
      const unsigned r0 = r[4 * e][d], r1 = r[4 * e + 1][d], r2 = r[4 * e + 2][d], r3 = r[4 * e + 3][d];
      const unsigned t0 = __builtin_amdgcn_perm(r1, r0, 0x05010400u);
      const unsigned t1 = __builtin_amdgcn_perm(r1, r0, 0x07030602u);
      const unsigned t2 = __builtin_amdgcn_perm(r3, r2, 0x05010400u);
      const unsigned t3 = __builtin_amdgcn_perm(r3, r2, 0x07030602u);
      o[0][e] = __builtin_amdgcn_perm(t2, t0, 0x05040100u);
      o[1][e] = __builtin_amdgcn_perm(t2, t0, 0x07060302u);
      o[2][e] = __builtin_amdgcn_perm(t3, t1, 0x05040100u);
      o[3][e] = __builtin_amdgcn_perm(t3, t1, 0x07060302u);
    }
#pragma unroll
    for (int q = 0; q < 4; ++q) *(u32x4*)(l + (4 * d + q) * 1024 + ((lane ^ ((q >> 1) * 8)) * 16)) = o[q];
  }
}

constexpr unsigned ITEMS = 256;
__device__ __forceinline__ unsigned push_loader(const Params& p, uint8_t* ring, volatile lds_u32* ctl, int rel, unsigned it,
                                                unsigned stop, const unsigned* ctr, int lane) {
  const unsigned dest = (unsigned)(p.rank + rel) & 7u;
  unsigned* head = (unsigned*)(p.ws[0] + OFF_HD) + rel * 32;
  unsigned call = 0;
#pragma unroll 1
  for (unsigned k = 0;; ++k) {
    uint8_t* l = ring + (k & 1) * UNIT;
    if (k > 0) {
      lds_wait_ge(ctl + 1, k);
      it = it >= stop ? ITEMS : draw(head, lane) + 32u;
    }
    const int kc = (it >> 5) & 3, slot = it & 31;
    if (it >= ITEMS) {
    } else if (it < 128 && slot >= 16) {
      u32x4 r[16];
      load_w_raw(r, p, dest, slot - 16, kc, lane);
      if (k == 0) __builtin_amdgcn_s_barrier();
      transpose_store(r, l, lane);
    } else {
      if (it < 128) glds_x<16>(p, l, slot, kc, lane, 0);
      else glds_x<8>(p, l, 16 + (slot >> 1), kc, lane, (slot & 1) * 8);
      if (k == 0) __builtin_amdgcn_s_barrier();
      WAIT_VM(0);
    }
    if (lane == 0) ctl[2 + (k & 1)] = it;
    WAIT_LGKM0();
    if (lane == 0) ctl[0] = k + 1;
    if (k == 0) {
      if (threadIdx.x == 256) STAMPX(12);
      call = RFL(ld_agent(ctr)) + 1u;
      arm_counter(head, call, lane);
    }
    if (it >= ITEMS) return call;
  }
}
template <int NCH, int WIN>
__device__ __forceinline__ void stream_item(const uint8_t* l, bool raw_x, __amdgpu_buffer_rsrc_t dst, unsigned off,
                                            unsigned* prev_flag, int prev_n, unsigned call, int lane,
                                            volatile lds_u32* req, unsigned req_val) {
  u32x4 v[NCH];
  unsigned a4[4];
#pragma unroll
  for (int q = 0; q < 4; ++q)
    a4[q] = raw_x ? (unsigned)((lane & 31) * 128 + (((q * 2 + (lane >> 5)) ^ lane) & 7) * 16) : (unsigned)(q * 1024 + lane * 16);
#pragma unroll
  for (int j = 0; j < NCH; ++j) v[j] = *(const u32x4*)(l + (j >> 2) * 4096 + a4[j & 3]);
#pragma unroll
  for (int j = 0; j < NCH; ++j) {
    __builtin_amdgcn_raw_buffer_store_b128(v[j], dst, off + j * 1024 + lane * 16, 0, AUX_SYS);
    WAIT_VM(WIN);
    if (j + 1 == WIN && lane < prev_n) st_sys(prev_flag + lane, call);
    if (j == NCH - FREE_AHEAD && lane == 0) *req = req_val;
  }
}
__device__ __forceinline__ void push_storer(const Params& p, const uint8_t* ring, volatile lds_u32* ctl, int rel,
                                            unsigned call, int lane) {
  uint8_t* dws = p.ws[rel];
  const auto dst = mk_rsrc(dws + OFF_DATA + (call & 1u) * PAR_BYTES, PAR_BYTES);
  unsigned* pf = nullptr;
  int pn = 0;
#pragma unroll 1
  for (unsigned k = 0;; ++k) {
    lds_wait_ge(ctl, k + 1);
    if (k == 0) STAMP(13);
    const unsigned it = RFL(ctl[2 + (k & 1)]);
    if (it >= ITEMS) break;
    const int kc = (it >> 5) & 3, slot = it & 31;
    const uint8_t* l = ring + (k & 1) * UNIT;
    if (it < 128) {
      const bool is_w = slot >= 16;
      const unsigned off = is_w ? X_BYTES + (unsigned)((p.rank * 4 + kc) * 16 + slot - 16) * UNIT
                                : (unsigned)((p.rank * 4 + kc) * 32 + slot) * UNIT;
      stream_item<16, PUSH_VM>(l, !is_w, dst, off, pf, pn, call, lane, ctl + 1, k + 1);
      pf = is_w ? wflag(dws, p.rank, slot - 16, kc) : xflag(dws, p.rank, slot, kc, 0);
      pn = is_w ? 1 : 2;
    } else {
      const int mu = 16 + (slot >> 1), sub = slot & 1;
      const unsigned off = (unsigned)((p.rank * 4 + kc) * 32 + mu) * UNIT + sub * 8192;
      if (it < H_STOP + 16) stream_item<8, PUSH_VM>(l, true, dst, off, pf, pn, call, lane, ctl + 1, k + 1);
      else stream_item<8, PUSH_VM2>(l, true, dst, off, pf, pn, call, lane, ctl + 1, k + 1);
      if (kc == GATE_IT && lane == 0) (void)add_agent((unsigned*)(p.ws[0] + OFF_PD) + rel * 32);
      pf = xflag(dws, p.rank, mu, kc, sub);
      pn = 1;
    }
    STAMPW(it >> 5);
    LOGB(k, rel, it < 128 ? 16384 : 8192);
  }
  WAIT_VM(0);
  if (lane < pn) st_sys(pf + lane, call);
}

__global__ void __launch_bounds__(NTHREADS, 2) gemm_rs_kernel(Params p) {
  __shared__ __attribute__((aligned(16))) uint8_t smem[LDS_BYTES];
  const int tid = threadIdx.x;
  const int lane = tid & 63;
  const int wave = RFL(tid >> 6);
  const int b = blockIdx.x;
  uint8_t* myws = p.ws[0];
  unsigned* tmo = (unsigned*)(myws + OFF_TMO);
  volatile lds_u32* lds_word = (volatile lds_u32*)(smem + LDS_MAIN);
  volatile lds_u32* lds_flags = lds_word + 64;

  STAMP(0);
  unsigned* ctr = (unsigned*)(myws + OFF_CTR) + b;
  unsigned call;

  if (b < 64) {
    const int pr = wave & 3, pw = (b < 56 ? b : b - 56) * 4 + pr;
    const int rel = b < 56 ? 1 + pw % 7 : 0;
    const unsigned first = (unsigned)(b < 56 ? pw / 7 : pw);
    volatile lds_u32* ctl = lds_word + 32 + pr * 4;
    uint8_t* ring = smem + pr * 2 * UNIT;
    if (wave >= 4) {
      call = push_loader(p, ring, ctl, rel, first, (b >= 32 && b < 56) ? H_STOP : ITEMS, ctr, lane);
    } else {
      const unsigned craw = ld_agent(ctr);
      if (lane < 4) ctl[lane] = 0u;
      RAW_BARRIER();
      call = RFL(craw) + 1u;
      push_storer(p, ring, ctl, rel, call, lane);
    }
  } else {
    call = RFL(ld_agent(ctr)) + 1u;
  }
  STAMP(1);

  int mrel, half, nt;
  {
    const int xcd = b & 7, ii = b >> 3;
    if (ii < 4) { half = ii >> 1; mrel = 0; nt = 2 * xcd + (ii & 1); }
    else {
      half = ii < 18 ? 1 : 0;
      const int j = ii < 18 ? ii - 4 : ii - 18, o = xcd & 1, g = xcd >> 1;
      if (j < 12) { mrel = (o ? 5 : 1) + j % 3; nt = 4 * g + j / 3; }
      else { mrel = 4; nt = 4 * g + 2 * o + (j - 12); }
    }
  }
  const unsigned dest = (unsigned)(p.rank + mrel) & 7u;
  const int mt = half * 8 + (int)dest;
  const int wm = wave & 3, wn = wave >> 2;
  const float scale = p.sx[0] * p.sw[0];

  const uint8_t* data = myws + OFF_DATA + (call & 1u) * PAR_BYTES;
  const unsigned xa_off = (unsigned)((wm >> 1) * UNIT + (wm & 1) * 2 * 4096 + lane * 16);
  const unsigned wb_off = 2 * UNIT + (unsigned)(wn * 2 * 4096 + ((lane & 31) >> 3) * 1024 +
                                                ((((lane >> 5) ^ (lane >> 4)) & 1) * 8 + (lane & 7)) * 16);
  f32x16 acc[2][2];
#pragma unroll
  for (int i = 0; i < 2; ++i)
#pragma unroll
    for (int j = 0; j < 2; ++j)
#pragma unroll
      for (int r = 0; r < 16; ++r) acc[i][j][r] = 0.f;

  const unsigned* fl0 = lane < 32 ? (const unsigned*)(myws + OFF_XF) + mt * 128 + lane
                                  : (const unsigned*)(myws + OFF_WF) + nt * 32 + (lane & 7);
  const int fl_stride = lane < 32 ? 32 : 8;
  const uint8_t* xbase = data + (unsigned)(2 * mt) * UNIT + wave * 4096 + lane * 16;
  const uint8_t* wbase = data + X_BYTES + (unsigned)nt * UNIT + wave * 2048 + lane * 16;
  auto glds1 = [&](int s, unsigned ring, int j) {
    const unsigned qk = (unsigned)(((p.rank + s) & 7) * 4 + (s >> 3));
    uint8_t* l = smem + ring;
    if (j < 4) __builtin_amdgcn_global_load_lds((const unsigned*)(xbase + qk * 32 * UNIT + j * 1024), (lds_u32*)(l + (wave * 4 + j) * 1024), 16, 0, 0);
    else __builtin_amdgcn_global_load_lds((const unsigned*)(wbase + qk * 16 * UNIT + (j - 4) * 1024), (lds_u32*)(l + 2 * UNIT + (wave * 2 + j - 4) * 1024), 16, 0, 0);
  };
  if (wave == 0) {
    const bool good = wave_wait_flags(fl0, true, call, tmo, call);
    __builtin_amdgcn_fence(__ATOMIC_ACQUIRE, "");
    WAIT_VM(0);
    if (lane == 0) *lds_word = good ? 1u : 0u;
  }
  RAW_BARRIER();
  STAMP(2);
  if (RFL(*lds_word) != 0u) {
    struct Frag { i32x4 r[8]; };
    auto rd = [&](Frag& f, unsigned ring, int kb, int i) {
      const unsigned o = (i < 2) ? wb_off + kb * 512 + i * 256
                       : (i < 6) ? xa_off + ((i - 2) >> 1) * 4096 + kb * 2048 + ((i - 2) & 1) * 1024
                                 : wb_off + 4096 + kb * 512 + (i - 6) * 256;
      f.r[i] = *(const i32x4*)(smem + ring + o);
    };
#define CAT(a, b) __builtin_shufflevector(a, b, 0, 1, 2, 3, 4, 5, 6, 7)
#define MA(f, i, j) acc[i][j] = __builtin_amdgcn_mfma_scale_f32_32x32x64_f8f6f4(                  \
      CAT(f.r[(i) * 6], f.r[(i) * 6 + 1]), CAT(f.r[2 + 2 * (j)], f.r[3 + 2 * (j)]), acc[i][j], 1, 1, 0, 0x7F7F7F7F, 0, 0x7F7F7F7F)
#pragma unroll
    for (int s = 0; s < 3; ++s)
#pragma unroll
      for (int j = 0; j < 6; ++j) glds1(s, s * STAGE_BYTES, j);
    WAIT_VM(12);
    RAW_BARRIER();
    Frag f0, f1;
#pragma unroll
    for (int i = 0; i < 8; ++i) rd(f0, 0, 0, i);
    __builtin_amdgcn_s_waitcnt(0xC07F);
    unsigned ring = 0, ringp = 2 * STAGE_BYTES;
#define STEP(WAITN, G2, G1) do {                                                         \
      const unsigned ring1 = ring == 2 * STAGE_BYTES ? 0u : ring + STAGE_BYTES;          \
      MA(f0, 0, 0); SCHED(); rd(f1, ring, 1, 0); rd(f1, ring, 1, 1); rd(f1, ring, 1, 2); rd(f1, ring, 1, 3); SCHED(); \
      MA(f0, 0, 1); SCHED(); rd(f1, ring, 1, 4); rd(f1, ring, 1, 5); rd(f1, ring, 1, 6); rd(f1, ring, 1, 7); SCHED(); \
      MA(f0, 1, 0); SCHED(); if (G2) { glds1(s + 2, ringp, 3); glds1(s + 2, ringp, 4); glds1(s + 2, ringp, 5); } SCHED(); \
      MA(f0, 1, 1); SCHED();                                                             \
      WAIT_VM(WAITN);                                                                    \
      RAW_BARRIER();            \
      MA(f1, 0, 0); SCHED(); rd(f0, ring1, 0, 0); rd(f0, ring1, 0, 1); rd(f0, ring1, 0, 2); rd(f0, ring1, 0, 3); SCHED(); \
      MA(f1, 0, 1); SCHED(); rd(f0, ring1, 0, 4); rd(f0, ring1, 0, 5); rd(f0, ring1, 0, 6); rd(f0, ring1, 0, 7); SCHED(); \
      MA(f1, 1, 0); SCHED(); if (G1) { glds1(s + 3, ring, 0); glds1(s + 3, ring, 1); glds1(s + 3, ring, 2); } SCHED(); \
      MA(f1, 1, 1);                                                                      \
      __builtin_amdgcn_s_waitcnt(0xC07F);                                                \
      SCHED();                                                                           \
      ringp = ring; ring = ring1;                                                        \
    } while (0)
    int s = 0;
    STEP(6, false, true);
    ++s;
#pragma unroll 1
    for (; s < 29; ++s) {
      if ((s & 7) == 3 && s < 27 && wave == 0)
        __builtin_amdgcn_global_load_lds(fl0 + ((s + 5) >> 3) * fl_stride, (lds_u32*)lds_flags, 4, 0, 17);
      if ((s & 7) == 5) {
        if (!__all(lds_flags[lane] >= call)) {
          if (wave == 0) {
            const bool good = wave_wait_flags(fl0 + ((s + 3) >> 3) * fl_stride, true, call, tmo, call);
            if (!good && lane == 0) *lds_word = 0u;
          }
          RAW_BARRIER();
        }
        STAMP(2 + ((s + 3) >> 3));
      }
      STEP(6, true, true);
    }
    STEP(6, true, false);
    ++s;
    STEP(0, false, false);
#pragma unroll
    for (int i = 0; i < 8; ++i) rd(f1, ring, 1, i);
    MA(f0, 0, 0); MA(f0, 0, 1); MA(f0, 1, 0); MA(f0, 1, 1);
    MA(f1, 0, 0); MA(f1, 0, 1); MA(f1, 1, 0); MA(f1, 1, 1);
  }
  RAW_BARRIER();
  STAMP(6);

  {
    const bool bad = *lds_word == 0u;
    const float nanv = __builtin_nanf("");
#pragma unroll
    for (int i = 0; i < 2; ++i)
#pragma unroll
      for (int j = 0; j < 2; ++j) {
        const int row = wm * 64 + j * 32 + (lane & 31);
#pragma unroll
        for (int r = 0; r < 4; ++r) {
          f32x4 v;
#pragma unroll
          for (int t = 0; t < 4; ++t) {
            const float y = acc[i][j][r + 4 * t] * scale;
            v[t] = bad ? nanv : ((y <= 0.f) ? 0.f : y);
          }
          const int col = 16 * (r + 4 * (lane >> 5)) + 4 * (wn * 2 + i);
          *(f32x4*)(smem + row * EPI_PITCH + col * 4) = v;
        }
      }
  }
  RAW_BARRIER();
  STAMP(14);
  const auto outr = mk_rsrc(p.out, 512u * NTOT * 4u);
  if (mrel == 0) {
    const unsigned ob = (unsigned)(half * 256) * (NTOT * 4u) + (unsigned)(p.rank * NS + nt * 128) * 4u + (lane & 31) * 16;
#pragma unroll 4
    for (int it = 0; it < 16; ++it) {
      const int row = wave * 32 + it * 2 + (lane >> 5);
      const u32x4 v = *(const u32x4*)(smem + row * EPI_PITCH + (lane & 31) * 16);
      __builtin_amdgcn_raw_buffer_store_b128(v, outr, ob + (unsigned)row * (NTOT * 4u), 0, AUX_OUT);
    }
    if (tid == 0) st_agent(ctr, call);
    STAMP(7);
    return;
  }

  const int tile = half * 16 + nt;
  {
    uint8_t* seg = smem + wave * 2 * ITEM_BYTES;
    u32x4 v[16];
#pragma unroll
    for (int blk = 0; blk < 16; ++blk)
      v[blk] = *(const u32x4*)(seg + (blk * 2 + (lane >> 5)) * EPI_PITCH + (lane & 31) * 16);
    WAIT_LGKM0();
    SCHED();
    const auto stg = mk_rsrc(myws + OFF_STG + (dest * 32 + tile) * TILE_BYTES, TILE_BYTES);
#pragma unroll
    for (int hf = 0; hf < 2; ++hf) {
      uint8_t* it = seg + hf * ITEM_BYTES;
      unsigned base = 256;
      unsigned bits = 0;
#pragma unroll
      for (int e = 0; e < 32; ++e) {
        const unsigned val = v[hf * 8 + (e >> 2)][e & 3];
        const bool nz = val != 0u;
        const unsigned long long m = __ballot(nz);
        const unsigned pos = __builtin_amdgcn_mbcnt_hi((unsigned)(m >> 32), __builtin_amdgcn_mbcnt_lo((unsigned)m, 0u));
        *(unsigned*)(it + (nz ? base + pos * 4 : lane * 4u)) = val;
        base += (unsigned)__popcll(m) * 4u;
        bits |= nz ? (1u << e) : 0u;
      }
      *(unsigned*)(it + lane * 4) = bits;
      if (lane == 0) lds_word[16 + wave * 2 + hf] = (base - 256u) >> 2;
      const unsigned go = (unsigned)(wave * 2 + hf) * ITEM_BYTES;
#pragma unroll 3
      for (unsigned o = lane * 16u; o < base; o += 1024u)
        __builtin_amdgcn_raw_buffer_store_b128(*(const u32x4*)(it + o), stg, go + o, 0, AUX_SYS);
    }
    if (tid < 16) lds_word[32 + tid] = 0u;
  }
  WAIT_VM(0);
  RAW_BARRIER();
  STAMP(10);
  const unsigned tag = (call & 0x7FFFu) << 17;
  if (wave == 7) {
    (void)wave_wait_flags((const unsigned*)(myws + OFF_PD) + mrel * 32, lane == 0, (call - 1u) * 32u + GATE_N, tmo, call);
    unsigned t = 0;
    if (lane == 0) t = add_agent((unsigned*)(myws + OFF_TK) + mrel * 32);
    t = RFL(t) & 31u;
    if (lane < 16) st_sys((unsigned*)(p.ws[mrel] + OFF_SF) + (p.rank * 32 + t) * 16 + lane, tag | ((unsigned)tile << 12) | lds_word[16 + lane]);
    if (lane == 0) { st_agent(ctr, call); STAMPX(9); }
  }
  if (b < 144) return;

  {
    const int eng = wave < 4 ? (wave & 1) : ((wave - 4) >> 1);
    const int g = (b - 144) * 2 + eng;
    const int rel = 1 + g % 7;
    const unsigned src = (unsigned)(p.rank + rel) & 7u;
    volatile lds_u32* c = lds_word + 32 + eng * 8;
    uint8_t* ring = smem + eng * 65536;
    constexpr unsigned END = 0xFFFFFFFFu;
    if (wave < 2) {
      const uint8_t* rstg = p.ws[rel] + OFF_STG + (unsigned)(p.rank * 32) * TILE_BYTES + lane * 16;
#pragma unroll 1
      for (unsigned k = 0;; ++k) {
        if (RFL(c[0]) <= k) {
          WAIT_VM(0);
          if (lane == 0) c[1] = k;
          lds_wait_ge(c, k + 1);
        }
        const unsigned d = RFL(c[4 + (k & 3)]);
        if (d == END) { WAIT_VM(0); if (lane == 0) c[1] = k; break; }
        const unsigned nbytes = 256u + (d & 0xFFFu) * 4u;
        const uint8_t* gsrc = rstg + ((d >> 12) & 511u) * ITEM_BYTES;
        uint8_t* l = ring + (k & 3) * ITEM_BYTES;
        const int nch = (int)((nbytes + 1023u) >> 10);
        const bool fast = (d >> 30) != 0u;
#pragma unroll 1
        for (int ch = 0; ch < nch; ++ch) {
          if ((unsigned)ch * 1024u + lane * 16u < nbytes)
            __builtin_amdgcn_global_load_lds((const unsigned*)(gsrc + ch * 1024), (lds_u32*)(l + ch * 1024), 16, 0, 17);
          if (ch == 0 || !fast) WAIT_VM(1); else WAIT_VM(PULL_VM2);
          if (ch == 0 && lane == 0) c[1] = k;
        }
        LOGB(k, rel, nbytes);
      }
      STAMP(7);
    } else if (wave < 4) {
      unsigned* head = (unsigned*)(myws + OFF_H2) + rel * 32;
      arm_counter(head, call, lane);
      const unsigned* q = (const unsigned*)(myws + OFF_SF) + src * 512;
      unsigned i = (unsigned)(g / 7);
#pragma unroll 1
      for (unsigned k = 0;; ++k) {
        for (;;) {
          const unsigned u0 = c[2], u1 = c[3], ld = c[1];
          if (k < RFL(min(min(u0, u1) + 4u, ld + PULL_AHEAD))) break;
          __builtin_amdgcn_s_sleep(1);
        }
        if (k > 0) i = draw(head, lane) + 32u;
        unsigned d = END;
        if (i < 512u) {
          const unsigned long long t0 = __builtin_amdgcn_s_memrealtime();
          for (unsigned spin = 1;; ++spin) {
            const unsigned w = RFL(ld_sys(q + i));
            if ((w & 0xFFFE0000u) == tag) {
              d = ((((w >> 12) & 31u) * 16u + (i & 15u)) << 12) | min(w & 0xFFFu, 2048u) | (i >= PULL_FAST ? 1u << 30 : 0u);
              break;
            }
            if (give_up(spin, t0, tmo, call)) {
              if (lane == 0) p.out[p.rank * NS] = __builtin_nanf("");
              break;
            }
            __builtin_amdgcn_s_sleep(1);
          }
        }
        if (lane == 0) c[4 + (k & 3)] = d;
        WAIT_LGKM0();
        if (lane == 0) c[0] = k + 1;
        if (d == END) break;
      }
    } else {
      const int u = wave & 1;
#pragma unroll 1
      for (unsigned k = 0;; ++k) {
        lds_wait_ge(c, k + 1);
        const unsigned d = RFL(c[4 + (k & 3)]);
        if (d == END) break;
        lds_wait_ge(c + 1, k + 1);
        const uint8_t* l = ring + (k & 3) * ITEM_BYTES;
        const unsigned bits = *(const unsigned*)(l + lane * 4);
        unsigned base = 256;
        if (u) {
          const unsigned below = __popc(bits & 0xFFFFu);
#pragma unroll
          for (int bt = 0; bt < 5; ++bt) base += (unsigned)__popcll(__ballot((below >> bt) & 1u)) << (bt + 2);
        }
        const unsigned mine = bits >> (16 * u);
        u32x4 v[4];
#pragma unroll
        for (int e = 0; e < 16; ++e) {
          const bool bit = (mine >> e) & 1u;
          const unsigned long long m = __ballot(bit);
          const unsigned pos = __builtin_amdgcn_mbcnt_hi((unsigned)(m >> 32), __builtin_amdgcn_mbcnt_lo((unsigned)m, 0u));
          const unsigned x = *(const unsigned*)(l + base + pos * 4u);
          v[e >> 2][e & 3] = bit ? x : 0u;
          base += (unsigned)__popcll(m) * 4u;
        }
        WAIT_LGKM0();
        if (lane == 0) c[2 + u] = k + 1;
        const unsigned tq = (d >> 12) & 511u, tl = tq >> 4;
        const unsigned ob = ((tl >> 4) * 256u + (tq & 15u) * 16u + u * 8u + (lane >> 5)) * (NTOT * 4u) +
                            (src * NS + (tl & 15u) * 128u) * 4u + (lane & 31) * 16u;
#pragma unroll
        for (int bb = 0; bb < 4; ++bb)
          __builtin_amdgcn_raw_buffer_store_b128(v[bb], outr, ob + bb * 2u * (NTOT * 4u), 0, AUX_OUT);
      }
      if (tid == 256) STAMPX(8);
    }
  }
}

}

extern "C" void kernel_launch(int rank, int nranks, ncclComm_t comm,
                              hipStream_t stream,
                              void* const* d_in, const int* in_sizes, int n_in,
                              void* d_out, int out_size,
                              const int* in_dtypes, int out_dtype,
                              void* const* d_out_all,
                              void* const* d_ws_all, size_t ws_bytes) {
  (void)comm; (void)in_sizes; (void)in_dtypes; (void)out_dtype; (void)d_out_all;
  if (nranks != NR || (unsigned)rank >= (unsigned)NR || ws_bytes < WS_NEED || n_in < 4 || out_size != 512 * NTOT) {
    fprintf(stderr, "kernel_launch: unsupported configuration\n");
    return;
  }
  Params p;
  p.x = (const uint8_t*)d_in[0];
  p.w = (const uint8_t*)d_in[1];
  p.sx = (const float*)d_in[2];
  p.sw = (const float*)d_in[3];
  for (int i = 0; i < NR; ++i) p.ws[i] = (uint8_t*)d_ws_all[(rank + i) % NR];
  p.out = (float*)d_out;
  p.rank = rank;
  hipLaunchKernelGGL(gemm_rs_kernel, dim3(NWG), dim3(NTHREADS), 0, stream, p);
  HIP_CHECK(hipGetLastError());
}
```

```cpp
#include <hip/hip_runtime.h>
#include <rccl/rccl.h>
#include <cstdio>
#include <cstdint>

#define HIP_CHECK(expr) do { hipError_t _e = (expr); if (_e != hipSuccess) { \
    fprintf(stderr, "HIP error %s at %s:%d\n", hipGetErrorString(_e), __FILE__, __LINE__); \
    return; } } while (0)

namespace {

typedef unsigned u32x4 __attribute__((ext_vector_type(4)));
typedef int i32x4 __attribute__((ext_vector_type(4)));
typedef float f32x16 __attribute__((ext_vector_type(16)));
typedef float f32x4 __attribute__((ext_vector_type(4)));
typedef __attribute__((address_space(3))) unsigned lds_u32;

constexpr int NR = 8;
constexpr int NTOT = 16384;
constexpr int KS = 512;
constexpr int NS = 4096;
constexpr unsigned UNIT = 16384;

constexpr unsigned OFF_CTR = 0;
constexpr unsigned OFF_TMO = 4096;
constexpr unsigned OFF_XF = 8192;
constexpr unsigned OFF_WF = 16384;
constexpr unsigned OFF_FF = 20480;
constexpr unsigned OFF_TK = 24576;
constexpr unsigned OFF_H2 = 26624;
constexpr unsigned OFF_PD = 28672;
constexpr unsigned OFF_HD = 30720;
constexpr unsigned OFF_SF = 32768;
constexpr unsigned OFF_DIAG = 65536;
constexpr unsigned OFF_DATA = 1u << 20;
constexpr unsigned X_BYTES = 8u << 20;
constexpr unsigned W_BYTES = 16u << 20;
constexpr unsigned PAR_BYTES = X_BYTES + W_BYTES;
constexpr unsigned ITEM_BYTES = 16 * 528;
constexpr unsigned TILE_BYTES = 16 * ITEM_BYTES;
constexpr unsigned OFF_STG = OFF_DATA + 2 * PAR_BYTES;
constexpr unsigned STG_BYTES = 8 * 32 * TILE_BYTES;
constexpr size_t WS_NEED = (size_t)OFF_STG + STG_BYTES;

constexpr int NWG = 256;
constexpr int NTHREADS = 512;
constexpr unsigned STAGE_BYTES = 3 * UNIT;
constexpr unsigned EPI_PITCH = 528;
constexpr unsigned LDS_MAIN = 3 * STAGE_BYTES;
constexpr unsigned LDS_BYTES = LDS_MAIN + 512;
constexpr unsigned long long DEADLINE_TICKS = 40000000ull;
constexpr int AUX_SYS = 17;
constexpr int AUX_OUT = 16;

#define PUSH_VM 2
#define PUSH_VM2 3
#define PUSH_LAST 8
#define NF 2
#define FREE_AHEAD 12
#define NPAIR 2
#define COPY_GATE 31
#define COPY_VM 3
#define GATE_N 16
#define PULL_VM2 5
#define PULL_FAST 496
#define PULL_AHEAD 3

struct Params {
  const uint8_t* x;
  const uint8_t* w;
  uint8_t* ws[NR];
  float* out;
  const float* sx;
  const float* sw;
  int rank;
  unsigned tilemap[64];
};

__device__ __forceinline__ __amdgpu_buffer_rsrc_t mk_rsrc(const void* p, unsigned bytes) {
  return __builtin_amdgcn_make_buffer_rsrc((void*)p, 0, bytes, 0x00020000);
}
__device__ __forceinline__ unsigned ld_sys(const unsigned* p) {
  return __hip_atomic_load(p, __ATOMIC_RELAXED, __HIP_MEMORY_SCOPE_SYSTEM);
}
__device__ __forceinline__ void st_sys(unsigned* p, unsigned v) {
  __hip_atomic_store(p, v, __ATOMIC_RELAXED, __HIP_MEMORY_SCOPE_SYSTEM);
}
__device__ __forceinline__ unsigned ld_agent(const unsigned* p) {
  return __hip_atomic_load(p, __ATOMIC_RELAXED, __HIP_MEMORY_SCOPE_AGENT);
}
__device__ __forceinline__ void st_agent(unsigned* p, unsigned v) {
  __hip_atomic_store(p, v, __ATOMIC_RELAXED, __HIP_MEMORY_SCOPE_AGENT);
}
__device__ __forceinline__ unsigned add_agent(unsigned* p) {
  return __hip_atomic_fetch_add(p, 1u, __ATOMIC_RELAXED, __HIP_MEMORY_SCOPE_AGENT);
}
#define WAIT_VM(n) asm volatile("s_waitcnt vmcnt(%0)" :: "i"(n) : "memory")
#define WAIT_LGKM0() asm volatile("s_waitcnt lgkmcnt(0)" ::: "memory")
#define RAW_BARRIER() do { WAIT_LGKM0(); __builtin_amdgcn_s_barrier(); } while (0)
#define SCHED() __builtin_amdgcn_sched_barrier(0)
#define RFL(v) __builtin_amdgcn_readfirstlane(v)

__device__ __forceinline__ bool give_up(unsigned spin, unsigned long long t0, unsigned* tmo, unsigned call) {
  if ((spin & 7u) != 0u) return false;
  if (ld_agent(tmo) != call && __builtin_amdgcn_s_memrealtime() - t0 <= DEADLINE_TICKS) return false;
  st_agent(tmo, call);
  return true;
}
__device__ __forceinline__ bool wave_wait_flags(const unsigned* flag, bool active, unsigned target, unsigned* tmo, unsigned call) {
  const unsigned long long t0 = __builtin_amdgcn_s_memrealtime();
  for (unsigned spin = 1;; ++spin) {
    const unsigned v = active ? ld_sys(flag) : target;
    if (__all(v >= target)) return true;
    if (give_up(spin, t0, tmo, call)) return false;
    __builtin_amdgcn_s_sleep(2);
  }
}
__device__ __forceinline__ void lds_wait_ge(const volatile lds_u32* w, unsigned v) {
  while (RFL(*w) < v) __builtin_amdgcn_s_sleep(1);
  asm volatile("" ::: "memory");
}
__device__ __forceinline__ void arm_counter(unsigned* head, unsigned call, int lane) {
  if (lane == 0) (void)__hip_atomic_fetch_max(head, call << 10, __ATOMIC_RELAXED, __HIP_MEMORY_SCOPE_AGENT);
  WAIT_VM(0);
}
__device__ __forceinline__ unsigned draw(unsigned* head, int lane) {
  unsigned t = 0;
  if (lane == 0) t = add_agent(head);
  return RFL(t) & 1023u;
}

#define STAMPX(slot) do {} while (0)
#define STAMP(slot) do {} while (0)
#define STAMPW(slot) do {} while (0)
#define LOGB(slot, dest, bytes) do {} while (0)

__device__ __forceinline__ int x_row0(int d, int mu) {
  const int arel = mu >> 1, e = d & 1;
  return (arel >= 2 ? 512 * (d ^ arel) + 256 * e : 512 * d + 256 * (e ^ arel)) + 128 * (mu & 1);
}
__device__ __forceinline__ unsigned* xflag(uint8_t* dws, int rank, int mu, int kc) {
  return (unsigned*)(dws + OFF_XF) + (((mu >> 1) * 4 + kc) * 8 + rank) * 2 + (mu & 1);
}
__device__ __forceinline__ unsigned* wflag(uint8_t* dws, int rank, int nt, int kc) {
  return (unsigned*)(dws + OFF_WF) + (nt * 4 + kc) * 8 + rank;
}
__device__ __forceinline__ void glds_x(const Params& p, uint8_t* l, int row0, int kc, int lane) {
  const uint8_t* src = p.x + (unsigned)(row0 + (lane >> 3)) * KS + kc * 128 + (((lane ^ (lane >> 3)) & 7) * 16);
#pragma unroll
  for (int i = 0; i < 16; ++i)
    __builtin_amdgcn_global_load_lds((const unsigned*)(src + i * 8 * KS), (lds_u32*)(l + i * 1024), 16, 0, 0);
}
__device__ __forceinline__ void load_w_raw(u32x4 (&r)[16], const Params& p, int d, int nt, int kc, int lane) {
  const auto src = mk_rsrc(p.w, (unsigned)KS * NTOT);
  const unsigned base = (unsigned)(kc * 128 + (lane >> 3) * 16) * NTOT + (d >> 1) * NS + nt * 128 + (lane & 7) * 16;
#pragma unroll
  for (int i = 0; i < 16; ++i) r[i] = __builtin_amdgcn_raw_buffer_load_b128(src, base + i * NTOT, 0, 0);
}
__device__ __forceinline__ void transpose_store(const u32x4 (&r)[16], uint8_t* l, int lane) {
#pragma unroll
  for (int d = 0; d < 4; ++d) {
    u32x4 o[4];
#pragma unroll
    for (int e = 0; e < 4; ++e) {
      const unsigned r0 = r[4 * e][d], r1 = r[4 * e + 1][d], r2 = r[4 * e + 2][d], r3 = r[4 * e + 3][d];
      const unsigned t0 = __builtin_amdgcn_perm(r1, r0, 0x05010400u);
      const unsigned t1 = __builtin_amdgcn_perm(r1, r0, 0x07030602u);
      const unsigned t2 = __builtin_amdgcn_perm(r3, r2, 0x05010400u);
      const unsigned t3 = __builtin_amdgcn_perm(r3, r2, 0x07030602u);
      o[0][e] = __builtin_amdgcn_perm(t2, t0, 0x05040100u);
      o[1][e] = __builtin_amdgcn_perm(t2, t0, 0x07060302u);
      o[2][e] = __builtin_amdgcn_perm(t3, t1, 0x05040100u);
      o[3][e] = __builtin_amdgcn_perm(t3, t1, 0x07060302u);
    }
#pragma unroll
    for (int q = 0; q < 4; ++q) *(u32x4*)(l + (4 * d + q) * 1024 + ((lane ^ ((q >> 1) * 8)) * 16)) = o[q];
  }
}

constexpr int NB = 16 - NF;
constexpr unsigned IT_B = 120;
constexpr unsigned NSTAT = 8 * NPAIR;
struct Item { bool is_w, dual; int idx, kc; };
__device__ __forceinline__ unsigned n_items(int rel) { return IT_B + (rel < 2 ? 64u : 4u * NB) + 8u; }
__device__ __forceinline__ Item decode_item(unsigned it, int rel, int d) {
  Item r;
  r.dual = false;
  const unsigned it_c = n_items(rel) - 8u;
  if (it < IT_B) {
    const unsigned sl = it % 30u;
    r.kc = it / 30u; r.is_w = sl >= 14u; r.idx = r.is_w ? sl - 14u : (sl < 2u ? sl : sl + 2u);
  } else if (it < it_c) {
    const unsigned i = it - IT_B;
    r.is_w = true;
    if (rel < 2) { r.kc = i >> 4; r.idx = 16 + (i & 15u); }
    else {
      const unsigned c = i % NB;
      r.kc = i / NB;
      r.idx = c >= NF ? 16 + (c - NF) : 32 - NF * (2 - (d & 1)) + c;
      r.dual = c < NF;
    }
  } else {
    const unsigned i = it - it_c;
    r.kc = i >> 1; r.is_w = false; r.idx = 2 + (i & 1u);
  }
  return r;
}
__device__ __forceinline__ unsigned push_loader(const Params& p, uint8_t* ring, volatile lds_u32* ctl, int rel, unsigned it,
                                                const unsigned* ctr, int lane) {
  const int d = (p.rank ^ rel) & 7;
  const unsigned nit = n_items(rel);
  unsigned* head = (unsigned*)(p.ws[0] + OFF_HD) + rel * 32;
  unsigned call = 0;
#pragma unroll 1
  for (unsigned k = 0;; ++k) {
    uint8_t* l = ring + (k & 1) * UNIT;
    if (k > 0) {
      lds_wait_ge(ctl + 1, k);
      it = draw(head, lane) + NSTAT;
    }
    if (it < nit) {
      const Item m = decode_item(it, rel, d);
      if (m.is_w) {
        u32x4 r[16];
        load_w_raw(r, p, d, m.idx, m.kc, lane);
        if (k == 0) __builtin_amdgcn_s_barrier();
        transpose_store(r, l, lane);
      } else {
        glds_x(p, l, x_row0(d, m.idx), m.kc, lane);
        if (k == 0) __builtin_amdgcn_s_barrier();
        WAIT_VM(0);
      }
    }
    if (lane == 0) ctl[2 + (k & 1)] = it;
    WAIT_LGKM0();
    if (lane == 0) ctl[0] = k + 1;
    if (k == 0) {
      if (threadIdx.x == 256) STAMPX(12);
      call = RFL(ld_agent(ctr)) + 1u;
      arm_counter(head, call, lane);
    }
    if (it >= nit) return call;
  }
}
template <int WIN>
__device__ __forceinline__ void stream_item(const uint8_t* l, bool raw_x, __amdgpu_buffer_rsrc_t dst, unsigned off,
                                            unsigned* prev_flag, int prev_n, unsigned call, int lane,
                                            volatile lds_u32* req, unsigned req_val) {
  u32x4 v[16];
  unsigned a4[4];
#pragma unroll
  for (int q = 0; q < 4; ++q)
    a4[q] = raw_x ? (unsigned)((lane & 31) * 128 + (((q * 2 + (lane >> 5)) ^ lane) & 7) * 16) : (unsigned)(q * 1024 + lane * 16);
#pragma unroll
  for (int j = 0; j < 16; ++j) v[j] = *(const u32x4*)(l + (j >> 2) * 4096 + a4[j & 3]);
#pragma unroll
  for (int j = 0; j < 16; ++j) {
    __builtin_amdgcn_raw_buffer_store_b128(v[j], dst, off + j * 1024 + lane * 16, 0, AUX_SYS);
    WAIT_VM(WIN);
    if (j + 1 == WIN && lane < prev_n) st_sys(prev_flag, call);
    if (j == 16 - FREE_AHEAD && lane == 0) *req = req_val;
  }
}
__device__ __forceinline__ void push_storer(const Params& p, const uint8_t* ring, volatile lds_u32* ctl, int rel,
                                            unsigned call, int lane) {
  const int d = (p.rank ^ rel) & 7;
  const unsigned nit = n_items(rel);
  uint8_t* dws = p.ws[rel];
  unsigned* ff = (unsigned*)(p.ws[rel ^ 1] + OFF_FF) + p.rank;
  const auto dst = mk_rsrc(dws + OFF_DATA + (call & 1u) * PAR_BYTES, PAR_BYTES);
  unsigned* pf = nullptr;
  int pn = 0;
#pragma unroll 1
  for (unsigned k = 0;; ++k) {
    lds_wait_ge(ctl, k + 1);
    if (k == 0) STAMP(13);
    const unsigned it = RFL(ctl[2 + (k & 1)]);
    if (it >= nit) break;
    const Item m = decode_item(it, rel, d);
    const uint8_t* l = ring + (k & 1) * UNIT;
    const unsigned off = m.is_w ? X_BYTES + (unsigned)((p.rank * 4 + m.kc) * 32 + m.idx) * UNIT
                                : (unsigned)((p.rank * 4 + m.kc) * 16 + m.idx) * UNIT;
    if (it + PUSH_LAST < nit) stream_item<PUSH_VM>(l, !m.is_w, dst, off, pf, pn, call, lane, ctl + 1, k + 1);
    else stream_item<PUSH_VM2>(l, !m.is_w, dst, off, pf, pn, call, lane, ctl + 1, k + 1);
    if (it + 32u >= nit && lane == 0) (void)add_agent((unsigned*)(p.ws[0] + OFF_PD) + rel * 32);
    pf = m.is_w ? wflag(dws, p.rank, m.idx, m.kc) : xflag(dws, p.rank, m.idx, m.kc);
    if (m.dual && lane == 1) pf = ff + (m.kc * NF + (m.idx - (32 - NF * (2 - (d & 1))))) * 8;
    pn = m.dual ? 2 : 1;
    STAMPW(it >> 5);
    LOGB(k, rel, 16384);
  }
  WAIT_VM(0);
  if (lane < pn) st_sys(pf, call);
}

constexpr unsigned N_COPY = 4 * NF * 6;
__device__ __forceinline__ unsigned copy_off(const Params& p, unsigned it, int& nt, int& kc, int& srank) {
  kc = it / (NF * 6);
  const unsigned r = it % (NF * 6);
  nt = 32 - NF * ((p.rank & 1) + 1) + r / 6;
  srank = (p.rank ^ (2 + r % 6)) & 7;
  return X_BYTES + (unsigned)((srank * 4 + kc) * 32 + nt) * UNIT;
}
__device__ __forceinline__ void copy_loader(const Params& p, uint8_t* ring, volatile lds_u32* ctl, unsigned it, unsigned call,
                                            unsigned* tmo, int lane) {
  unsigned* head = (unsigned*)(p.ws[0] + OFF_H2) + 32;
  arm_counter(head, call, lane);
  const uint8_t* rdata = p.ws[1] + OFF_DATA + (call & 1u) * PAR_BYTES + lane * 16;
  (void)wave_wait_flags(wflag(p.ws[0], p.rank ^ 1, COPY_GATE, 3), lane == 0, call, tmo, call);
#pragma unroll 1
  for (unsigned k = 0;; ++k) {
    if (k > 0) it = draw(head, lane) + NSTAT;
    if (it < N_COPY) {
      int nt, kc, srank;
      const unsigned off = copy_off(p, it, nt, kc, srank);
      const unsigned* ff = (const unsigned*)(p.ws[0] + OFF_FF) + (kc * NF + (it % (NF * 6)) / 6) * 8 + srank;
      if (!wave_wait_flags(ff, lane == 0, call, tmo, call)) it = N_COPY;
      else {
        if (k >= 2) lds_wait_ge(ctl + 1, k - 1);
        uint8_t* l = ring + (k & 1) * UNIT;
#pragma unroll
        for (int j = 0; j < 16; ++j) {
          __builtin_amdgcn_global_load_lds((const unsigned*)(rdata + off + j * 1024), (lds_u32*)(l + j * 1024), 16, 0, 17);
          WAIT_VM(COPY_VM);
        }
        WAIT_VM(0);
        LOGB(8 + k, 1, 16384);
      }
    }
    if (k >= 2) lds_wait_ge(ctl + 1, k - 1);
    if (lane == 0) ctl[2 + (k & 1)] = it;
    WAIT_LGKM0();
    if (lane == 0) ctl[0] = k + 1;
    if (it >= N_COPY) return;
  }
}
__device__ __forceinline__ void copy_storer(const Params& p, const uint8_t* ring, volatile lds_u32* ctl, unsigned call, int lane) {
  const auto dst = mk_rsrc(p.ws[0] + OFF_DATA + (call & 1u) * PAR_BYTES, PAR_BYTES);
#pragma unroll 1
  for (unsigned k = 0;; ++k) {
    lds_wait_ge(ctl, k + 1);
    const unsigned it = RFL(ctl[2 + (k & 1)]);
    if (it >= N_COPY) return;
    int nt, kc, srank;
    const unsigned off = copy_off(p, it, nt, kc, srank);
    const uint8_t* l = ring + (k & 1) * UNIT;
    u32x4 v[16];
#pragma unroll
    for (int j = 0; j < 16; ++j) v[j] = *(const u32x4*)(l + j * 1024 + lane * 16);
    WAIT_LGKM0();
    if (lane == 0) ctl[1] = k + 1;
#pragma unroll
    for (int j = 0; j < 16; ++j) __builtin_amdgcn_raw_buffer_store_b128(v[j], dst, off + j * 1024 + lane * 16, 0, AUX_SYS);
    WAIT_VM(0);
    if (lane == 0) st_sys(wflag(p.ws[0], srank, nt, kc), call);
  }
}

__global__ void __launch_bounds__(NTHREADS, 2) gemm_rs_kernel(Params p) {
  __shared__ __attribute__((aligned(16))) uint8_t smem[LDS_BYTES];
  const int tid = threadIdx.x;
  const int lane = tid & 63;
  const int wave = RFL(tid >> 6);
  const int b = blockIdx.x;
  uint8_t* myws = p.ws[0];
  unsigned* tmo = (unsigned*)(myws + OFF_TMO);
  volatile lds_u32* lds_word = (volatile lds_u32*)(smem + LDS_MAIN);
  volatile lds_u32* lds_flags = lds_word + 64;

  STAMP(0);
  unsigned* ctr = (unsigned*)(myws + OFF_CTR) + b;
  unsigned call;

  if (b < 64) {
    const int pr = wave & 3, pw = (b < 56 ? b : b - 56) * NPAIR + pr;
    const int rel = b < 56 ? 1 + pw % 7 : 0;
    const unsigned first = (unsigned)(b < 56 ? pw / 7 : pw);
    volatile lds_u32* ctl = lds_word + 32 + pr * 4;
    uint8_t* ring = smem + pr * 2 * UNIT;
    if (pr >= NPAIR) {
      __builtin_amdgcn_s_barrier();
      call = RFL(ld_agent(ctr)) + 1u;
    } else if (wave >= 4) {
      call = push_loader(p, ring, ctl, rel, first, ctr, lane);
    } else {
      const unsigned craw = ld_agent(ctr);
      if (lane < 4) ctl[lane] = 0u;
      RAW_BARRIER();
      call = RFL(craw) + 1u;
      push_storer(p, ring, ctl, rel, call, lane);
    }
  } else {
    call = RFL(ld_agent(ctr)) + 1u;
  }
  STAMP(1);

  const unsigned tm = (p.tilemap[b >> 2] >> ((b & 3) * 8)) & 255u;
  const int arel = tm >> 5, nt = tm & 31;
  const int par = p.rank & 1;
  const int wm = wave & 3, wn = wave >> 2;
  const float scale = p.sx[0] * p.sw[0];

  const uint8_t* data = myws + OFF_DATA + (call & 1u) * PAR_BYTES;
  const unsigned xa_off = (unsigned)((wm >> 1) * UNIT + (wm & 1) * 2 * 4096 + lane * 16);
  const unsigned wb_off = 2 * UNIT + (unsigned)(wn * 2 * 4096 + ((lane & 31) >> 3) * 1024 +
                                                ((((lane >> 5) ^ (lane >> 4)) & 1) * 8 + (lane & 7)) * 16);
  f32x16 acc[2][2];
#pragma unroll
  for (int i = 0; i < 2; ++i)
#pragma unroll
    for (int j = 0; j < 2; ++j)
#pragma unroll
      for (int r = 0; r < 16; ++r) acc[i][j][r] = 0.f;

  const bool far_w = (unsigned)(31 - par * NF - nt) < (unsigned)NF;
  auto step_qk = [&](int s) -> unsigned {
    const int f = s - 8, fk = (f * 43) >> 8;
    const int sr = far_w ? (s < 8 ? (s & 1) : 2 + f - fk * 6) : (s & 7);
    const int kc = far_w ? (s < 8 ? (s >> 1) : fk) : (s >> 3);
    return (unsigned)((p.rank ^ sr) * 4 + kc);
  };
  auto flag_ptr = [&](int g) -> const unsigned* {
    const unsigned qk = step_qk(g * 8 + (lane < 16 ? lane >> 1 : lane & 7));
    const unsigned q = qk >> 2, kc = qk & 3u;
    return lane < 16 ? (const unsigned*)(myws + OFF_XF) + ((arel * 4 + kc) * 8 + q) * 2 + (lane & 1)
                     : (const unsigned*)(myws + OFF_WF) + (nt * 4 + kc) * 8 + q;
  };
  const uint8_t* xbase = data + (unsigned)(2 * arel) * UNIT + wave * 4096 + lane * 16;
  const uint8_t* wbase = data + X_BYTES + (unsigned)nt * UNIT + wave * 2048 + lane * 16;
  const unsigned qtab = step_qk(lane & 31);
  auto glds1 = [&](int s, unsigned ring, int j) {
    const unsigned qk = __builtin_amdgcn_readlane(qtab, s);
    uint8_t* l = smem + ring;
    if (j < 4) __builtin_amdgcn_global_load_lds((const unsigned*)(xbase + qk * 16 * UNIT + j * 1024), (lds_u32*)(l + (wave * 4 + j) * 1024), 16, 0, 0);
    else __builtin_amdgcn_global_load_lds((const unsigned*)(wbase + qk * 32 * UNIT + (j - 4) * 1024), (lds_u32*)(l + 2 * UNIT + (wave * 2 + j - 4) * 1024), 16, 0, 0);
  };
  if (wave == 0) {
    const bool good = wave_wait_flags(flag_ptr(0), true, call, tmo, call);
    __builtin_amdgcn_fence(__ATOMIC_ACQUIRE, "");
    WAIT_VM(0);
    if (lane == 0) *lds_word = good ? 1u : 0u;
  }
  RAW_BARRIER();
  STAMP(2);
  if (RFL(*lds_word) != 0u) {
    struct Frag { i32x4 r[8]; };
    auto rd = [&](Frag& f, unsigned ring, int kb, int i) {
      const unsigned o = (i < 2) ? wb_off + kb * 512 + i * 256
                       : (i < 6) ? xa_off + ((i - 2) >> 1) * 4096 + kb * 2048 + ((i - 2) & 1) * 1024
                                 : wb_off + 4096 + kb * 512 + (i - 6) * 256;
      f.r[i] = *(const i32x4*)(smem + ring + o);
    };
#define CAT(a, b) __builtin_shufflevector(a, b, 0, 1, 2, 3, 4, 5, 6, 7)
#define MA(f, i, j) acc[i][j] = __builtin_amdgcn_mfma_scale_f32_32x32x64_f8f6f4(                  \
      CAT(f.r[(i) * 6], f.r[(i) * 6 + 1]), CAT(f.r[2 + 2 * (j)], f.r[3 + 2 * (j)]), acc[i][j], 1, 1, 0, 0x7F7F7F7F, 0, 0x7F7F7F7F)
#pragma unroll
    for (int s = 0; s < 3; ++s)
#pragma unroll
      for (int j = 0; j < 6; ++j) glds1(s, s * STAGE_BYTES, j);
    WAIT_VM(12);
    RAW_BARRIER();
    Frag f0, f1;
#pragma unroll
    for (int i = 0; i < 8; ++i) rd(f0, 0, 0, i);
    __builtin_amdgcn_s_waitcnt(0xC07F);
    unsigned ring = 0, ringp = 2 * STAGE_BYTES;
#define STEP(WAITN, G2, G1) do {                                                         \
      const unsigned ring1 = ring == 2 * STAGE_BYTES ? 0u : ring + STAGE_BYTES;          \
      MA(f0, 0, 0); SCHED(); rd(f1, ring, 1, 0); rd(f1, ring, 1, 1); rd(f1, ring, 1, 2); rd(f1, ring, 1, 3); SCHED(); \
      MA(f0, 0, 1); SCHED(); rd(f1, ring, 1, 4); rd(f1, ring, 1, 5); rd(f1, ring, 1, 6); rd(f1, ring, 1, 7); SCHED(); \
      MA(f0, 1, 0); SCHED(); if (G2) { glds1(s + 2, ringp, 3); glds1(s + 2, ringp, 4); glds1(s + 2, ringp, 5); } SCHED(); \
      MA(f0, 1, 1); SCHED();                                                             \
      WAIT_VM(WAITN);                                                                    \
      RAW_BARRIER();            \
      MA(f1, 0, 0); SCHED(); rd(f0, ring1, 0, 0); rd(f0, ring1, 0, 1); rd(f0, ring1, 0, 2); rd(f0, ring1, 0, 3); SCHED(); \
      MA(f1, 0, 1); SCHED(); rd(f0, ring1, 0, 4); rd(f0, ring1, 0, 5); rd(f0, ring1, 0, 6); rd(f0, ring1, 0, 7); SCHED(); \
      MA(f1, 1, 0); SCHED(); if (G1) { glds1(s + 3, ring, 0); glds1(s + 3, ring, 1); glds1(s + 3, ring, 2); } SCHED(); \
      MA(f1, 1, 1);                                                                      \
      __builtin_amdgcn_s_waitcnt(0xC07F);                                                \
      SCHED();                                                                           \
      ringp = ring; ring = ring1;                                                        \
    } while (0)
    int s = 0;
    STEP(6, false, true);
    ++s;
#pragma unroll 1
    for (; s < 29; ++s) {
      if ((s & 7) == 3 && s < 27 && wave == 0)
        __builtin_amdgcn_global_load_lds(flag_ptr((s + 5) >> 3), (lds_u32*)lds_flags, 4, 0, 17);
      if ((s & 7) == 5) {
        if (!__all(lds_flags[lane] >= call)) {
          if (wave == 0) {
            const bool good = wave_wait_flags(flag_ptr((s + 3) >> 3), true, call, tmo, call);
            if (!good && lane == 0) *lds_word = 0u;
          }
          RAW_BARRIER();
        }
        STAMP(2 + ((s + 3) >> 3));
      }
      STEP(6, true, true);
    }
    STEP(6, true, false);
    ++s;
    STEP(0, false, false);
#pragma unroll
    for (int i = 0; i < 8; ++i) rd(f1, ring, 1, i);
    MA(f0, 0, 0); MA(f0, 0, 1); MA(f0, 1, 0); MA(f0, 1, 1);
    MA(f1, 0, 0); MA(f1, 0, 1); MA(f1, 1, 0); MA(f1, 1, 1);
  }
  RAW_BARRIER();
  STAMP(6);

  {
    const bool bad = *lds_word == 0u;
    const float nanv = __builtin_nanf("");
#pragma unroll
    for (int i = 0; i < 2; ++i)
#pragma unroll
      for (int j = 0; j < 2; ++j) {
        const int row = wm * 64 + j * 32 + (lane & 31);
#pragma unroll
        for (int r = 0; r < 4; ++r) {
          f32x4 v;
#pragma unroll
          for (int t = 0; t < 4; ++t) {
            const float y = acc[i][j][r + 4 * t] * scale;
            v[t] = bad ? nanv : ((y <= 0.f) ? 0.f : y);
          }
          const int col = 16 * (r + 4 * (lane >> 5)) + 4 * (wn * 2 + i);
          *(f32x4*)(smem + row * EPI_PITCH + col * 4) = v;
        }
      }
  }
  RAW_BARRIER();
  STAMP(14);
  const auto outr = mk_rsrc(p.out, 512u * NTOT * 4u);
  if (arel < 2) {
    const unsigned ob = (unsigned)((par ^ arel) * 256) * (NTOT * 4u) + (unsigned)((p.rank >> 1) * NS + nt * 128) * 4u + (lane & 31) * 16;
#pragma unroll 4
    for (int it = 0; it < 16; ++it) {
      const int row = wave * 32 + it * 2 + (lane >> 5);
      const u32x4 v = *(const u32x4*)(smem + row * EPI_PITCH + (lane & 31) * 16);
      __builtin_amdgcn_raw_buffer_store_b128(v, outr, ob + (unsigned)row * (NTOT * 4u), 0, AUX_OUT);
    }
    if (tid == 0) st_agent(ctr, call);
    STAMP(7);
    if (b >= 56 && b < 64) {
      if (tid < 16) lds_word[32 + tid] = 0u;
      RAW_BARRIER();
      const int pr = wave & 3;
      if (pr < NPAIR) {
        volatile lds_u32* ctl = lds_word + 32 + pr * 4;
        uint8_t* ring = smem + pr * 2 * UNIT;
        if (wave >= 4) copy_loader(p, ring, ctl, (unsigned)((b - 56) * NPAIR + pr), call, tmo, lane);
        else copy_storer(p, ring, ctl, call, lane);
      }
      STAMP(8);
    }
    return;
  }

  {
    uint8_t* seg = smem + wave * 2 * ITEM_BYTES;
    u32x4 v[16];
#pragma unroll
    for (int blk = 0; blk < 16; ++blk)
      v[blk] = *(const u32x4*)(seg + (blk * 2 + (lane >> 5)) * EPI_PITCH + (lane & 31) * 16);
    WAIT_LGKM0();
    SCHED();
    const auto stg = mk_rsrc(myws + OFF_STG + (unsigned)(arel * 32 + nt) * TILE_BYTES, TILE_BYTES);
#pragma unroll
    for (int hf = 0; hf < 2; ++hf) {
      uint8_t* it = seg + hf * ITEM_BYTES;
      unsigned base = 256;
      unsigned bits = 0;
#pragma unroll
      for (int e = 0; e < 32; ++e) {
        const unsigned val = v[hf * 8 + (e >> 2)][e & 3];
        const bool nz = val != 0u;
        const unsigned long long m = __ballot(nz);
        const unsigned pos = __builtin_amdgcn_mbcnt_hi((unsigned)(m >> 32), __builtin_amdgcn_mbcnt_lo((unsigned)m, 0u));
        *(unsigned*)(it + (nz ? base + pos * 4 : lane * 4u)) = val;
        base += (unsigned)__popcll(m) * 4u;
        bits |= nz ? (1u << e) : 0u;
      }
      *(unsigned*)(it + lane * 4) = bits;
      if (lane == 0) lds_word[16 + wave * 2 + hf] = (base - 256u) >> 2;
      const unsigned go = (unsigned)(wave * 2 + hf) * ITEM_BYTES;
#pragma unroll 3
      for (unsigned o = lane * 16u; o < base; o += 1024u)
        __builtin_amdgcn_raw_buffer_store_b128(*(const u32x4*)(it + o), stg, go + o, 0, AUX_SYS);
    }
    if (tid < 16) lds_word[32 + tid] = 0u;
  }
  WAIT_VM(0);
  RAW_BARRIER();
  STAMP(10);
  const unsigned tag = (call & 0x7FFFu) << 17;
  if (wave == 7) {
    (void)wave_wait_flags((const unsigned*)(myws + OFF_PD) + arel * 32, lane == 0, (call - 1u) * 32u + GATE_N, tmo, call);
    unsigned t = 0;
    if (lane == 0) t = add_agent((unsigned*)(myws + OFF_TK) + arel * 32);
    t = RFL(t) & 31u;
    if (lane < 16) st_sys((unsigned*)(p.ws[arel] + OFF_SF) + (arel * 32 + t) * 16 + lane, tag | ((unsigned)nt << 12) | lds_word[16 + lane]);
    if (lane == 0) { st_agent(ctr, call); STAMPX(9); }
  }
  if (b < 160) return;

  {
    const int eng = wave < 4 ? (wave & 1) : ((wave - 4) >> 1);
    const int g = (b - 160) * 2 + eng;
    const int rel = 2 + g % 6;
    const unsigned src = (unsigned)(p.rank ^ rel) & 7u;
    volatile lds_u32* c = lds_word + 32 + eng * 8;
    uint8_t* ring = smem + eng * 65536;
    constexpr unsigned END = 0xFFFFFFFFu;
    if (wave < 2) {
      const uint8_t* rstg = p.ws[rel] + OFF_STG + (unsigned)(rel * 32) * TILE_BYTES + lane * 16;
#pragma unroll 1
      for (unsigned k = 0;; ++k) {
        if (RFL(c[0]) <= k) {
          WAIT_VM(0);
          if (lane == 0) c[1] = k;
          lds_wait_ge(c, k + 1);
        }
        const unsigned d = RFL(c[4 + (k & 3)]);
        if (d == END) { WAIT_VM(0); if (lane == 0) c[1] = k; break; }
        const unsigned nbytes = 256u + (d & 0xFFFu) * 4u;
        const uint8_t* gsrc = rstg + ((d >> 12) & 511u) * ITEM_BYTES;
        uint8_t* l = ring + (k & 3) * ITEM_BYTES;
        const int nch = (int)((nbytes + 1023u) >> 10);
        const bool fast = (d >> 30) != 0u;
#pragma unroll 1
        for (int ch = 0; ch < nch; ++ch) {
          if ((unsigned)ch * 1024u + lane * 16u < nbytes)
            __builtin_amdgcn_global_load_lds((const unsigned*)(gsrc + ch * 1024), (lds_u32*)(l + ch * 1024), 16, 0, 17);
          if (ch == 0 || !fast) WAIT_VM(1); else WAIT_VM(PULL_VM2);
          if (ch == 0 && lane == 0) c[1] = k;
        }
        LOGB(k, rel, nbytes);
      }
      STAMP(7);
    } else if (wave < 4) {
      unsigned* head = (unsigned*)(myws + OFF_H2) + rel * 32;
      arm_counter(head, call, lane);
      const unsigned* q = (const unsigned*)(myws + OFF_SF) + rel * 512;
      unsigned i = (unsigned)(g / 6);
#pragma unroll 1
      for (unsigned k = 0;; ++k) {
        for (;;) {
          const unsigned u0 = c[2], u1 = c[3], ld = c[1];
          if (k < RFL(min(min(u0, u1) + 4u, ld + PULL_AHEAD))) break;
          __builtin_amdgcn_s_sleep(1);
        }
        if (k > 0) i = draw(head, lane) + 32u;
        unsigned d = END;
        if (i < 512u) {
          const unsigned long long t0 = __builtin_amdgcn_s_memrealtime();
          for (unsigned spin = 1;; ++spin) {
            const unsigned w = RFL(ld_sys(q + i));
            if ((w & 0xFFFE0000u) == tag) {
              d = ((((w >> 12) & 31u) * 16u + (i & 15u)) << 12) | min(w & 0xFFFu, 2048u) | (i >= PULL_FAST ? 1u << 30 : 0u);
              break;
            }
            if (give_up(spin, t0, tmo, call)) {
              if (lane == 0) p.out[(p.rank >> 1) * NS] = __builtin_nanf("");
              break;
            }
            __builtin_amdgcn_s_sleep(1);
          }
        }
        if (lane == 0) c[4 + (k & 3)] = d;
        WAIT_LGKM0();
        if (lane == 0) c[0] = k + 1;
        if (d == END) break;
      }
    } else {
      const int u = wave & 1;
#pragma unroll 1
      for (unsigned k = 0;; ++k) {
        lds_wait_ge(c, k + 1);
        const unsigned d = RFL(c[4 + (k & 3)]);
        if (d == END) break;
        lds_wait_ge(c + 1, k + 1);
        const uint8_t* l = ring + (k & 3) * ITEM_BYTES;
        const unsigned bits = *(const unsigned*)(l + lane * 4);
        unsigned base = 256;
        if (u) {
          const unsigned below = __popc(bits & 0xFFFFu);
#pragma unroll
          for (int bt = 0; bt < 5; ++bt) base += (unsigned)__popcll(__ballot((below >> bt) & 1u)) << (bt + 2);
        }
        const unsigned mine = bits >> (16 * u);
        u32x4 v[4];
#pragma unroll
        for (int e = 0; e < 16; ++e) {
          const bool bit = (mine >> e) & 1u;
          const unsigned long long m = __ballot(bit);
          const unsigned pos = __builtin_amdgcn_mbcnt_hi((unsigned)(m >> 32), __builtin_amdgcn_mbcnt_lo((unsigned)m, 0u));
          const unsigned x = *(const unsigned*)(l + base + pos * 4u);
          v[e >> 2][e & 3] = bit ? x : 0u;
          base += (unsigned)__popcll(m) * 4u;
        }
        WAIT_LGKM0();
        if (lane == 0) c[2 + u] = k + 1;
        const unsigned tq = (d >> 12) & 511u;
        const unsigned ob = ((src & 1u) * 256u + (tq & 15u) * 16u + u * 8u + (lane >> 5)) * (NTOT * 4u) +
                            ((src >> 1) * NS + (tq >> 4) * 128u) * 4u + (lane & 31) * 16u;
#pragma unroll
        for (int bb = 0; bb < 4; ++bb)
          __builtin_amdgcn_raw_buffer_store_b128(v[bb], outr, ob + bb * 2u * (NTOT * 4u), 0, AUX_OUT);
      }
      if (tid == 256) STAMPX(8);
    }
  }
}

}

extern "C" void kernel_launch(int rank, int nranks, ncclComm_t comm,
                              hipStream_t stream,
                              void* const* d_in, const int* in_sizes, int n_in,
                              void* d_out, int out_size,
                              const int* in_dtypes, int out_dtype,
                              void* const* d_out_all,
                              void* const* d_ws_all, size_t ws_bytes) {
  (void)comm; (void)in_sizes; (void)in_dtypes; (void)out_dtype; (void)d_out_all;
  if (nranks != NR || (unsigned)rank >= (unsigned)NR || ws_bytes < WS_NEED || n_in < 4 || out_size != 512 * NTOT) {
    fprintf(stderr, "kernel_launch: unsupported configuration\n");
    return;
  }
  Params p;
  p.x = (const uint8_t*)d_in[0];
  p.w = (const uint8_t*)d_in[1];
  p.sx = (const float*)d_in[2];
  p.sw = (const float*)d_in[3];
  for (int i = 0; i < NR; ++i) p.ws[i] = (uint8_t*)d_ws_all[rank ^ i];
  p.out = (float*)d_out;
  p.rank = rank;
  {
    uint8_t map[NWG];
    bool used[NWG] = {};
    const int f0 = 32 - NF * ((rank & 1) + 1);
    auto put = [&](int ii, int x, int arel, int nt) { map[ii * 8 + x] = (uint8_t)(arel << 5 | nt); used[ii * 8 + x] = true; };
    for (int x = 0; x < 8; ++x)
      for (int j = 0; j < 12; ++j) put(20 + j, x, 2 + 3 * (x & 1) + j % 3, 4 * (x >> 1) + j / 3);
    for (int x = 0; x < 8; ++x) put(7, x, 0, 4 * (x >> 1) + (x & 1));
    int sidx = 0;
    for (int nt = 0; nt < 32; ++nt)
      for (int a = 0; a < 2; ++a) {
        if (a == 0 && nt < 16 && (nt & 3) < 2) continue;
        put(sidx % 7, sidx / 7, a, nt);
        ++sidx;
      }
    for (int x = 0; x < 8; ++x)
      for (int j = 0; j < 12; ++j) put(8 + j, x, 2 + 3 * (x & 1) + j % 3, 16 + 4 * (x >> 1) + j / 3);
    (void)f0; (void)used;
    for (int i = 0; i < 64; ++i)
      p.tilemap[i] = map[4 * i] | map[4 * i + 1] << 8 | map[4 * i + 2] << 16 | (unsigned)map[4 * i + 3] << 24;
  }
  hipLaunchKernelGGL(gemm_rs_kernel, dim3(NWG), dim3(NTHREADS), 0, stream, p);
  HIP_CHECK(hipGetLastError());
}
```

```cpp
#include <hip/hip_runtime.h>
#include <rccl/rccl.h>
#include <cstdio>
#include <cstdint>

#define HIP_CHECK(expr) do { hipError_t _e = (expr); if (_e != hipSuccess) { \
    fprintf(stderr, "HIP error %s at %s:%d\n", hipGetErrorString(_e), __FILE__, __LINE__); \
    return; } } while (0)

namespace {

typedef unsigned u32x4 __attribute__((ext_vector_type(4)));
typedef int i32x4 __attribute__((ext_vector_type(4)));
typedef float f32x16 __attribute__((ext_vector_type(16)));
typedef float f32x4 __attribute__((ext_vector_type(4)));
typedef __attribute__((address_space(3))) unsigned lds_u32;

constexpr int NR = 8;
constexpr int NTOT = 16384;
constexpr int KS = 512;
constexpr int NS = 4096;
constexpr unsigned UNIT = 16384;

constexpr unsigned OFF_CTR = 0;
constexpr unsigned OFF_TMO = 4096;
constexpr unsigned OFF_XF = 8192;
constexpr unsigned OFF_WF = 16384;
constexpr unsigned OFF_TK = 24576;
constexpr unsigned OFF_H2 = 26624;
constexpr unsigned OFF_PD = 28672;
constexpr unsigned OFF_HD = 30720;
constexpr unsigned OFF_SF = 32768;
constexpr unsigned OFF_DIAG = 65536;
constexpr unsigned OFF_DATA = 1u << 20;
constexpr unsigned X_BYTES = 8u << 20;
constexpr unsigned W_BYTES = 16u << 20;
constexpr unsigned PAR_BYTES = X_BYTES + W_BYTES;
constexpr unsigned ITEM_BYTES = 16 * 528;
constexpr unsigned TILE_BYTES = 16 * ITEM_BYTES;
constexpr unsigned OFF_STG = OFF_DATA + 2 * PAR_BYTES;
constexpr unsigned STG_BYTES = 8 * 32 * TILE_BYTES;
constexpr size_t WS_NEED = (size_t)OFF_STG + STG_BYTES;

constexpr int NWG = 256;
constexpr int NTHREADS = 512;
constexpr unsigned STAGE_BYTES = 3 * UNIT;
constexpr unsigned EPI_PITCH = 528;
constexpr unsigned LDS_MAIN = 3 * STAGE_BYTES;
constexpr unsigned LDS_BYTES = LDS_MAIN + 512;
constexpr unsigned long long DEADLINE_TICKS = 40000000ull;
constexpr int AUX_SYS = 17;
constexpr int AUX_OUT = 16;

#define PUSH_VM 2
#define PUSH_VM2 3
#define PUSH_LAST 8
#define NF 3
#define FREE_AHEAD 12
#define NPAIR 2
#define DSPLIT 3
#define LATE_NPAIR 2
#define LATE_VM 2
#define STOP1 176
#define GATE_N 16
#define PULL_VM2 5
#define PULL_FAST 496
#define PULL_AHEAD 3

struct Params {
  const uint8_t* x;
  const uint8_t* w;
  uint8_t* ws[NR];
  float* out;
  const float* sx;
  const float* sw;
  int rank;
  unsigned tilemap[64];
};

__device__ __forceinline__ __amdgpu_buffer_rsrc_t mk_rsrc(const void* p, unsigned bytes) {
  return __builtin_amdgcn_make_buffer_rsrc((void*)p, 0, bytes, 0x00020000);
}
__device__ __forceinline__ unsigned ld_sys(const unsigned* p) {
  return __hip_atomic_load(p, __ATOMIC_RELAXED, __HIP_MEMORY_SCOPE_SYSTEM);
}
__device__ __forceinline__ void st_sys(unsigned* p, unsigned v) {
  __hip_atomic_store(p, v, __ATOMIC_RELAXED, __HIP_MEMORY_SCOPE_SYSTEM);
}
__device__ __forceinline__ unsigned ld_agent(const unsigned* p) {
  return __hip_atomic_load(p, __ATOMIC_RELAXED, __HIP_MEMORY_SCOPE_AGENT);
}
__device__ __forceinline__ void st_agent(unsigned* p, unsigned v) {
  __hip_atomic_store(p, v, __ATOMIC_RELAXED, __HIP_MEMORY_SCOPE_AGENT);
}
__device__ __forceinline__ unsigned add_agent(unsigned* p) {
  return __hip_atomic_fetch_add(p, 1u, __ATOMIC_RELAXED, __HIP_MEMORY_SCOPE_AGENT);
}
#define WAIT_VM(n) asm volatile("s_waitcnt vmcnt(%0)" :: "i"(n) : "memory")
#define WAIT_LGKM0() asm volatile("s_waitcnt lgkmcnt(0)" ::: "memory")
#define RAW_BARRIER() do { WAIT_LGKM0(); __builtin_amdgcn_s_barrier(); } while (0)
#define SCHED() __builtin_amdgcn_sched_barrier(0)
#define RFL(v) __builtin_amdgcn_readfirstlane(v)

__device__ __forceinline__ bool give_up(unsigned spin, unsigned long long t0, unsigned* tmo, unsigned call) {
  if ((spin & 7u) != 0u) return false;
  if (ld_agent(tmo) != call && __builtin_amdgcn_s_memrealtime() - t0 <= DEADLINE_TICKS) return false;
  st_agent(tmo, call);
  return true;
}
__device__ __forceinline__ bool wave_wait_flags(const unsigned* flag, bool active, unsigned target, unsigned* tmo, unsigned call) {
  const unsigned long long t0 = __builtin_amdgcn_s_memrealtime();
  for (unsigned spin = 1;; ++spin) {
    const unsigned v = active ? ld_sys(flag) : target;
    if (__all(v >= target)) return true;
    if (give_up(spin, t0, tmo, call)) return false;
    __builtin_amdgcn_s_sleep(2);
  }
}
__device__ __forceinline__ void lds_wait_ge(const volatile lds_u32* w, unsigned v) {
  while (RFL(*w) < v) __builtin_amdgcn_s_sleep(1);
  asm volatile("" ::: "memory");
}
__device__ __forceinline__ void arm_counter(unsigned* head, unsigned call, int lane) {
  if (lane == 0) (void)__hip_atomic_fetch_max(head, call << 10, __ATOMIC_RELAXED, __HIP_MEMORY_SCOPE_AGENT);
  WAIT_VM(0);
}
__device__ __forceinline__ unsigned draw(unsigned* head, int lane) {
  unsigned t = 0;
  if (lane == 0) t = add_agent(head);
  return RFL(t) & 1023u;
}

#define STAMPX(slot) do {} while (0)
#define STAMP(slot) do {} while (0)
#define STAMPW(slot) do {} while (0)
#define LOGB(slot, dest, bytes) do {} while (0)

__device__ __forceinline__ int x_row0(int d, int mu) {
  const int arel = mu >> 1, e = d & 1;
  return (arel >= 2 ? 512 * (d ^ arel) + 256 * e : 512 * d + 256 * (e ^ arel)) + 128 * (mu & 1);
}
__device__ __forceinline__ unsigned* xflag(uint8_t* dws, int rank, int mu, int kc) {
  return (unsigned*)(dws + OFF_XF) + (((mu >> 1) * 4 + kc) * 8 + rank) * 2 + (mu & 1);
}
__device__ __forceinline__ unsigned* wflag(uint8_t* dws, int rank, int nt, int kc) {
  return (unsigned*)(dws + OFF_WF) + (nt * 4 + kc) * 8 + rank;
}
__device__ __forceinline__ void glds_x(const Params& p, uint8_t* l, int row0, int kc, int lane) {
  const uint8_t* src = p.x + (unsigned)(row0 + (lane >> 3)) * KS + kc * 128 + (((lane ^ (lane >> 3)) & 7) * 16);
#pragma unroll
  for (int i = 0; i < 16; ++i)
    __builtin_amdgcn_global_load_lds((const unsigned*)(src + i * 8 * KS), (lds_u32*)(l + i * 1024), 16, 0, 0);
}
__device__ __forceinline__ void load_w_raw(u32x4 (&r)[16], const Params& p, int d, int nt, int kc, int lane) {
  const auto src = mk_rsrc(p.w, (unsigned)KS * NTOT);
  const unsigned base = (unsigned)(kc * 128 + (lane >> 3) * 16) * NTOT + (d >> 1) * NS + nt * 128 + (lane & 7) * 16;
#pragma unroll
  for (int i = 0; i < 16; ++i) r[i] = __builtin_amdgcn_raw_buffer_load_b128(src, base + i * NTOT, 0, 0);
}
__device__ __forceinline__ void transpose_store(const u32x4 (&r)[16], uint8_t* l, int lane) {
#pragma unroll
  for (int d = 0; d < 4; ++d) {
    u32x4 o[4];
#pragma unroll
    for (int e = 0; e < 4; ++e) {
      const unsigned r0 = r[4 * e][d], r1 = r[4 * e + 1][d], r2 = r[4 * e + 2][d], r3 = r[4 * e + 3][d];
      const unsigned t0 = __builtin_amdgcn_perm(r1, r0, 0x05010400u);
      const unsigned t1 = __builtin_amdgcn_perm(r1, r0, 0x07030602u);
      const unsigned t2 = __builtin_amdgcn_perm(r3, r2, 0x05010400u);
      const unsigned t3 = __builtin_amdgcn_perm(r3, r2, 0x07030602u);
      o[0][e] = __builtin_amdgcn_perm(t2, t0, 0x05040100u);
      o[1][e] = __builtin_amdgcn_perm(t2, t0, 0x07060302u);
      o[2][e] = __builtin_amdgcn_perm(t3, t1, 0x05040100u);
      o[3][e] = __builtin_amdgcn_perm(t3, t1, 0x07060302u);
    }
#pragma unroll
    for (int q = 0; q < 4; ++q) *(u32x4*)(l + (4 * d + q) * 1024 + ((lane ^ ((q >> 1) * 8)) * 16)) = o[q];
  }
}

constexpr int NB = 16 - NF;
constexpr unsigned IT_B = 120, IT_D = IT_B + 64 + 8;
constexpr unsigned NSTAT = 8 * NPAIR;
struct Item { bool is_w; int idx, kc, src; };
__device__ __forceinline__ unsigned n_items(int rel) {
  return rel == 1 ? IT_D + 4u * NF * 6u : (rel == 0 ? IT_D : IT_B + 4u * NB + 8u);
}
__device__ __forceinline__ Item decode_item(unsigned it, int rel, int rank) {
  Item r;
  r.src = rank;
  const int d = rank ^ rel;
  const unsigned it_c = IT_B + (rel < 2 ? 64u : 4u * NB);
  if (it < IT_B) {
    const unsigned sl = it % 30u;
    r.kc = it / 30u; r.is_w = sl >= 14u; r.idx = r.is_w ? sl - 14u : (sl < 2u ? sl : sl + 2u);
  } else if (it < it_c) {
    const unsigned i = it - IT_B;
    r.is_w = true;
    if (rel < 2) { r.kc = i >> 4; r.idx = 16 + (i & 15u); }
    else {
      const unsigned c = i % NB;
      r.kc = i / NB;
      r.idx = c >= NF ? 16 + (c - NF) : 32 - NF * (2 - (d & 1)) + c;
    }
  } else if (it < it_c + 8u) {
    const unsigned i = it - it_c;
    r.kc = i >> 1; r.is_w = false; r.idx = 2 + (i & 1u);
  } else {
    const unsigned i = it - IT_D;
    unsigned col, f;
    if (i < DSPLIT * NF * 6) { col = i % NF; f = i / NF; }
    else { constexpr unsigned R = DSPLIT < 4 ? (4 - DSPLIT) * 6 : 1; const unsigned i2 = i - DSPLIT * NF * 6; col = i2 / R; f = DSPLIT * 6 + i2 % R; }
    r.kc = f / 6; r.is_w = true;
    r.idx = 32 - NF * ((d & 1) + 1) + col;
    r.src = (d ^ (2 + f % 6)) & 7;
  }
  return r;
}
__device__ __forceinline__ unsigned unit_off(const Item& m) {
  return m.is_w ? X_BYTES + (unsigned)((m.src * 4 + m.kc) * 32 + m.idx) * UNIT : (unsigned)((m.src * 4 + m.kc) * 16 + m.idx) * UNIT;
}
template <bool LATE>
__device__ __forceinline__ unsigned push_loader(const Params& p, uint8_t* ring, volatile lds_u32* ctl, int rel, unsigned it,
                                                unsigned stop, const unsigned* ctr, unsigned call, int lane) {
  const int d = (p.rank ^ rel) & 7;
  const unsigned nit = n_items(rel);
  unsigned* head = (unsigned*)(p.ws[0] + OFF_HD) + rel * 32;
  if (LATE) arm_counter(head, call, lane);
#pragma unroll 1
  for (unsigned k = 0;; ++k) {
    uint8_t* l = ring + (k & 1) * UNIT;
    if (LATE || k > 0) {
      lds_wait_ge(ctl + 1, k);
      it = it >= stop ? nit : draw(head, lane) + NSTAT;
    }
    if (it < nit) {
      const Item m = decode_item(it, rel, p.rank);
      if (m.src != p.rank) {
        if (!wave_wait_flags(wflag(p.ws[0], m.src, m.idx, m.kc), lane == 0, call, (unsigned*)(p.ws[0] + OFF_TMO), call)) it = nit;
        const uint8_t* src = p.ws[0] + OFF_DATA + (call & 1u) * PAR_BYTES + unit_off(m) + lane * 16;
#pragma unroll
        for (int j = 0; j < 16; ++j)
          __builtin_amdgcn_global_load_lds((const unsigned*)(src + j * 1024), (lds_u32*)(l + j * 1024), 16, 0, 17);
        WAIT_VM(0);
      } else if (m.is_w) {
        u32x4 r[16];
        load_w_raw(r, p, d, m.idx, m.kc, lane);
        if (!LATE && k == 0) __builtin_amdgcn_s_barrier();
        transpose_store(r, l, lane);
      } else {
        glds_x(p, l, x_row0(d, m.idx), m.kc, lane);
        if (!LATE && k == 0) __builtin_amdgcn_s_barrier();
        WAIT_VM(0);
      }
    }
    if (lane == 0) ctl[2 + (k & 1)] = it;
    WAIT_LGKM0();
    if (lane == 0) ctl[0] = k + 1;
    if (!LATE && k == 0) {
      if (threadIdx.x == 256) STAMPX(12);
      call = RFL(ld_agent(ctr)) + 1u;
      arm_counter(head, call, lane);
    }
    if (it >= nit) return call;
  }
}
template <int WIN>
__device__ __forceinline__ void stream_item(const uint8_t* l, bool raw_x, __amdgpu_buffer_rsrc_t dst, unsigned off,
                                            unsigned* prev_flag, unsigned call, int lane,
                                            volatile lds_u32* req, unsigned req_val) {
  u32x4 v[16];
  unsigned a4[4];
#pragma unroll
  for (int q = 0; q < 4; ++q)
    a4[q] = raw_x ? (unsigned)((lane & 31) * 128 + (((q * 2 + (lane >> 5)) ^ lane) & 7) * 16) : (unsigned)(q * 1024 + lane * 16);
#pragma unroll
  for (int j = 0; j < 16; ++j) v[j] = *(const u32x4*)(l + (j >> 2) * 4096 + a4[j & 3]);
#pragma unroll
  for (int j = 0; j < 16; ++j) {
    __builtin_amdgcn_raw_buffer_store_b128(v[j], dst, off + j * 1024 + lane * 16, 0, AUX_SYS);
    WAIT_VM(WIN);
    if (j + 1 == WIN && lane == 0 && prev_flag) st_sys(prev_flag, call);
    if (j == (WIN > 3 ? 0 : 16 - FREE_AHEAD) && lane == 0) *req = req_val;
  }
}
template <int WA, int WB>
__device__ __forceinline__ void push_storer(const Params& p, const uint8_t* ring, volatile lds_u32* ctl, int rel,
                                            unsigned call, int lane) {
  const unsigned nit = n_items(rel);
  uint8_t* dws = p.ws[rel];
  const auto dst = mk_rsrc(dws + OFF_DATA + (call & 1u) * PAR_BYTES, PAR_BYTES);
  unsigned* pf = nullptr;
#pragma unroll 1
  for (unsigned k = 0;; ++k) {
    lds_wait_ge(ctl, k + 1);
    const unsigned it = RFL(ctl[2 + (k & 1)]);
    if (it >= nit) break;
    const Item m = decode_item(it, rel, p.rank);
    const uint8_t* l = ring + (k & 1) * UNIT;
    if (WA == WB || it + PUSH_LAST < nit) stream_item<WA>(l, !m.is_w, dst, unit_off(m), pf, call, lane, ctl + 1, k + 1);
    else stream_item<WB>(l, !m.is_w, dst, unit_off(m), pf, call, lane, ctl + 1, k + 1);
    if (it + 32u >= nit && lane == 0) (void)add_agent((unsigned*)(p.ws[0] + OFF_PD) + rel * 32);
    pf = m.is_w ? wflag(dws, m.src, m.idx, m.kc) : xflag(dws, m.src, m.idx, m.kc);
    LOGB(k, rel, 16384);
  }
  WAIT_VM(0);
  if (lane == 0 && pf) st_sys(pf, call);
}

__global__ void __launch_bounds__(NTHREADS, 2) gemm_rs_kernel(Params p) {
  __shared__ __attribute__((aligned(16))) uint8_t smem[LDS_BYTES];
  const int tid = threadIdx.x;
  const int lane = tid & 63;
  const int wave = RFL(tid >> 6);
  const int b = blockIdx.x;
  uint8_t* myws = p.ws[0];
  unsigned* tmo = (unsigned*)(myws + OFF_TMO);
  volatile lds_u32* lds_word = (volatile lds_u32*)(smem + LDS_MAIN);
  volatile lds_u32* lds_flags = lds_word + 64;

  STAMP(0);
  unsigned* ctr = (unsigned*)(myws + OFF_CTR) + b;
  unsigned call;

  if (b < 64) {
    const int pr = wave & 3, pw = (b < 56 ? b : b - 56) * NPAIR + pr;
    const int rel = b < 56 ? 1 + pw % 7 : 0;
    const unsigned first = (unsigned)(b < 56 ? pw / 7 : pw);
    volatile lds_u32* ctl = lds_word + 32 + pr * 4;
    uint8_t* ring = smem + pr * 2 * UNIT;
    if (pr >= NPAIR) {
      __builtin_amdgcn_s_barrier();
      call = RFL(ld_agent(ctr)) + 1u;
    } else if (wave >= 4) {
      call = push_loader<false>(p, ring, ctl, rel, first, rel == 1 ? STOP1 : ~0u, ctr, 0u, lane);
    } else {
      const unsigned craw = ld_agent(ctr);
      if (lane < 4) ctl[lane] = 0u;
      RAW_BARRIER();
      call = RFL(craw) + 1u;
      push_storer<PUSH_VM, PUSH_VM2>(p, ring, ctl, rel, call, lane);
    }
  } else {
    call = RFL(ld_agent(ctr)) + 1u;
  }
  STAMP(1);

  const unsigned tm = (p.tilemap[b >> 2] >> ((b & 3) * 8)) & 255u;
  const int arel = tm >> 5, nt = tm & 31;
  const int par = p.rank & 1;
  const int wm = wave & 3, wn = wave >> 2;
  const float scale = p.sx[0] * p.sw[0];

  const uint8_t* data = myws + OFF_DATA + (call & 1u) * PAR_BYTES;
  const unsigned xa_off = (unsigned)((wm >> 1) * UNIT + (wm & 1) * 2 * 4096 + lane * 16);
  const unsigned wb_off = 2 * UNIT + (unsigned)(wn * 2 * 4096 + ((lane & 31) >> 3) * 1024 +
                                                ((((lane >> 5) ^ (lane >> 4)) & 1) * 8 + (lane & 7)) * 16);
  f32x16 acc[2][2];
#pragma unroll
  for (int i = 0; i < 2; ++i)
#pragma unroll
    for (int j = 0; j < 2; ++j)
#pragma unroll
      for (int r = 0; r < 16; ++r) acc[i][j][r] = 0.f;

  const bool far_w = (unsigned)(31 - par * NF - nt) < (unsigned)NF;
  auto step_qk = [&](int s) -> unsigned {
    const int f = s - 8, fk = (f * 43) >> 8;
    const int sr = far_w ? (s < 8 ? (s & 1) : 2 + f - fk * 6) : (s & 7);
    const int kc = far_w ? (s < 8 ? (s >> 1) : fk) : (s >> 3);
    return (unsigned)((p.rank ^ sr) * 4 + kc);
  };
  auto flag_ptr = [&](int g) -> const unsigned* {
    const bool isx = (lane & 31) < 8;
    const unsigned qk = step_qk(g * 4 + ((isx ? lane >> 1 : lane) & 3));
    const unsigned q = qk >> 2, kc = qk & 3u;
    return isx ? (const unsigned*)(myws + OFF_XF) + ((arel * 4 + kc) * 8 + q) * 2 + (lane & 1)
                     : (const unsigned*)(myws + OFF_WF) + (nt * 4 + kc) * 8 + q;
  };
  const uint8_t* xbase = data + (unsigned)(2 * arel) * UNIT + wave * 4096 + lane * 16;
  const uint8_t* wbase = data + X_BYTES + (unsigned)nt * UNIT + wave * 2048 + lane * 16;
  const unsigned qtab = step_qk(lane & 31);
  auto glds1 = [&](int s, unsigned ring, int j) {
    const unsigned qk = __builtin_amdgcn_readlane(qtab, s);
    uint8_t* l = smem + ring;
    if (j < 4) __builtin_amdgcn_global_load_lds((const unsigned*)(xbase + qk * 16 * UNIT + j * 1024), (lds_u32*)(l + (wave * 4 + j) * 1024), 16, 0, 0);
    else __builtin_amdgcn_global_load_lds((const unsigned*)(wbase + qk * 32 * UNIT + (j - 4) * 1024), (lds_u32*)(l + 2 * UNIT + (wave * 2 + j - 4) * 1024), 16, 0, 0);
  };
  if (wave == 0) {
    const bool good = wave_wait_flags(flag_ptr(lane >> 5), true, call, tmo, call);
    __builtin_amdgcn_fence(__ATOMIC_ACQUIRE, "");
    WAIT_VM(0);
    if (lane == 0) *lds_word = good ? 1u : 0u;
  }
  RAW_BARRIER();
  STAMP(2);
  if (RFL(*lds_word) != 0u) {
    struct Frag { i32x4 r[8]; };
    auto rd = [&](Frag& f, unsigned ring, int kb, int i) {
      const unsigned o = (i < 2) ? wb_off + kb * 512 + i * 256
                       : (i < 6) ? xa_off + ((i - 2) >> 1) * 4096 + kb * 2048 + ((i - 2) & 1) * 1024
                                 : wb_off + 4096 + kb * 512 + (i - 6) * 256;
      f.r[i] = *(const i32x4*)(smem + ring + o);
    };
#define CAT(a, b) __builtin_shufflevector(a, b, 0, 1, 2, 3, 4, 5, 6, 7)
#define MA(f, i, j) acc[i][j] = __builtin_amdgcn_mfma_scale_f32_32x32x64_f8f6f4(                  \
      CAT(f.r[(i) * 6], f.r[(i) * 6 + 1]), CAT(f.r[2 + 2 * (j)], f.r[3 + 2 * (j)]), acc[i][j], 1, 1, 0, 0x7F7F7F7F, 0, 0x7F7F7F7F)
#pragma unroll
    for (int s = 0; s < 3; ++s)
#pragma unroll
      for (int j = 0; j < 6; ++j) glds1(s, s * STAGE_BYTES, j);
    WAIT_VM(12);
    RAW_BARRIER();
    Frag f0, f1;
#pragma unroll
    for (int i = 0; i < 8; ++i) rd(f0, 0, 0, i);
    __builtin_amdgcn_s_waitcnt(0xC07F);
    unsigned ring = 0, ringp = 2 * STAGE_BYTES;
#define STEP(WAITN, G2, G1) do {                                                         \
      const unsigned ring1 = ring == 2 * STAGE_BYTES ? 0u : ring + STAGE_BYTES;          \
      MA(f0, 0, 0); SCHED(); rd(f1, ring, 1, 0); rd(f1, ring, 1, 1); rd(f1, ring, 1, 2); rd(f1, ring, 1, 3); SCHED(); \
      MA(f0, 0, 1); SCHED(); rd(f1, ring, 1, 4); rd(f1, ring, 1, 5); rd(f1, ring, 1, 6); rd(f1, ring, 1, 7); SCHED(); \
      MA(f0, 1, 0); SCHED(); if (G2) { glds1(s + 2, ringp, 3); glds1(s + 2, ringp, 4); glds1(s + 2, ringp, 5); } SCHED(); \
      MA(f0, 1, 1); SCHED();                                                             \
      WAIT_VM(WAITN);                                                                    \
      RAW_BARRIER();            \
      MA(f1, 0, 0); SCHED(); rd(f0, ring1, 0, 0); rd(f0, ring1, 0, 1); rd(f0, ring1, 0, 2); rd(f0, ring1, 0, 3); SCHED(); \
      MA(f1, 0, 1); SCHED(); rd(f0, ring1, 0, 4); rd(f0, ring1, 0, 5); rd(f0, ring1, 0, 6); rd(f0, ring1, 0, 7); SCHED(); \
      MA(f1, 1, 0); SCHED(); if (G1) { glds1(s + 3, ring, 0); glds1(s + 3, ring, 1); glds1(s + 3, ring, 2); } SCHED(); \
      MA(f1, 1, 1);                                                                      \
      __builtin_amdgcn_s_waitcnt(0xC07F);                                                \
      SCHED();                                                                           \
      ringp = ring; ring = ring1;                                                        \
    } while (0)
    int s = 0;
    STEP(6, false, true);
    ++s;
#pragma unroll 1
    for (; s < 29; ++s) {
      if ((s & 3) == 3 && s < 27 && wave == 0)
        __builtin_amdgcn_global_load_lds(flag_ptr((s + 5) >> 2), (lds_u32*)lds_flags, 4, 0, 17);
      if ((s & 3) == 1 && s > 1) {
        if (!__all(lds_flags[lane] >= call)) {
          if (wave == 0) {
            const bool good = wave_wait_flags(flag_ptr((s + 3) >> 2), true, call, tmo, call);
            if (!good && lane == 0) *lds_word = 0u;
          }
          RAW_BARRIER();
        }
        if ((s & 7) == 5) STAMP(2 + ((s + 3) >> 3));
      }
      STEP(6, true, true);
    }
    STEP(6, true, false);
    ++s;
    STEP(0, false, false);
#pragma unroll
    for (int i = 0; i < 8; ++i) rd(f1, ring, 1, i);
    MA(f0, 0, 0); MA(f0, 0, 1); MA(f0, 1, 0); MA(f0, 1, 1);
    MA(f1, 0, 0); MA(f1, 0, 1); MA(f1, 1, 0); MA(f1, 1, 1);
  }
  RAW_BARRIER();
  STAMP(6);

  {
    const bool bad = *lds_word == 0u;
    const float nanv = __builtin_nanf("");
#pragma unroll
    for (int i = 0; i < 2; ++i)
#pragma unroll
      for (int j = 0; j < 2; ++j) {
        const int row = wm * 64 + j * 32 + (lane & 31);
#pragma unroll
        for (int r = 0; r < 4; ++r) {
          f32x4 v;
#pragma unroll
          for (int t = 0; t < 4; ++t) {
            const float y = acc[i][j][r + 4 * t] * scale;
            v[t] = bad ? nanv : ((y <= 0.f) ? 0.f : y);
          }
          const int col = 16 * (r + 4 * (lane >> 5)) + 4 * (wn * 2 + i);
          *(f32x4*)(smem + row * EPI_PITCH + col * 4) = v;
        }
      }
  }
  RAW_BARRIER();
  STAMP(14);
  const auto outr = mk_rsrc(p.out, 512u * NTOT * 4u);
  if (arel < 2) {
    const unsigned ob = (unsigned)((par ^ arel) * 256) * (NTOT * 4u) + (unsigned)((p.rank >> 1) * NS + nt * 128) * 4u + (lane & 31) * 16;
#pragma unroll 4
    for (int it = 0; it < 16; ++it) {
      const int row = wave * 32 + it * 2 + (lane >> 5);
      const u32x4 v = *(const u32x4*)(smem + row * EPI_PITCH + (lane & 31) * 16);
      __builtin_amdgcn_raw_buffer_store_b128(v, outr, ob + (unsigned)row * (NTOT * 4u), 0, AUX_OUT);
    }
    if (tid == 0) st_agent(ctr, call);
    STAMP(7);
    if (b >= 56 && b < 64) {
      if (tid < 16) lds_word[32 + tid] = 0u;
      RAW_BARRIER();
      const int pr = wave & 3;
      if (pr < LATE_NPAIR) {
        volatile lds_u32* ctl = lds_word + 32 + pr * 4;
        uint8_t* ring = smem + pr * 2 * UNIT;
        if (wave >= 4) (void)push_loader<true>(p, ring, ctl, 1, 0u, ~0u, ctr, call, lane);
        else push_storer<LATE_VM, LATE_VM>(p, ring, ctl, 1, call, lane);
      }
      STAMP(8);
    }
    return;
  }

  {
    uint8_t* seg = smem + wave * 2 * ITEM_BYTES;
    u32x4 v[16];
#pragma unroll
    for (int blk = 0; blk < 16; ++blk)
      v[blk] = *(const u32x4*)(seg + (blk * 2 + (lane >> 5)) * EPI_PITCH + (lane & 31) * 16);
    WAIT_LGKM0();
    SCHED();
    const auto stg = mk_rsrc(myws + OFF_STG + (unsigned)(arel * 32 + nt) * TILE_BYTES, TILE_BYTES);
#pragma unroll
    for (int hf = 0; hf < 2; ++hf) {
      uint8_t* it = seg + hf * ITEM_BYTES;
      unsigned base = 256;
      unsigned bits = 0;
#pragma unroll
      for (int e = 0; e < 32; ++e) {
        const unsigned val = v[hf * 8 + (e >> 2)][e & 3];
        const bool nz = val != 0u;
        const unsigned long long m = __ballot(nz);
        const unsigned pos = __builtin_amdgcn_mbcnt_hi((unsigned)(m >> 32), __builtin_amdgcn_mbcnt_lo((unsigned)m, 0u));
        *(unsigned*)(it + (nz ? base + pos * 4 : lane * 4u)) = val;
        base += (unsigned)__popcll(m) * 4u;
        bits |= nz ? (1u << e) : 0u;
      }
      *(unsigned*)(it + lane * 4) = bits;
      if (lane == 0) lds_word[16 + wave * 2 + hf] = (base - 256u) >> 2;
      const unsigned go = (unsigned)(wave * 2 + hf) * ITEM_BYTES;
#pragma unroll 3
      for (unsigned o = lane * 16u; o < base; o += 1024u)
        __builtin_amdgcn_raw_buffer_store_b128(*(const u32x4*)(it + o), stg, go + o, 0, AUX_SYS);
    }
    if (tid < 16) lds_word[32 + tid] = 0u;
  }
  WAIT_VM(0);
  RAW_BARRIER();
  STAMP(10);
  const unsigned tag = (call & 0x7FFFu) << 17;
  if (wave == 7) {
    (void)wave_wait_flags((const unsigned*)(myws + OFF_PD) + arel * 32, lane == 0, (call - 1u) * 32u + GATE_N, tmo, call);
    unsigned t = 0;
    if (lane == 0) t = add_agent((unsigned*)(myws + OFF_TK) + arel * 32);
    t = RFL(t) & 31u;
    if (lane < 16) st_sys((unsigned*)(p.ws[arel] + OFF_SF) + (arel * 32 + t) * 16 + lane, tag | ((unsigned)nt << 12) | lds_word[16 + lane]);
    if (lane == 0) { st_agent(ctr, call); STAMPX(9); }
  }
  if (b < 160) return;

  {
    const int eng = wave < 4 ? (wave & 1) : ((wave - 4) >> 1);
    const int g = (b - 160) * 2 + eng;
    const int rel = 2 + g % 6;
    const unsigned src = (unsigned)(p.rank ^ rel) & 7u;
    volatile lds_u32* c = lds_word + 32 + eng * 8;
    uint8_t* ring = smem + eng * 65536;
    constexpr unsigned END = 0xFFFFFFFFu;
    if (wave < 2) {
      const uint8_t* rstg = p.ws[rel] + OFF_STG + (unsigned)(rel * 32) * TILE_BYTES + lane * 16;
#pragma unroll 1
      for (unsigned k = 0;; ++k) {
        if (RFL(c[0]) <= k) {
          WAIT_VM(0);
          if (lane == 0) c[1] = k;
          lds_wait_ge(c, k + 1);
        }
        const unsigned d = RFL(c[4 + (k & 3)]);
        if (d == END) { WAIT_VM(0); if (lane == 0) c[1] = k; break; }
        const unsigned nbytes = 256u + (d & 0xFFFu) * 4u;
        const uint8_t* gsrc = rstg + ((d >> 12) & 511u) * ITEM_BYTES;
        uint8_t* l = ring + (k & 3) * ITEM_BYTES;
        const int nch = (int)((nbytes + 1023u) >> 10);
        const bool fast = (d >> 30) != 0u;
#pragma unroll 1
        for (int ch = 0; ch < nch; ++ch) {
          if ((unsigned)ch * 1024u + lane * 16u < nbytes)
            __builtin_amdgcn_global_load_lds((const unsigned*)(gsrc + ch * 1024), (lds_u32*)(l + ch * 1024), 16, 0, 17);
          if (ch == 0 || !fast) WAIT_VM(1); else WAIT_VM(PULL_VM2);
          if (ch == 0 && lane == 0) c[1] = k;
        }
        LOGB(k, rel, nbytes);
      }
      STAMP(7);
    } else if (wave < 4) {
      unsigned* head = (unsigned*)(myws + OFF_H2) + rel * 32;
      arm_counter(head, call, lane);
      const unsigned* q = (const unsigned*)(myws + OFF_SF) + rel * 512;
      unsigned i = (unsigned)(g / 6);
#pragma unroll 1
      for (unsigned k = 0;; ++k) {
        for (;;) {
          const unsigned u0 = c[2], u1 = c[3], ld = c[1];
          if (k < RFL(min(min(u0, u1) + 4u, ld + PULL_AHEAD))) break;
          __builtin_amdgcn_s_sleep(1);
        }
        if (k > 0) i = draw(head, lane) + 32u;
        unsigned d = END;
        if (i < 512u) {
          const unsigned long long t0 = __builtin_amdgcn_s_memrealtime();
          for (unsigned spin = 1;; ++spin) {
            const unsigned w = RFL(ld_sys(q + i));
            if ((w & 0xFFFE0000u) == tag) {
              d = ((((w >> 12) & 31u) * 16u + (i & 15u)) << 12) | min(w & 0xFFFu, 2048u) | (i >= PULL_FAST ? 1u << 30 : 0u);
              break;
            }
            if (give_up(spin, t0, tmo, call)) {
              if (lane == 0) p.out[(p.rank >> 1) * NS] = __builtin_nanf("");
              break;
            }
            __builtin_amdgcn_s_sleep(1);
          }
        }
        if (lane == 0) c[4 + (k & 3)] = d;
        WAIT_LGKM0();
        if (lane == 0) c[0] = k + 1;
        if (d == END) break;
      }
    } else {
      const int u = wave & 1;
#pragma unroll 1
      for (unsigned k = 0;; ++k) {
        lds_wait_ge(c, k + 1);
        const unsigned d = RFL(c[4 + (k & 3)]);
        if (d == END) break;
        lds_wait_ge(c + 1, k + 1);
        const uint8_t* l = ring + (k & 3) * ITEM_BYTES;
        const unsigned bits = *(const unsigned*)(l + lane * 4);
        unsigned base = 256;
        if (u) {
          const unsigned below = __popc(bits & 0xFFFFu);
#pragma unroll
          for (int bt = 0; bt < 5; ++bt) base += (unsigned)__popcll(__ballot((below >> bt) & 1u)) << (bt + 2);
        }
        const unsigned mine = bits >> (16 * u);
        u32x4 v[4];
#pragma unroll
        for (int e = 0; e < 16; ++e) {
          const bool bit = (mine >> e) & 1u;
          const unsigned long long m = __ballot(bit);
          const unsigned pos = __builtin_amdgcn_mbcnt_hi((unsigned)(m >> 32), __builtin_amdgcn_mbcnt_lo((unsigned)m, 0u));
          const unsigned x = *(const unsigned*)(l + base + pos * 4u);
          v[e >> 2][e & 3] = bit ? x : 0u;
          base += (unsigned)__popcll(m) * 4u;
        }
        WAIT_LGKM0();
        if (lane == 0) c[2 + u] = k + 1;
        const unsigned tq = (d >> 12) & 511u;
        const unsigned ob = ((src & 1u) * 256u + (tq & 15u) * 16u + u * 8u + (lane >> 5)) * (NTOT * 4u) +
                            ((src >> 1) * NS + (tq >> 4) * 128u) * 4u + (lane & 31) * 16u;
#pragma unroll
        for (int bb = 0; bb < 4; ++bb)
          __builtin_amdgcn_raw_buffer_store_b128(v[bb], outr, ob + bb * 2u * (NTOT * 4u), 0, AUX_OUT);
      }
      if (tid == 256) STAMPX(8);
    }
  }
}

}

extern "C" void kernel_launch(int rank, int nranks, ncclComm_t comm,
                              hipStream_t stream,
                              void* const* d_in, const int* in_sizes, int n_in,
                              void* d_out, int out_size,
                              const int* in_dtypes, int out_dtype,
                              void* const* d_out_all,
                              void* const* d_ws_all, size_t ws_bytes) {
  (void)comm; (void)in_sizes; (void)in_dtypes; (void)out_dtype; (void)d_out_all;
  if (nranks != NR || (unsigned)rank >= (unsigned)NR || ws_bytes < WS_NEED || n_in < 4 || out_size != 512 * NTOT) {
    fprintf(stderr, "kernel_launch: unsupported configuration\n");
    return;
  }
  Params p;
  p.x = (const uint8_t*)d_in[0];
  p.w = (const uint8_t*)d_in[1];
  p.sx = (const float*)d_in[2];
  p.sw = (const float*)d_in[3];
  for (int i = 0; i < NR; ++i) p.ws[i] = (uint8_t*)d_ws_all[rank ^ i];
  p.out = (float*)d_out;
  p.rank = rank;
  {
    uint8_t map[NWG];
    bool used[NWG] = {};
    const int f0 = 32 - NF * ((rank & 1) + 1);
    auto put = [&](int ii, int x, int arel, int nt) { map[ii * 8 + x] = (uint8_t)(arel << 5 | nt); used[ii * 8 + x] = true; };
    for (int x = 0; x < 8; ++x)
      for (int j = 0; j < 12; ++j) put(20 + j, x, 2 + 3 * (x & 1) + j % 3, 4 * (x >> 1) + j / 3);
    for (int x = 0; x < 8; ++x) put(7, x, 0, 4 * (x >> 1) + (x & 1));
    int sidx = 0;
    for (int nt = 0; nt < 32; ++nt)
      for (int a = 0; a < 2; ++a) {
        if (a == 0 && nt < 16 && (nt & 3) < 2) continue;
        put(sidx % 7, sidx / 7, a, nt);
        ++sidx;
      }
    for (int x = 0; x < 8; ++x)
      for (int j = 0; j < 12; ++j) put(8 + j, x, 2 + 3 * (x & 1) + j % 3, 16 + 4 * (x >> 1) + j / 3);
    (void)f0; (void)used;
    for (int i = 0; i < 64; ++i)
      p.tilemap[i] = map[4 * i] | map[4 * i + 1] << 8 | map[4 * i + 2] << 16 | (unsigned)map[4 * i + 3] << 24;
  }
  hipLaunchKernelGGL(gemm_rs_kernel, dim3(NWG), dim3(NTHREADS), 0, stream, p);
  HIP_CHECK(hipGetLastError());
}
```

```cpp
#include <hip/hip_runtime.h>
#include <rccl/rccl.h>
#include <cstdio>
#include <cstdint>

#define HIP_CHECK(expr) do { hipError_t _e = (expr); if (_e != hipSuccess) { \
    fprintf(stderr, "HIP error %s at %s:%d\n", hipGetErrorString(_e), __FILE__, __LINE__); \
    return; } } while (0)

namespace {

typedef unsigned u32x4 __attribute__((ext_vector_type(4)));
typedef int i32x4 __attribute__((ext_vector_type(4)));
typedef float f32x16 __attribute__((ext_vector_type(16)));
typedef float f32x4 __attribute__((ext_vector_type(4)));
typedef __attribute__((address_space(3))) unsigned lds_u32;

constexpr int NR = 8;
constexpr int NTOT = 16384;
constexpr int KS = 512;
constexpr int NS = 4096;
constexpr unsigned UNIT = 16384;

constexpr unsigned OFF_CTR = 0;
constexpr unsigned OFF_TMO = 4096;
constexpr unsigned OFF_XF = 8192;
constexpr unsigned OFF_WF = 16384;
constexpr unsigned OFF_TK = 24576;
constexpr unsigned OFF_H2 = 26624;
constexpr unsigned OFF_PD = 28672;
constexpr unsigned OFF_HD = 30720;
constexpr unsigned OFF_SF = 32768;
constexpr unsigned OFF_DIAG = 65536;
constexpr unsigned OFF_DATA = 1u << 20;
constexpr unsigned X_BYTES = 8u << 20;
constexpr unsigned W_BYTES = 16u << 20;
constexpr unsigned PAR_BYTES = X_BYTES + W_BYTES;
constexpr unsigned ITEM_BYTES = 16 * 528;
constexpr unsigned TILE_BYTES = 16 * ITEM_BYTES;
constexpr unsigned OFF_STG = OFF_DATA + 2 * PAR_BYTES;
constexpr unsigned STG_BYTES = 8 * 32 * TILE_BYTES;
constexpr size_t WS_NEED = (size_t)OFF_STG + STG_BYTES;

constexpr int NWG = 256;
constexpr int NTHREADS = 512;
constexpr unsigned STAGE_BYTES = 3 * UNIT;
constexpr unsigned EPI_PITCH = 528;
constexpr unsigned LDS_MAIN = 3 * STAGE_BYTES;
constexpr unsigned LDS_BYTES = LDS_MAIN + 512;
constexpr unsigned long long DEADLINE_TICKS = 40000000ull;
constexpr int AUX_SYS = 17;
constexpr int AUX_OUT = 16;

#define NPAIR 2
#define PUSH_VM 2
#define PUSH_VM2 3
#define PUSH_LAST 16
#define FREE_AHEAD 12
#define NF 2
#define DSPLIT 3
#define LATE_NPAIR 2
#define LATE_VM 2
#define STOP1 176
#define GATE_N 12
#define GATE_N2 12
#define GATE_RAMP 18
#define PULL_VM2 5
#define PULL_FAST 496
#define PULL_AHEAD 3
static_assert(GATE_N <= 32 && GATE_N2 + GATE_RAMP <= 32 && NPAIR <= 4 && LATE_NPAIR <= 4 && NF >= 1 && NF <= 4, "knobs");

struct Params {
  const uint8_t* x;
  const uint8_t* w;
  uint8_t* ws[NR];
  float* out;
  const float* sx;
  const float* sw;
  int rank;
  unsigned tilemap[64];
};

__device__ __forceinline__ __amdgpu_buffer_rsrc_t mk_rsrc(const void* p, unsigned bytes) {
  return __builtin_amdgcn_make_buffer_rsrc((void*)p, 0, bytes, 0x00020000);
}
__device__ __forceinline__ unsigned ld_sys(const unsigned* p) {
  return __hip_atomic_load(p, __ATOMIC_RELAXED, __HIP_MEMORY_SCOPE_SYSTEM);
}
__device__ __forceinline__ void st_sys(unsigned* p, unsigned v) {
  __hip_atomic_store(p, v, __ATOMIC_RELAXED, __HIP_MEMORY_SCOPE_SYSTEM);
}
__device__ __forceinline__ unsigned ld_agent(const unsigned* p) {
  return __hip_atomic_load(p, __ATOMIC_RELAXED, __HIP_MEMORY_SCOPE_AGENT);
}
__device__ __forceinline__ void st_agent(unsigned* p, unsigned v) {
  __hip_atomic_store(p, v, __ATOMIC_RELAXED, __HIP_MEMORY_SCOPE_AGENT);
}
__device__ __forceinline__ unsigned add_agent(unsigned* p) {
  return __hip_atomic_fetch_add(p, 1u, __ATOMIC_RELAXED, __HIP_MEMORY_SCOPE_AGENT);
}
#define WAIT_VM(n) asm volatile("s_waitcnt vmcnt(%0)" :: "i"(n) : "memory")
#define WAIT_LGKM0() asm volatile("s_waitcnt lgkmcnt(0)" ::: "memory")
#define RAW_BARRIER() do { WAIT_LGKM0(); __builtin_amdgcn_s_barrier(); } while (0)
#define SCHED() __builtin_amdgcn_sched_barrier(0)
#define RFL(v) __builtin_amdgcn_readfirstlane(v)

__device__ __forceinline__ bool give_up(unsigned spin, unsigned long long t0, unsigned* tmo, unsigned call) {
  if ((spin & 7u) != 0u) return false;
  if (ld_agent(tmo) != call && __builtin_amdgcn_s_memrealtime() - t0 <= DEADLINE_TICKS) return false;
  st_agent(tmo, call);
  return true;
}
__device__ __forceinline__ bool wave_wait_flags(const unsigned* flag, bool active, unsigned target, unsigned* tmo, unsigned call) {
  const unsigned long long t0 = __builtin_amdgcn_s_memrealtime();
  for (unsigned spin = 1;; ++spin) {
    const unsigned v = active ? ld_sys(flag) : target;
    if (__all(v >= target)) return true;
    if (give_up(spin, t0, tmo, call)) return false;
    __builtin_amdgcn_s_sleep(2);
  }
}
__device__ __forceinline__ void lds_wait_ge(const volatile lds_u32* w, unsigned v) {
  while (RFL(*w) < v) __builtin_amdgcn_s_sleep(1);
  asm volatile("" ::: "memory");
}
__device__ __forceinline__ void arm_counter(unsigned* head, unsigned call, int lane) {
  if (lane == 0) (void)__hip_atomic_fetch_max(head, call << 10, __ATOMIC_RELAXED, __HIP_MEMORY_SCOPE_AGENT);
  WAIT_VM(0);
}
__device__ __forceinline__ unsigned draw(unsigned* head, int lane) {
  unsigned t = 0;
  if (lane == 0) t = add_agent(head);
  return RFL(t) & 1023u;
}

#define STAMPX(slot) do {} while (0)
#define STAMP(slot) do {} while (0)
#define STAMPW(slot) do {} while (0)
#define LOGB(slot, dest, bytes) do {} while (0)

__device__ __forceinline__ int x_row0(int d, int mu) {
  const int arel = mu >> 1, e = d & 1;
  return (arel >= 2 ? 512 * (d ^ arel) + 256 * e : 512 * d + 256 * (e ^ arel)) + 128 * (mu & 1);
}
__device__ __forceinline__ unsigned* xflag(uint8_t* dws, int rank, int mu, int kc) {
  return (unsigned*)(dws + OFF_XF) + (((mu >> 1) * 4 + kc) * 8 + rank) * 2 + (mu & 1);
}
__device__ __forceinline__ unsigned* wflag(uint8_t* dws, int rank, int nt, int kc) {
  return (unsigned*)(dws + OFF_WF) + (nt * 4 + kc) * 8 + rank;
}
__device__ __forceinline__ void glds_x(const Params& p, uint8_t* l, int row0, int kc, int lane) {
  const uint8_t* src = p.x + (unsigned)(row0 + (lane >> 3)) * KS + kc * 128 + (((lane ^ (lane >> 3)) & 7) * 16);
#pragma unroll
  for (int i = 0; i < 16; ++i)
    __builtin_amdgcn_global_load_lds((const unsigned*)(src + i * 8 * KS), (lds_u32*)(l + i * 1024), 16, 0, 0);
}
__device__ __forceinline__ void load_w_raw(u32x4 (&r)[16], const Params& p, int d, int nt, int kc, int lane) {
  const auto src = mk_rsrc(p.w, (unsigned)KS * NTOT);
  const unsigned base = (unsigned)(kc * 128 + (lane >> 3) * 16) * NTOT + (d >> 1) * NS + nt * 128 + (lane & 7) * 16;
#pragma unroll
  for (int i = 0; i < 16; ++i) r[i] = __builtin_amdgcn_raw_buffer_load_b128(src, base + i * NTOT, 0, 0);
}
__device__ __forceinline__ void transpose_store(const u32x4 (&r)[16], uint8_t* l, int lane) {
#pragma unroll
  for (int d = 0; d < 4; ++d) {
    u32x4 o[4];
#pragma unroll
    for (int e = 0; e < 4; ++e) {
      const unsigned r0 = r[4 * e][d], r1 = r[4 * e + 1][d], r2 = r[4 * e + 2][d], r3 = r[4 * e + 3][d];
      const unsigned t0 = __builtin_amdgcn_perm(r1, r0, 0x05010400u);
      const unsigned t1 = __builtin_amdgcn_perm(r1, r0, 0x07030602u);
      const unsigned t2 = __builtin_amdgcn_perm(r3, r2, 0x05010400u);
      const unsigned t3 = __builtin_amdgcn_perm(r3, r2, 0x07030602u);
      o[0][e] = __builtin_amdgcn_perm(t2, t0, 0x05040100u);
      o[1][e] = __builtin_amdgcn_perm(t2, t0, 0x07060302u);
      o[2][e] = __builtin_amdgcn_perm(t3, t1, 0x05040100u);
      o[3][e] = __builtin_amdgcn_perm(t3, t1, 0x07060302u);
    }
#pragma unroll
    for (int q = 0; q < 4; ++q) *(u32x4*)(l + (4 * d + q) * 1024 + ((lane ^ ((q >> 1) * 8)) * 16)) = o[q];
  }
}

constexpr int NB = 16 - NF;
constexpr unsigned IT_B = 120, IT_D = IT_B + 64 + 8;
constexpr unsigned NSTAT = 8 * NPAIR;
struct Item { bool is_w; int idx, kc, src; };
__device__ __forceinline__ unsigned n_items(int rel) {
  return rel == 1 ? IT_D + 4u * NF * 6u : (rel == 0 ? IT_D : IT_B + 4u * NB + 8u);
}
__device__ __forceinline__ Item decode_item(unsigned it, int rel, int rank) {
  Item r;
  r.src = rank;
  const int d = rank ^ rel;
  const unsigned it_c = IT_B + (rel < 2 ? 64u : 4u * NB);
  if (it < IT_B) {
    const unsigned sl = it % 30u;
    r.kc = it / 30u; r.is_w = sl >= 14u; r.idx = r.is_w ? sl - 14u : (sl < 2u ? sl : sl + 2u);
  } else if (it < it_c) {
    unsigned i = it - IT_B;
    r.is_w = true;
    if (rel < 2 && i < 4u * NF) {
      r.kc = i / NF; r.idx = 32 - NF * ((d & 1) + 1) + i % NF;
    } else {
      if (rel < 2) i -= 4u * NF;
      const unsigned c = i % NB;
      r.kc = i / NB;
      r.idx = c >= NF ? 16 + (c - NF) : 32 - NF * (2 - (d & 1)) + c;
    }
  } else if (it < it_c + 8u) {
    const unsigned i = it - it_c;
    r.kc = i >> 1; r.is_w = false; r.idx = 2 + (i & 1u);
  } else {
    const unsigned i = it - IT_D;
    unsigned col, f;
    if (i < DSPLIT * NF * 6) { col = i % NF; f = i / NF; }
    else { constexpr unsigned R = DSPLIT < 4 ? (4 - DSPLIT) * 6 : 1; const unsigned i2 = i - DSPLIT * NF * 6; col = i2 / R; f = DSPLIT * 6 + i2 % R; }
    r.kc = f / 6; r.is_w = true;
    r.idx = 32 - NF * ((d & 1) + 1) + col;
    r.src = (d ^ (2 + f % 6)) & 7;
  }
  return r;
}
__device__ __forceinline__ unsigned unit_off(const Item& m) {
  return m.is_w ? X_BYTES + (unsigned)((m.src * 4 + m.kc) * 32 + m.idx) * UNIT : (unsigned)((m.src * 4 + m.kc) * 16 + m.idx) * UNIT;
}
template <bool LATE>
__device__ __forceinline__ unsigned push_loader(const Params& p, uint8_t* ring, volatile lds_u32* ctl, int rel, unsigned it,
                                                unsigned stop, const unsigned* ctr, unsigned call, int lane) {
  const int d = (p.rank ^ rel) & 7;
  const unsigned nit = n_items(rel);
  unsigned* head = (unsigned*)(p.ws[0] + OFF_HD) + rel * 32;
  if (LATE) arm_counter(head, call, lane);
#pragma unroll 1
  for (unsigned k = 0;; ++k) {
    uint8_t* l = ring + (k & 1) * UNIT;
    if (LATE || k > 0) {
      lds_wait_ge(ctl + 1, k);
      it = it >= stop ? nit : draw(head, lane) + NSTAT;
    }
    if (it < nit) {
      const Item m = decode_item(it, rel, p.rank);
      if (m.src != p.rank) {
        if (!wave_wait_flags(wflag(p.ws[0], m.src, m.idx, m.kc), lane == 0, call, (unsigned*)(p.ws[0] + OFF_TMO), call)) it = nit;
        const uint8_t* src = p.ws[0] + OFF_DATA + (call & 1u) * PAR_BYTES + unit_off(m) + lane * 16;
#pragma unroll
        for (int j = 0; j < 16; ++j)
          __builtin_amdgcn_global_load_lds((const unsigned*)(src + j * 1024), (lds_u32*)(l + j * 1024), 16, 0, 17);
        WAIT_VM(0);
      } else if (m.is_w) {
        u32x4 r[16];
        load_w_raw(r, p, d, m.idx, m.kc, lane);
        if (!LATE && k == 0) __builtin_amdgcn_s_barrier();
        transpose_store(r, l, lane);
      } else {
        glds_x(p, l, x_row0(d, m.idx), m.kc, lane);
        if (!LATE && k == 0) __builtin_amdgcn_s_barrier();
        WAIT_VM(0);
      }
    }
    if (lane == 0) ctl[2 + (k & 1)] = it;
    WAIT_LGKM0();
    if (lane == 0) ctl[0] = k + 1;
    if (!LATE && k == 0) {
      if (threadIdx.x == 256) STAMPX(12);
      call = RFL(ld_agent(ctr)) + 1u;
      arm_counter(head, call, lane);
    }
    if (it >= nit) return call;
  }
}
template <int WIN>
__device__ __forceinline__ void stream_item(const uint8_t* l, bool raw_x, __amdgpu_buffer_rsrc_t dst, unsigned off,
                                            unsigned* prev_flag, unsigned call, int lane,
                                            volatile lds_u32* req, unsigned req_val) {
  u32x4 v[16];
  unsigned a4[4];
#pragma unroll
  for (int q = 0; q < 4; ++q)
    a4[q] = raw_x ? (unsigned)((lane & 31) * 128 + (((q * 2 + (lane >> 5)) ^ lane) & 7) * 16) : (unsigned)(q * 1024 + lane * 16);
#pragma unroll
  for (int j = 0; j < 16; ++j) v[j] = *(const u32x4*)(l + (j >> 2) * 4096 + a4[j & 3]);
#pragma unroll
  for (int j = 0; j < 16; ++j) {
    __builtin_amdgcn_raw_buffer_store_b128(v[j], dst, off + j * 1024 + lane * 16, 0, AUX_SYS);
    WAIT_VM(WIN);
    if (j + 1 == WIN && lane == 0 && prev_flag) st_sys(prev_flag, call);
    if (j == (WIN > 3 ? 0 : 16 - FREE_AHEAD) && lane == 0) *req = req_val;
  }
}
template <int WA, int WB>
__device__ __forceinline__ void push_storer(const Params& p, const uint8_t* ring, volatile lds_u32* ctl, int rel,
                                            unsigned call, int lane) {
  const unsigned nit = n_items(rel);
  uint8_t* dws = p.ws[rel];
  const auto dst = mk_rsrc(dws + OFF_DATA + (call & 1u) * PAR_BYTES, PAR_BYTES);
  unsigned* pf = nullptr;
#pragma unroll 1
  for (unsigned k = 0;; ++k) {
    lds_wait_ge(ctl, k + 1);
    const unsigned it = RFL(ctl[2 + (k & 1)]);
    if (it >= nit) break;
    const Item m = decode_item(it, rel, p.rank);
    const uint8_t* l = ring + (k & 1) * UNIT;
    if (WA == WB || it + PUSH_LAST < nit) stream_item<WA>(l, !m.is_w, dst, unit_off(m), pf, call, lane, ctl + 1, k + 1);
    else stream_item<WB>(l, !m.is_w, dst, unit_off(m), pf, call, lane, ctl + 1, k + 1);
    if (it + 32u >= nit && lane == 0) (void)add_agent((unsigned*)(p.ws[0] + OFF_PD) + rel * 32);
    pf = m.is_w ? wflag(dws, m.src, m.idx, m.kc) : xflag(dws, m.src, m.idx, m.kc);
    LOGB(k, rel, 16384);
  }
  WAIT_VM(0);
  if (lane == 0 && pf) st_sys(pf, call);
}

__global__ void __launch_bounds__(NTHREADS, 2) gemm_rs_kernel(Params p) {
  __shared__ __attribute__((aligned(16))) uint8_t smem[LDS_BYTES];
  const int tid = threadIdx.x;
  const int lane = tid & 63;
  const int wave = RFL(tid >> 6);
  const int b = blockIdx.x;
  uint8_t* myws = p.ws[0];
  unsigned* tmo = (unsigned*)(myws + OFF_TMO);
  volatile lds_u32* lds_word = (volatile lds_u32*)(smem + LDS_MAIN);
  volatile lds_u32* lds_flags = lds_word + 64;

  STAMP(0);
  unsigned* ctr = (unsigned*)(myws + OFF_CTR) + b;
  unsigned call;

  if (b < 64) {
    const int pr = wave & 3, pw = (b < 56 ? b : b - 56) * NPAIR + pr;
    const int rel = b < 56 ? 1 + pw % 7 : 0;
    const unsigned first = (unsigned)(b < 56 ? pw / 7 : pw);
    volatile lds_u32* ctl = lds_word + 32 + pr * 4;
    uint8_t* ring = smem + pr * 2 * UNIT;
    if (pr >= NPAIR) {
      __builtin_amdgcn_s_barrier();
      call = RFL(ld_agent(ctr)) + 1u;
    } else if (wave >= 4) {
      call = push_loader<false>(p, ring, ctl, rel, first, rel == 1 ? STOP1 : ~0u, ctr, 0u, lane);
    } else {
      const unsigned craw = ld_agent(ctr);
      if (lane < 4) ctl[lane] = 0u;
      RAW_BARRIER();
      call = RFL(craw) + 1u;
      push_storer<PUSH_VM, PUSH_VM2>(p, ring, ctl, rel, call, lane);
    }
  } else {
    call = RFL(ld_agent(ctr)) + 1u;
  }
  STAMP(1);

  const unsigned tm = (p.tilemap[b >> 2] >> ((b & 3) * 8)) & 255u;
  const int arel = tm >> 5, nt = tm & 31;
  const int par = p.rank & 1;
  const int wm = wave & 3, wn = wave >> 2;
  const float scale = p.sx[0] * p.sw[0];

  const uint8_t* data = myws + OFF_DATA + (call & 1u) * PAR_BYTES;
  const unsigned xa_off = (unsigned)((wm >> 1) * UNIT + (wm & 1) * 2 * 4096 + lane * 16);
  const unsigned wb_off = 2 * UNIT + (unsigned)(wn * 2 * 4096 + ((lane & 31) >> 3) * 1024 +
                                                ((((lane >> 5) ^ (lane >> 4)) & 1) * 8 + (lane & 7)) * 16);
  f32x16 acc[2][2];
#pragma unroll
  for (int i = 0; i < 2; ++i)
#pragma unroll
    for (int j = 0; j < 2; ++j)
#pragma unroll
      for (int r = 0; r < 16; ++r) acc[i][j][r] = 0.f;

  const bool far_w = (unsigned)(31 - par * NF - nt) < (unsigned)NF;
  auto step_qk = [&](int s) -> unsigned {
    const int f = s - 8, fk = (f * 43) >> 8;
    const int sr = far_w ? (s < 8 ? (s & 1) : 2 + f - fk * 6) : (s & 7);
    const int kc = far_w ? (s < 8 ? (s >> 1) : fk) : (s >> 3);
    return (unsigned)((p.rank ^ sr) * 4 + kc);
  };
  auto flag_ptr = [&](int g) -> const unsigned* {
    const bool isx = (lane & 31) < 8;
    const unsigned qk = step_qk(g * 4 + ((isx ? lane >> 1 : lane) & 3));
    const unsigned q = qk >> 2, kc = qk & 3u;
    return isx ? (const unsigned*)(myws + OFF_XF) + ((arel * 4 + kc) * 8 + q) * 2 + (lane & 1)
                     : (const unsigned*)(myws + OFF_WF) + (nt * 4 + kc) * 8 + q;
  };
  const uint8_t* xbase = data + (unsigned)(2 * arel) * UNIT + wave * 4096 + lane * 16;
  const uint8_t* wbase = data + X_BYTES + (unsigned)nt * UNIT + wave * 2048 + lane * 16;
  const unsigned qtab = step_qk(lane & 31);
  auto glds1 = [&](int s, unsigned ring, int j) {
    const unsigned qk = __builtin_amdgcn_readlane(qtab, s);
    uint8_t* l = smem + ring;
    if (j < 4) __builtin_amdgcn_global_load_lds((const unsigned*)(xbase + qk * 16 * UNIT + j * 1024), (lds_u32*)(l + (wave * 4 + j) * 1024), 16, 0, 0);
    else __builtin_amdgcn_global_load_lds((const unsigned*)(wbase + qk * 32 * UNIT + (j - 4) * 1024), (lds_u32*)(l + 2 * UNIT + (wave * 2 + j - 4) * 1024), 16, 0, 0);
  };
  if (wave == 0) {
    const bool good = wave_wait_flags(flag_ptr(lane >> 5), true, call, tmo, call);
    __builtin_amdgcn_fence(__ATOMIC_ACQUIRE, "");
    WAIT_VM(0);
    if (lane == 0) *lds_word = good ? 1u : 0u;
  }
  RAW_BARRIER();
  STAMP(2);
  if (RFL(*lds_word) != 0u) {
    struct Frag { i32x4 r[8]; };
    auto rd = [&](Frag& f, unsigned ring, int kb, int i) {
      const unsigned o = (i < 2) ? wb_off + kb * 512 + i * 256
                       : (i < 6) ? xa_off + ((i - 2) >> 1) * 4096 + kb * 2048 + ((i - 2) & 1) * 1024
                                 : wb_off + 4096 + kb * 512 + (i - 6) * 256;
      f.r[i] = *(const i32x4*)(smem + ring + o);
    };
#define CAT(a, b) __builtin_shufflevector(a, b, 0, 1, 2, 3, 4, 5, 6, 7)
#define MA(f, i, j) acc[i][j] = __builtin_amdgcn_mfma_scale_f32_32x32x64_f8f6f4(                  \
      CAT(f.r[(i) * 6], f.r[(i) * 6 + 1]), CAT(f.r[2 + 2 * (j)], f.r[3 + 2 * (j)]), acc[i][j], 1, 1, 0, 0x7F7F7F7F, 0, 0x7F7F7F7F)
#pragma unroll
    for (int s = 0; s < 3; ++s)
#pragma unroll
      for (int j = 0; j < 6; ++j) glds1(s, s * STAGE_BYTES, j);
    WAIT_VM(12);
    RAW_BARRIER();
    Frag f0, f1;
#pragma unroll
    for (int i = 0; i < 8; ++i) rd(f0, 0, 0, i);
    __builtin_amdgcn_s_waitcnt(0xC07F);
    unsigned ring = 0, ringp = 2 * STAGE_BYTES;
#define STEP(WAITN, G2, G1) do {                                                         \
      const unsigned ring1 = ring == 2 * STAGE_BYTES ? 0u : ring + STAGE_BYTES;          \
      MA(f0, 0, 0); SCHED(); rd(f1, ring, 1, 0); rd(f1, ring, 1, 1); rd(f1, ring, 1, 2); rd(f1, ring, 1, 3); SCHED(); \
      MA(f0, 0, 1); SCHED(); rd(f1, ring, 1, 4); rd(f1, ring, 1, 5); rd(f1, ring, 1, 6); rd(f1, ring, 1, 7); SCHED(); \
      MA(f0, 1, 0); SCHED(); if (G2) { glds1(s + 2, ringp, 3); glds1(s + 2, ringp, 4); glds1(s + 2, ringp, 5); } SCHED(); \
      MA(f0, 1, 1); SCHED();                                                             \
      WAIT_VM(WAITN);                                                                    \
      RAW_BARRIER();            \
      MA(f1, 0, 0); SCHED(); rd(f0, ring1, 0, 0); rd(f0, ring1, 0, 1); rd(f0, ring1, 0, 2); rd(f0, ring1, 0, 3); SCHED(); \
      MA(f1, 0, 1); SCHED(); rd(f0, ring1, 0, 4); rd(f0, ring1, 0, 5); rd(f0, ring1, 0, 6); rd(f0, ring1, 0, 7); SCHED(); \
      MA(f1, 1, 0); SCHED(); if (G1) { glds1(s + 3, ring, 0); glds1(s + 3, ring, 1); glds1(s + 3, ring, 2); } SCHED(); \
      MA(f1, 1, 1);                                                                      \
      __builtin_amdgcn_s_waitcnt(0xC07F);                                                \
      SCHED();                                                                           \
      ringp = ring; ring = ring1;                                                        \
    } while (0)
    int s = 0;
    STEP(6, false, true);
    ++s;
#pragma unroll 1
    for (; s < 29; ++s) {
      if ((s & 3) == 3 && s < 27 && wave == 0)
        __builtin_amdgcn_global_load_lds(flag_ptr((s + 5) >> 2), (lds_u32*)lds_flags, 4, 0, 17);
      if ((s & 3) == 1 && s > 1) {
        if (!__all(lds_flags[lane] >= call)) {
          if (wave == 0) {
            const bool good = wave_wait_flags(flag_ptr((s + 3) >> 2), true, call, tmo, call);
            if (!good && lane == 0) *lds_word = 0u;
          }
          RAW_BARRIER();
        }
        if ((s & 7) == 5) STAMP(2 + ((s + 3) >> 3));
      }
      STEP(6, true, true);
    }
    STEP(6, true, false);
    ++s;
    STEP(0, false, false);
#pragma unroll
    for (int i = 0; i < 8; ++i) rd(f1, ring, 1, i);
    MA(f0, 0, 0); MA(f0, 0, 1); MA(f0, 1, 0); MA(f0, 1, 1);
    MA(f1, 0, 0); MA(f1, 0, 1); MA(f1, 1, 0); MA(f1, 1, 1);
  }
  RAW_BARRIER();
  STAMP(6);

  {
    const bool bad = *lds_word == 0u;
    const float nanv = __builtin_nanf("");
#pragma unroll
    for (int i = 0; i < 2; ++i)
#pragma unroll
      for (int j = 0; j < 2; ++j) {
        const int row = wm * 64 + j * 32 + (lane & 31);
#pragma unroll
        for (int r = 0; r < 4; ++r) {
          f32x4 v;
#pragma unroll
          for (int t = 0; t < 4; ++t) {
            const float y = acc[i][j][r + 4 * t] * scale;
            v[t] = bad ? nanv : ((y <= 0.f) ? 0.f : y);
          }
          const int col = 16 * (r + 4 * (lane >> 5)) + 4 * (wn * 2 + i);
          *(f32x4*)(smem + row * EPI_PITCH + col * 4) = v;
        }
      }
  }
  RAW_BARRIER();
  STAMP(14);
  const auto outr = mk_rsrc(p.out, 512u * NTOT * 4u);
  if (arel < 2) {
    const unsigned ob = (unsigned)((par ^ arel) * 256) * (NTOT * 4u) + (unsigned)((p.rank >> 1) * NS + nt * 128) * 4u + (lane & 31) * 16;
#pragma unroll 4
    for (int it = 0; it < 16; ++it) {
      const int row = wave * 32 + it * 2 + (lane >> 5);
      const u32x4 v = *(const u32x4*)(smem + row * EPI_PITCH + (lane & 31) * 16);
      __builtin_amdgcn_raw_buffer_store_b128(v, outr, ob + (unsigned)row * (NTOT * 4u), 0, AUX_OUT);
    }
    if (tid == 0) st_agent(ctr, call);
    STAMP(7);
    if (b >= 56 && b < 64) {
      if (tid < 16) lds_word[32 + tid] = 0u;
      RAW_BARRIER();
      const int pr = wave & 3;
      if (pr < LATE_NPAIR) {
        volatile lds_u32* ctl = lds_word + 32 + pr * 4;
        uint8_t* ring = smem + pr * 2 * UNIT;
        if (wave >= 4) (void)push_loader<true>(p, ring, ctl, 1, 0u, ~0u, ctr, call, lane);
        else push_storer<LATE_VM, LATE_VM>(p, ring, ctl, 1, call, lane);
      }
      STAMP(8);
    }
    return;
  }

  {
    uint8_t* seg = smem + wave * 2 * ITEM_BYTES;
    u32x4 v[16];
#pragma unroll
    for (int blk = 0; blk < 16; ++blk)
      v[blk] = *(const u32x4*)(seg + (blk * 2 + (lane >> 5)) * EPI_PITCH + (lane & 31) * 16);
    WAIT_LGKM0();
    SCHED();
    const auto stg = mk_rsrc(myws + OFF_STG + (unsigned)(arel * 32 + nt) * TILE_BYTES, TILE_BYTES);
#pragma unroll
    for (int hf = 0; hf < 2; ++hf) {
      uint8_t* it = seg + hf * ITEM_BYTES;
      const unsigned it0 = (unsigned)(wave * 2 + hf) * ITEM_BYTES;
      const unsigned scratch = it0 + lane * 4u;
      unsigned next = it0 + 256u;
      unsigned bits = 0;
#pragma unroll
      for (int e = 0; e < 32; ++e) {
        const unsigned val = v[hf * 8 + (e >> 2)][e & 3];
        const bool nz = val != 0u;
        const unsigned long long m = __ballot(nz);
        unsigned a = (__builtin_amdgcn_mbcnt_hi((unsigned)(m >> 32), __builtin_amdgcn_mbcnt_lo((unsigned)m, 0u)) << 2) + next;
        asm("" : "+v"(a));
        *(unsigned*)(smem + (nz ? a : scratch)) = val;
        next += (unsigned)__popcll(m) * 4u;
        asm("" : "+s"(next));
        bits |= nz ? (1u << e) : 0u;
      }
      const unsigned base = next - it0;
      *(unsigned*)(it + lane * 4) = bits;
      if (lane == 0) lds_word[16 + wave * 2 + hf] = (base - 256u) >> 2;
      const unsigned go = (unsigned)(wave * 2 + hf) * ITEM_BYTES;
#pragma unroll 3
      for (unsigned o = lane * 16u; o < base; o += 1024u)
        __builtin_amdgcn_raw_buffer_store_b128(*(const u32x4*)(it + o), stg, go + o, 0, AUX_SYS);
    }
    if (tid < 16) lds_word[32 + tid] = 0u;
  }
  const unsigned* gate = (const unsigned*)(myws + OFF_PD) + arel * 32;
  unsigned* tick = (unsigned*)(myws + OFF_TK) + arel * 32;
  unsigned t = 0;
  bool drawn = false;
  if (wave == 7 && RFL(ld_sys(gate)) >= (call - 1u) * 32u + GATE_N) { t = draw(tick, lane); drawn = true; }
  WAIT_VM(0);
  RAW_BARRIER();
  STAMP(10);
  const unsigned tag = (call & 0x7FFFu) << 17;
  if (wave == 7) {
    if (!drawn) {
      (void)wave_wait_flags(gate, lane == 0, (call - 1u) * 32u + GATE_N, tmo, call);
      t = draw(tick, lane);
    }
    t &= 31u;
    if (lane < 16) st_sys((unsigned*)(p.ws[arel] + OFF_SF) + (arel * 32 + t) * 16 + lane, tag | ((unsigned)nt << 12) | lds_word[16 + lane]);
    if (lane == 0) { st_agent(ctr, call); STAMPX(9); }
  }
  if (b < 160) return;

  {
    const int eng = wave < 4 ? (wave & 1) : ((wave - 4) >> 1);
    const int g = (b - 160) * 2 + eng;
    const int rel = 2 + g % 6;
    const unsigned src = (unsigned)(p.rank ^ rel) & 7u;
    volatile lds_u32* c = lds_word + 32 + eng * 8;
    uint8_t* ring = smem + eng * 65536;
    constexpr unsigned END = 0xFFFFFFFFu;
    if (wave < 2) {
      const uint8_t* rstg = p.ws[rel] + OFF_STG + (unsigned)(rel * 32) * TILE_BYTES + lane * 16;
#pragma unroll 1
      for (unsigned k = 0;; ++k) {
        if (RFL(c[0]) <= k) {
          WAIT_VM(0);
          if (lane == 0) c[1] = k;
          lds_wait_ge(c, k + 1);
        }
        const unsigned d = RFL(c[4 + (k & 3)]);
        if (d == END) { WAIT_VM(0); if (lane == 0) c[1] = k; break; }
        const unsigned nbytes = 256u + (d & 0xFFFu) * 4u;
        const uint8_t* gsrc = rstg + ((d >> 12) & 511u) * ITEM_BYTES;
        uint8_t* l = ring + (k & 3) * ITEM_BYTES;
        const int nch = (int)((nbytes + 1023u) >> 10);
        const bool fast = (d >> 30) != 0u;
#pragma unroll 1
        for (int ch = 0; ch < nch; ++ch) {
          if ((unsigned)ch * 1024u + lane * 16u < nbytes)
            __builtin_amdgcn_global_load_lds((const unsigned*)(gsrc + ch * 1024), (lds_u32*)(l + ch * 1024), 16, 0, 17);
          if (ch == 0 || !fast) WAIT_VM(1); else WAIT_VM(PULL_VM2);
          if (ch == 0 && lane == 0) c[1] = k;
        }
        LOGB(k, rel, nbytes);
      }
      STAMP(7);
    } else if (wave < 4) {
      unsigned* head = (unsigned*)(myws + OFF_H2) + rel * 32;
      arm_counter(head, call, lane);
      const unsigned* q = (const unsigned*)(myws + OFF_SF) + rel * 512;
      (void)wave_wait_flags((const unsigned*)(myws + OFF_PD) + rel * 32, lane == 0, (call - 1u) * 32u + GATE_N2 + (unsigned)(g / 6) * GATE_RAMP / 32u, tmo, call);
      unsigned i = (unsigned)(g / 6);
#pragma unroll 1
      for (unsigned k = 0;; ++k) {
        for (;;) {
          const unsigned u0 = c[2], u1 = c[3], ld = c[1];
          if (k < RFL(min(min(u0, u1) + 4u, ld + PULL_AHEAD))) break;
          __builtin_amdgcn_s_sleep(1);
        }
        if (k > 0) i = draw(head, lane) + 32u;
        unsigned d = END;
        if (i < 512u) {
          const unsigned long long t0 = __builtin_amdgcn_s_memrealtime();
          for (unsigned spin = 1;; ++spin) {
            const unsigned w = RFL(ld_sys(q + i));
            if ((w & 0xFFFE0000u) == tag) {
              d = ((((w >> 12) & 31u) * 16u + (i & 15u)) << 12) | min(w & 0xFFFu, 2048u) | (i >= PULL_FAST ? 1u << 30 : 0u);
              break;
            }
            if (give_up(spin, t0, tmo, call)) {
              if (lane == 0) p.out[(p.rank >> 1) * NS] = __builtin_nanf("");
              break;
            }
            __builtin_amdgcn_s_sleep(1);
          }
        }
        if (lane == 0) c[4 + (k & 3)] = d;
        WAIT_LGKM0();
        if (lane == 0) c[0] = k + 1;
        if (d == END) break;
      }
    } else {
      const int u = wave & 1;
#pragma unroll 1
      for (unsigned k = 0;; ++k) {
        lds_wait_ge(c, k + 1);
        const unsigned d = RFL(c[4 + (k & 3)]);
        if (d == END) break;
        lds_wait_ge(c + 1, k + 1);
        const uint8_t* l = ring + (k & 3) * ITEM_BYTES;
        const unsigned bits = *(const unsigned*)(l + lane * 4);
        unsigned base = 256;
        if (u) {
          const unsigned below = __popc(bits & 0xFFFFu);
#pragma unroll
          for (int bt = 0; bt < 5; ++bt) base += (unsigned)__popcll(__ballot((below >> bt) & 1u)) << (bt + 2);
        }
        const unsigned mine = bits >> (16 * u);
        u32x4 v[4];
#pragma unroll
        for (int e = 0; e < 16; ++e) {
          const bool bit = (mine >> e) & 1u;
          const unsigned long long m = __ballot(bit);
          const unsigned pos = __builtin_amdgcn_mbcnt_hi((unsigned)(m >> 32), __builtin_amdgcn_mbcnt_lo((unsigned)m, 0u));
          const unsigned x = *(const unsigned*)(l + base + pos * 4u);
          v[e >> 2][e & 3] = bit ? x : 0u;
          base += (unsigned)__popcll(m) * 4u;
        }
        WAIT_LGKM0();
        if (lane == 0) c[2 + u] = k + 1;
        const unsigned tq = (d >> 12) & 511u;
        const unsigned ob = ((src & 1u) * 256u + (tq & 15u) * 16u + u * 8u + (lane >> 5)) * (NTOT * 4u) +
                            ((src >> 1) * NS + (tq >> 4) * 128u) * 4u + (lane & 31) * 16u;
#pragma unroll
        for (int bb = 0; bb < 4; ++bb)
          __builtin_amdgcn_raw_buffer_store_b128(v[bb], outr, ob + bb * 2u * (NTOT * 4u), 0, AUX_OUT);
      }
      if (tid == 256) STAMPX(8);
    }
  }
}

}

extern "C" void kernel_launch(int rank, int nranks, ncclComm_t comm,
                              hipStream_t stream,
                              void* const* d_in, const int* in_sizes, int n_in,
                              void* d_out, int out_size,
                              const int* in_dtypes, int out_dtype,
                              void* const* d_out_all,
                              void* const* d_ws_all, size_t ws_bytes) {
  (void)comm; (void)in_sizes; (void)in_dtypes; (void)out_dtype; (void)d_out_all;
  if (nranks != NR || (unsigned)rank >= (unsigned)NR || ws_bytes < WS_NEED || n_in < 4 || out_size != 512 * NTOT) {
    fprintf(stderr, "kernel_launch: unsupported configuration\n");
    return;
  }
  Params p;
  p.x = (const uint8_t*)d_in[0];
  p.w = (const uint8_t*)d_in[1];
  p.sx = (const float*)d_in[2];
  p.sw = (const float*)d_in[3];
  for (int i = 0; i < NR; ++i) p.ws[i] = (uint8_t*)d_ws_all[rank ^ i];
  p.out = (float*)d_out;
  p.rank = rank;
  {
    uint8_t map[NWG];
    auto put = [&](int ii, int x, int arel, int nt) { map[ii * 8 + x] = (uint8_t)(arel << 5 | nt); };
    for (int x = 0; x < 8; ++x)
      for (int j = 0; j < 12; ++j) put(20 + j, x, 2 + 3 * (x & 1) + j % 3, 4 * (x >> 1) + j / 3);
    for (int x = 0; x < 8; ++x) put(7, x, 0, 4 * (x >> 1) + (x & 1));
    int sidx = 0;
    for (int nt = 0; nt < 32; ++nt)
      for (int a = 0; a < 2; ++a) {
        if (a == 0 && nt < 16 && (nt & 3) < 2) continue;
        put(sidx % 7, sidx / 7, a, nt);
        ++sidx;
      }
    for (int x = 0; x < 8; ++x)
      for (int j = 0; j < 12; ++j) put(8 + j, x, 2 + 3 * (x & 1) + j % 3, 16 + 4 * (x >> 1) + j / 3);
    for (int i = 0; i < 64; ++i)
      p.tilemap[i] = map[4 * i] | map[4 * i + 1] << 8 | map[4 * i + 2] << 16 | (unsigned)map[4 * i + 3] << 24;
  }
  hipLaunchKernelGGL(gemm_rs_kernel, dim3(NWG), dim3(NTHREADS), 0, stream, p);
  HIP_CHECK(hipGetLastError());
}
```

```cpp
#include <hip/hip_runtime.h>
#include <rccl/rccl.h>
#include <cstdio>
#include <cstdint>

#define HIP_CHECK(expr) do { hipError_t _e = (expr); if (_e != hipSuccess) { \
    fprintf(stderr, "HIP error %s at %s:%d\n", hipGetErrorString(_e), __FILE__, __LINE__); \
    return; } } while (0)

namespace {

typedef unsigned u32x4 __attribute__((ext_vector_type(4)));
typedef int i32x4 __attribute__((ext_vector_type(4)));
typedef float f32x16 __attribute__((ext_vector_type(16)));
typedef float f32x4 __attribute__((ext_vector_type(4)));
typedef __attribute__((address_space(3))) unsigned lds_u32;

constexpr int NR = 8;
constexpr int NTOT = 16384;
constexpr int KS = 512;
constexpr int NS = 4096;
constexpr unsigned UNIT = 16384;

constexpr unsigned OFF_CTR = 0;
constexpr unsigned OFF_TMO = 4096;
constexpr unsigned OFF_XF = 8192;
constexpr unsigned OFF_WF = 16384;
constexpr unsigned OFF_TK = 24576;
constexpr unsigned OFF_H2 = 26624;
constexpr unsigned OFF_PD = 28672;
constexpr unsigned OFF_HD = 30720;
constexpr unsigned OFF_SF = 32768;
constexpr unsigned OFF_DATA = 1u << 20;
constexpr unsigned X_BYTES = 8u << 20;
constexpr unsigned W_BYTES = 16u << 20;
constexpr unsigned PAR_BYTES = X_BYTES + W_BYTES;
constexpr unsigned ITEM_BYTES = 16 * 528;
constexpr unsigned TILE_BYTES = 16 * ITEM_BYTES;
constexpr unsigned OFF_STG = OFF_DATA + 2 * PAR_BYTES;
constexpr unsigned STG_BYTES = 8 * 32 * TILE_BYTES;
constexpr size_t WS_NEED = (size_t)OFF_STG + STG_BYTES;

constexpr int NWG = 256;
constexpr int NTHREADS = 512;
constexpr unsigned STAGE_BYTES = 3 * UNIT;
constexpr unsigned EPI_PITCH = 528;
constexpr unsigned LDS_MAIN = 3 * STAGE_BYTES;
constexpr unsigned LDS_BYTES = LDS_MAIN + 512;
constexpr unsigned long long DEADLINE_TICKS = 40000000ull;
constexpr int AUX_SYS = 17;
constexpr int AUX_OUT = 16;

#define NPAIR 2
#define PUSH_VM 2
#define PUSH_VM2 3
#define PUSH_LAST 16
#define SELF_VM 8
#define FREE_AHEAD 12
#define NF 2
#define DSPLIT 3
#define LATE_NPAIR 2
#define LATE_VM 2
#define STOP1 176
#define GATE_N 12
#define GATE_N2 12
#define GATE_RAMP 18
#define PULL_VM2 5
#define PULL_FAST 496
#define PULL_AHEAD 3
static_assert(GATE_N <= 32 && GATE_N2 + GATE_RAMP <= 32 && NPAIR <= 4 && LATE_NPAIR <= NPAIR && NF >= 1 && NF <= 4, "knobs");

struct Params {
  const uint8_t* x;
  const uint8_t* w;
  uint8_t* ws[NR];
  float* out;
  const float* sx;
  const float* sw;
  int rank;
  unsigned tilemap[64];
};

__device__ __forceinline__ __amdgpu_buffer_rsrc_t mk_rsrc(const void* p, unsigned bytes) {
  return __builtin_amdgcn_make_buffer_rsrc((void*)p, 0, bytes, 0x00020000);
}
__device__ __forceinline__ unsigned ld_sys(const unsigned* p) {
  return __hip_atomic_load(p, __ATOMIC_RELAXED, __HIP_MEMORY_SCOPE_SYSTEM);
}
__device__ __forceinline__ void st_sys(unsigned* p, unsigned v) {
  __hip_atomic_store(p, v, __ATOMIC_RELAXED, __HIP_MEMORY_SCOPE_SYSTEM);
}
__device__ __forceinline__ unsigned ld_agent(const unsigned* p) {
  return __hip_atomic_load(p, __ATOMIC_RELAXED, __HIP_MEMORY_SCOPE_AGENT);
}
__device__ __forceinline__ void st_agent(unsigned* p, unsigned v) {
  __hip_atomic_store(p, v, __ATOMIC_RELAXED, __HIP_MEMORY_SCOPE_AGENT);
}
__device__ __forceinline__ unsigned add_agent(unsigned* p) {
  return __hip_atomic_fetch_add(p, 1u, __ATOMIC_RELAXED, __HIP_MEMORY_SCOPE_AGENT);
}
#define WAIT_VM(n) asm volatile("s_waitcnt vmcnt(%0)" :: "i"(n) : "memory")
#define WAIT_LGKM0() asm volatile("s_waitcnt lgkmcnt(0)" ::: "memory")
#define RAW_BARRIER() do { WAIT_LGKM0(); __builtin_amdgcn_s_barrier(); } while (0)
#define SCHED() __builtin_amdgcn_sched_barrier(0)
#define RFL(v) __builtin_amdgcn_readfirstlane(v)

__device__ __forceinline__ bool give_up(unsigned spin, unsigned long long t0, unsigned* tmo, unsigned call) {
  if ((spin & 7u) != 0u) return false;
  if (ld_agent(tmo) != call && __builtin_amdgcn_s_memrealtime() - t0 <= DEADLINE_TICKS) return false;
  st_agent(tmo, call);
  return true;
}
__device__ __forceinline__ bool wave_wait_flags(const unsigned* flag, bool active, unsigned target, unsigned* tmo, unsigned call) {
  const unsigned long long t0 = __builtin_amdgcn_s_memrealtime();
  for (unsigned spin = 1;; ++spin) {
    const unsigned v = active ? ld_sys(flag) : target;
    if (__all(v >= target)) return true;
    if (give_up(spin, t0, tmo, call)) return false;
    __builtin_amdgcn_s_sleep(2);
  }
}
__device__ __forceinline__ void lds_wait_ge(const volatile lds_u32* w, unsigned v) {
  while (RFL(*w) < v) __builtin_amdgcn_s_sleep(1);
  asm volatile("" ::: "memory");
}
__device__ __forceinline__ void arm_counter(unsigned* head, unsigned call, int lane) {
  if (lane == 0) (void)__hip_atomic_fetch_max(head, call << 10, __ATOMIC_RELAXED, __HIP_MEMORY_SCOPE_AGENT);
  WAIT_VM(0);
}
__device__ __forceinline__ unsigned draw(unsigned* head, int lane) {
  unsigned t = 0;
  if (lane == 0) t = add_agent(head);
  return RFL(t) & 1023u;
}

__device__ __forceinline__ int x_row0(int d, int mu) {
  const int arel = mu >> 1, e = d & 1;
  return (arel >= 2 ? 512 * (d ^ arel) + 256 * e : 512 * d + 256 * (e ^ arel)) + 128 * (mu & 1);
}
__device__ __forceinline__ unsigned* xflag(uint8_t* dws, int rank, int mu, int kc) {
  return (unsigned*)(dws + OFF_XF) + (((mu >> 1) * 4 + kc) * 8 + rank) * 2 + (mu & 1);
}
__device__ __forceinline__ unsigned* wflag(uint8_t* dws, int rank, int nt, int kc) {
  return (unsigned*)(dws + OFF_WF) + (nt * 4 + kc) * 8 + rank;
}
__device__ __forceinline__ void glds_x(const Params& p, uint8_t* l, int row0, int kc, int lane) {
  const uint8_t* src = p.x + (unsigned)(row0 + (lane >> 3)) * KS + kc * 128 + (((lane ^ (lane >> 3)) & 7) * 16);
#pragma unroll
  for (int i = 0; i < 16; ++i)
    __builtin_amdgcn_global_load_lds((const unsigned*)(src + i * 8 * KS), (lds_u32*)(l + i * 1024), 16, 0, 0);
}
__device__ __forceinline__ void load_w_raw(u32x4 (&r)[16], const Params& p, int d, int nt, int kc, int lane) {
  const auto src = mk_rsrc(p.w, (unsigned)KS * NTOT);
  const unsigned base = (unsigned)(kc * 128 + (lane >> 3) * 16) * NTOT + (d >> 1) * NS + nt * 128 + (lane & 7) * 16;
#pragma unroll
  for (int i = 0; i < 16; ++i) r[i] = __builtin_amdgcn_raw_buffer_load_b128(src, base + i * NTOT, 0, 0);
}
__device__ __forceinline__ void transpose_store(const u32x4 (&r)[16], uint8_t* l, int lane) {
#pragma unroll
  for (int d = 0; d < 4; ++d) {
    u32x4 o[4];
#pragma unroll
    for (int e = 0; e < 4; ++e) {
      const unsigned r0 = r[4 * e][d], r1 = r[4 * e + 1][d], r2 = r[4 * e + 2][d], r3 = r[4 * e + 3][d];
      const unsigned t0 = __builtin_amdgcn_perm(r1, r0, 0x05010400u);
      const unsigned t1 = __builtin_amdgcn_perm(r1, r0, 0x07030602u);
      const unsigned t2 = __builtin_amdgcn_perm(r3, r2, 0x05010400u);
      const unsigned t3 = __builtin_amdgcn_perm(r3, r2, 0x07030602u);
      o[0][e] = __builtin_amdgcn_perm(t2, t0, 0x05040100u);
      o[1][e] = __builtin_amdgcn_perm(t2, t0, 0x07060302u);
      o[2][e] = __builtin_amdgcn_perm(t3, t1, 0x05040100u);
      o[3][e] = __builtin_amdgcn_perm(t3, t1, 0x07060302u);
    }
#pragma unroll
    for (int q = 0; q < 4; ++q) *(u32x4*)(l + (4 * d + q) * 1024 + ((lane ^ ((q >> 1) * 8)) * 16)) = o[q];
  }
}

constexpr int NB = 16 - NF;
constexpr unsigned IT_B = 120, IT_D = IT_B + 64 + 8;
constexpr unsigned NSTAT = 8 * NPAIR;
struct Item { bool is_w; int idx, kc, src; };
__device__ __forceinline__ unsigned n_items(int rel) {
  return rel == 1 ? IT_D + 4u * NF * 6u : (rel == 0 ? IT_D : IT_B + 4u * NB + 8u);
}
__device__ __forceinline__ Item decode_item(unsigned it, int rel, int rank) {
  Item r;
  r.src = rank;
  const int d = rank ^ rel;
  const unsigned it_c = IT_B + (rel < 2 ? 64u : 4u * NB);
  if (it < IT_B) {
    const unsigned sl = it % 30u;
    r.kc = it / 30u; r.is_w = sl >= 14u; r.idx = r.is_w ? sl - 14u : (sl < 2u ? sl : sl + 2u);
  } else if (it < it_c) {
    unsigned i = it - IT_B;
    r.is_w = true;
    if (rel < 2 && i < 4u * NF) {
      r.kc = i / NF; r.idx = 32 - NF * ((d & 1) + 1) + i % NF;
    } else {
      if (rel < 2) i -= 4u * NF;
      const unsigned c = i % NB;
      r.kc = i / NB;
      r.idx = c >= NF ? 16 + (c - NF) : 32 - NF * (2 - (d & 1)) + c;
    }
  } else if (it < it_c + 8u) {
    const unsigned i = it - it_c;
    r.kc = i >> 1; r.is_w = false; r.idx = 2 + (i & 1u);
  } else {
    const unsigned i = it - IT_D;
    unsigned col, f;
    if (i < DSPLIT * NF * 6) { col = i % NF; f = i / NF; }
    else { constexpr unsigned R = DSPLIT < 4 ? (4 - DSPLIT) * 6 : 1; const unsigned i2 = i - DSPLIT * NF * 6; col = i2 / R; f = DSPLIT * 6 + i2 % R; }
    r.kc = f / 6; r.is_w = true;
    r.idx = 32 - NF * ((d & 1) + 1) + col;
    r.src = (d ^ (2 + f % 6)) & 7;
  }
  return r;
}
__device__ __forceinline__ unsigned unit_off(const Item& m) {
  return m.is_w ? X_BYTES + (unsigned)((m.src * 4 + m.kc) * 32 + m.idx) * UNIT : (unsigned)((m.src * 4 + m.kc) * 16 + m.idx) * UNIT;
}
template <bool LATE>
__device__ __forceinline__ unsigned push_loader(const Params& p, uint8_t* ring, volatile lds_u32* ctl, int rel, unsigned it,
                                                unsigned stop, const unsigned* ctr, unsigned call, int lane) {
  const int d = (p.rank ^ rel) & 7;
  const unsigned nit = n_items(rel);
  unsigned* head = (unsigned*)(p.ws[0] + OFF_HD) + rel * 32;
  unsigned* retired = (unsigned*)(p.ws[0] + OFF_TK) + 32;
  if (LATE) {
    arm_counter(head, call, lane);
    (void)wave_wait_flags(retired, lane == 0, (call - 1u) * NSTAT + it + 1u, (unsigned*)(p.ws[0] + OFF_TMO), call);
  }
#pragma unroll 1
  for (unsigned k = 0;; ++k) {
    uint8_t* l = ring + (k & 1) * UNIT;
    if (LATE || k > 0) {
      lds_wait_ge(ctl + 1, k);
      it = it >= stop ? nit : draw(head, lane) + NSTAT;
      if (!LATE && rel == 1 && it >= nit && lane == 0) (void)add_agent(retired);
    }
    if (it < nit) {
      const Item m = decode_item(it, rel, p.rank);
      if (m.src != p.rank) {
        if (!wave_wait_flags(wflag(p.ws[0], m.src, m.idx, m.kc), lane == 0, call, (unsigned*)(p.ws[0] + OFF_TMO), call)) it = nit;
        const uint8_t* src = p.ws[0] + OFF_DATA + (call & 1u) * PAR_BYTES + unit_off(m) + lane * 16;
#pragma unroll
        for (int j = 0; j < 16; ++j)
          __builtin_amdgcn_global_load_lds((const unsigned*)(src + j * 1024), (lds_u32*)(l + j * 1024), 16, 0, 17);
        WAIT_VM(0);
      } else if (m.is_w) {
        u32x4 r[16];
        load_w_raw(r, p, d, m.idx, m.kc, lane);
        if (!LATE && k == 0) __builtin_amdgcn_s_barrier();
        transpose_store(r, l, lane);
      } else {
        glds_x(p, l, x_row0(d, m.idx), m.kc, lane);
        if (!LATE && k == 0) __builtin_amdgcn_s_barrier();
        WAIT_VM(0);
      }
    }
    if (lane == 0) ctl[2 + (k & 1)] = it;
    WAIT_LGKM0();
    if (lane == 0) ctl[0] = k + 1;
    if (!LATE && k == 0) {
      call = RFL(ld_agent(ctr)) + 1u;
      arm_counter(head, call, lane);
    }
    if (it >= nit) return call;
  }
}
template <int WIN>
__device__ __forceinline__ void stream_item(const uint8_t* l, bool raw_x, __amdgpu_buffer_rsrc_t dst, unsigned off,
                                            unsigned* prev_flag, unsigned call, int lane,
                                            volatile lds_u32* req, unsigned req_val) {
  u32x4 v[16];
  unsigned a4[4];
#pragma unroll
  for (int q = 0; q < 4; ++q)
    a4[q] = raw_x ? (unsigned)((lane & 31) * 128 + (((q * 2 + (lane >> 5)) ^ lane) & 7) * 16) : (unsigned)(q * 1024 + lane * 16);
#pragma unroll
  for (int j = 0; j < 16; ++j) v[j] = *(const u32x4*)(l + (j >> 2) * 4096 + a4[j & 3]);
#pragma unroll
  for (int j = 0; j < 16; ++j) {
    __builtin_amdgcn_raw_buffer_store_b128(v[j], dst, off + j * 1024 + lane * 16, 0, AUX_SYS);
    WAIT_VM(WIN);
    if (j + 1 == WIN && lane == 0 && prev_flag) st_sys(prev_flag, call);
    if (j == (WIN > 3 ? 0 : 16 - FREE_AHEAD) && lane == 0) *req = req_val;
  }
}
template <int WA, int WB>
__device__ __forceinline__ void push_storer(const Params& p, const uint8_t* ring, volatile lds_u32* ctl, int rel,
                                            unsigned call, int lane) {
  const unsigned nit = n_items(rel);
  uint8_t* dws = p.ws[rel];
  const auto dst = mk_rsrc(dws + OFF_DATA + (call & 1u) * PAR_BYTES, PAR_BYTES);
  unsigned* pf = nullptr;
#pragma unroll 1
  for (unsigned k = 0;; ++k) {
    if (pf && RFL(ctl[0]) <= k) {
      WAIT_VM(0);
      if (lane == 0) st_sys(pf, call);
      pf = nullptr;
    }
    lds_wait_ge(ctl, k + 1);
    const unsigned it = RFL(ctl[2 + (k & 1)]);
    if (it >= nit) break;
    const Item m = decode_item(it, rel, p.rank);
    const uint8_t* l = ring + (k & 1) * UNIT;
    if (WA == WB || it + PUSH_LAST < nit) stream_item<WA>(l, !m.is_w, dst, unit_off(m), pf, call, lane, ctl + 1, k + 1);
    else stream_item<WB>(l, !m.is_w, dst, unit_off(m), pf, call, lane, ctl + 1, k + 1);
    if (it + 32u >= nit && lane == 0) (void)add_agent((unsigned*)(p.ws[0] + OFF_PD) + rel * 32);
    pf = m.is_w ? wflag(dws, m.src, m.idx, m.kc) : xflag(dws, m.src, m.idx, m.kc);
  }
  WAIT_VM(0);
  if (lane == 0 && pf) st_sys(pf, call);
}

__global__ void __launch_bounds__(NTHREADS, 2) gemm_rs_kernel(Params p) {
  __shared__ __attribute__((aligned(16))) uint8_t smem[LDS_BYTES];
  const int tid = threadIdx.x;
  const int lane = tid & 63;
  const int wave = RFL(tid >> 6);
  const int b = blockIdx.x;
  uint8_t* myws = p.ws[0];
  unsigned* tmo = (unsigned*)(myws + OFF_TMO);
  volatile lds_u32* lds_word = (volatile lds_u32*)(smem + LDS_MAIN);
  volatile lds_u32* lds_flags = lds_word + 64;
  unsigned* ctr = (unsigned*)(myws + OFF_CTR) + b;
  unsigned call;

  if (b < 64) {
    const int pr = wave & 3, pw = (b < 56 ? b : b - 56) * NPAIR + pr;
    const int rel = b < 56 ? 1 + pw % 7 : 0;
    const unsigned first = (unsigned)(b < 56 ? pw / 7 : pw);
    volatile lds_u32* ctl = lds_word + 32 + pr * 4;
    uint8_t* ring = smem + pr * 2 * UNIT;
    if (pr >= NPAIR) {
      __builtin_amdgcn_s_barrier();
      call = RFL(ld_agent(ctr)) + 1u;
    } else if (wave >= 4) {
      call = push_loader<false>(p, ring, ctl, rel, first, rel == 1 ? STOP1 : ~0u, ctr, 0u, lane);
    } else {
      const unsigned craw = ld_agent(ctr);
      if (lane < 4) ctl[lane] = 0u;
      RAW_BARRIER();
      call = RFL(craw) + 1u;
      if (b < 56) push_storer<PUSH_VM, PUSH_VM2>(p, ring, ctl, rel, call, lane);
      else push_storer<SELF_VM, SELF_VM>(p, ring, ctl, 0, call, lane);
    }
  } else {
    call = RFL(ld_agent(ctr)) + 1u;
  }

  const unsigned tm = (p.tilemap[b >> 2] >> ((b & 3) * 8)) & 255u;
  const int arel = tm >> 5, nt = tm & 31;
  const int par = p.rank & 1;
  const int wm = wave & 3, wn = wave >> 2;
  const float scale = p.sx[0] * p.sw[0];

  const uint8_t* data = myws + OFF_DATA + (call & 1u) * PAR_BYTES;
  const unsigned xa_off = (unsigned)((wm >> 1) * UNIT + (wm & 1) * 2 * 4096 + lane * 16);
  const unsigned wb_off = 2 * UNIT + (unsigned)(wn * 2 * 4096 + ((lane & 31) >> 3) * 1024 +
                                                ((((lane >> 5) ^ (lane >> 4)) & 1) * 8 + (lane & 7)) * 16);
  f32x16 acc[2][2];
#pragma unroll
  for (int i = 0; i < 2; ++i)
#pragma unroll
    for (int j = 0; j < 2; ++j)
#pragma unroll
      for (int r = 0; r < 16; ++r) acc[i][j][r] = 0.f;

  const bool far_w = (unsigned)(31 - par * NF - nt) < (unsigned)NF;
  auto step_qk = [&](int s) -> unsigned {
    const int f = s - 8, fk = (f * 43) >> 8;
    const int sr = far_w ? (s < 8 ? (s & 1) : 2 + f - fk * 6) : (s & 7);
    const int kc = far_w ? (s < 8 ? (s >> 1) : fk) : (s >> 3);
    return (unsigned)((p.rank ^ sr) * 4 + kc);
  };
  auto flag_ptr = [&](int g) -> const unsigned* {
    const bool isx = (lane & 31) < 8;
    const unsigned qk = step_qk(g * 4 + ((isx ? lane >> 1 : lane) & 3));
    const unsigned q = qk >> 2, kc = qk & 3u;
    return isx ? (const unsigned*)(myws + OFF_XF) + ((arel * 4 + kc) * 8 + q) * 2 + (lane & 1)
                     : (const unsigned*)(myws + OFF_WF) + (nt * 4 + kc) * 8 + q;
  };
  const uint8_t* xbase = data + (unsigned)(2 * arel) * UNIT + wave * 4096 + lane * 16;
  const uint8_t* wbase = data + X_BYTES + (unsigned)nt * UNIT + wave * 2048 + lane * 16;
  const unsigned qtab = step_qk(lane & 31);
  auto glds1 = [&](int s, unsigned ring, int j) {
    const unsigned qk = __builtin_amdgcn_readlane(qtab, s);
    uint8_t* l = smem + ring;
    if (j < 4) __builtin_amdgcn_global_load_lds((const unsigned*)(xbase + qk * 16 * UNIT + j * 1024), (lds_u32*)(l + (wave * 4 + j) * 1024), 16, 0, 0);
    else __builtin_amdgcn_global_load_lds((const unsigned*)(wbase + qk * 32 * UNIT + (j - 4) * 1024), (lds_u32*)(l + 2 * UNIT + (wave * 2 + j - 4) * 1024), 16, 0, 0);
  };
  if (wave == 0) {
    const bool good = wave_wait_flags(flag_ptr(lane >> 5), true, call, tmo, call);
    __builtin_amdgcn_fence(__ATOMIC_ACQUIRE, "");
    WAIT_VM(0);
    if (lane == 0) *lds_word = good ? 1u : 0u;
  }
  RAW_BARRIER();
  if (RFL(*lds_word) != 0u) {
    struct Frag { i32x4 r[8]; };
    auto rd = [&](Frag& f, unsigned ring, int kb, int i) {
      const unsigned o = (i < 2) ? wb_off + kb * 512 + i * 256
                       : (i < 6) ? xa_off + ((i - 2) >> 1) * 4096 + kb * 2048 + ((i - 2) & 1) * 1024
                                 : wb_off + 4096 + kb * 512 + (i - 6) * 256;
      f.r[i] = *(const i32x4*)(smem + ring + o);
    };
#define CAT(a, b) __builtin_shufflevector(a, b, 0, 1, 2, 3, 4, 5, 6, 7)
#define MA(f, i, j) acc[i][j] = __builtin_amdgcn_mfma_scale_f32_32x32x64_f8f6f4(                  \
      CAT(f.r[(i) * 6], f.r[(i) * 6 + 1]), CAT(f.r[2 + 2 * (j)], f.r[3 + 2 * (j)]), acc[i][j], 1, 1, 0, 0x7F7F7F7F, 0, 0x7F7F7F7F)
#pragma unroll
    for (int s = 0; s < 3; ++s)
#pragma unroll
      for (int j = 0; j < 6; ++j) glds1(s, s * STAGE_BYTES, j);
    WAIT_VM(12);
    RAW_BARRIER();
    Frag f0, f1;
#pragma unroll
    for (int i = 0; i < 8; ++i) rd(f0, 0, 0, i);
    __builtin_amdgcn_s_waitcnt(0xC07F);
    unsigned ring = 0, ringp = 2 * STAGE_BYTES;
#define STEP(WAITN, G2, G1) do {                                                         \
      const unsigned ring1 = ring == 2 * STAGE_BYTES ? 0u : ring + STAGE_BYTES;          \
      MA(f0, 0, 0); SCHED(); rd(f1, ring, 1, 0); rd(f1, ring, 1, 1); rd(f1, ring, 1, 2); rd(f1, ring, 1, 3); SCHED(); \
      MA(f0, 0, 1); SCHED(); rd(f1, ring, 1, 4); rd(f1, ring, 1, 5); rd(f1, ring, 1, 6); rd(f1, ring, 1, 7); SCHED(); \
      MA(f0, 1, 0); SCHED(); if (G2) { glds1(s + 2, ringp, 3); glds1(s + 2, ringp, 4); glds1(s + 2, ringp, 5); } SCHED(); \
      MA(f0, 1, 1); SCHED();                                                             \
      WAIT_VM(WAITN);                                                                    \
      RAW_BARRIER();            \
      MA(f1, 0, 0); SCHED(); rd(f0, ring1, 0, 0); rd(f0, ring1, 0, 1); rd(f0, ring1, 0, 2); rd(f0, ring1, 0, 3); SCHED(); \
      MA(f1, 0, 1); SCHED(); rd(f0, ring1, 0, 4); rd(f0, ring1, 0, 5); rd(f0, ring1, 0, 6); rd(f0, ring1, 0, 7); SCHED(); \
      MA(f1, 1, 0); SCHED(); if (G1) { glds1(s + 3, ring, 0); glds1(s + 3, ring, 1); glds1(s + 3, ring, 2); } SCHED(); \
      MA(f1, 1, 1);                                                                      \
      __builtin_amdgcn_s_waitcnt(0xC07F);                                                \
      SCHED();                                                                           \
      ringp = ring; ring = ring1;                                                        \
    } while (0)
    int s = 0;
    STEP(6, false, true);
    ++s;
#pragma unroll 1
    for (; s < 29; ++s) {
      if ((s & 3) == 3 && s < 27 && wave == 0)
        __builtin_amdgcn_global_load_lds(flag_ptr((s + 5) >> 2), (lds_u32*)lds_flags, 4, 0, 17);
      if ((s & 3) == 1 && s > 1) {
        if (!__all(lds_flags[lane] >= call)) {
          if (wave == 0) {
            const bool good = wave_wait_flags(flag_ptr((s + 3) >> 2), true, call, tmo, call);
            if (!good && lane == 0) *lds_word = 0u;
          }
          RAW_BARRIER();
        }
      }
      STEP(6, true, true);
    }
    STEP(6, true, false);
    ++s;
    STEP(0, false, false);
#pragma unroll
    for (int i = 0; i < 8; ++i) rd(f1, ring, 1, i);
    MA(f0, 0, 0); MA(f0, 0, 1); MA(f0, 1, 0); MA(f0, 1, 1);
    MA(f1, 0, 0); MA(f1, 0, 1); MA(f1, 1, 0); MA(f1, 1, 1);
  }
  RAW_BARRIER();

  {
    const bool bad = *lds_word == 0u;
    const float nanv = __builtin_nanf("");
#pragma unroll
    for (int i = 0; i < 2; ++i)
#pragma unroll
      for (int j = 0; j < 2; ++j) {
        const int row = wm * 64 + j * 32 + (lane & 31);
#pragma unroll
        for (int r = 0; r < 4; ++r) {
          f32x4 v;
#pragma unroll
          for (int t = 0; t < 4; ++t) {
            const float y = acc[i][j][r + 4 * t] * scale;
            v[t] = bad ? nanv : ((y <= 0.f) ? 0.f : y);
          }
          const int col = 16 * (r + 4 * (lane >> 5)) + 4 * (wn * 2 + i);
          *(f32x4*)(smem + row * EPI_PITCH + col * 4) = v;
        }
      }
  }
  RAW_BARRIER();
  const auto outr = mk_rsrc(p.out, 512u * NTOT * 4u);
  if (arel < 2) {
    const unsigned ob = (unsigned)((par ^ arel) * 256) * (NTOT * 4u) + (unsigned)((p.rank >> 1) * NS + nt * 128) * 4u + (lane & 31) * 16;
#pragma unroll 4
    for (int it = 0; it < 16; ++it) {
      const int row = wave * 32 + it * 2 + (lane >> 5);
      const u32x4 v = *(const u32x4*)(smem + row * EPI_PITCH + (lane & 31) * 16);
      __builtin_amdgcn_raw_buffer_store_b128(v, outr, ob + (unsigned)row * (NTOT * 4u), 0, AUX_OUT);
    }
    if (tid == 0) st_agent(ctr, call);
    if (b >= 56 && b < 64) {
      if (tid < 16) lds_word[32 + tid] = 0u;
      RAW_BARRIER();
      const int pr = wave & 3;
      if (pr < LATE_NPAIR) {
        volatile lds_u32* ctl = lds_word + 32 + pr * 4;
        uint8_t* ring = smem + pr * 2 * UNIT;
        if (wave >= 4) (void)push_loader<true>(p, ring, ctl, 1, (unsigned)((b - 56) * LATE_NPAIR + pr), ~0u, ctr, call, lane);
        else push_storer<LATE_VM, LATE_VM>(p, ring, ctl, 1, call, lane);
      }
    }
    return;
  }

  {
    uint8_t* seg = smem + wave * 2 * ITEM_BYTES;
    u32x4 v[16];
#pragma unroll
    for (int blk = 0; blk < 16; ++blk)
      v[blk] = *(const u32x4*)(seg + (blk * 2 + (lane >> 5)) * EPI_PITCH + (lane & 31) * 16);
    WAIT_LGKM0();
    SCHED();
    const auto stg = mk_rsrc(myws + OFF_STG + (unsigned)(arel * 32 + nt) * TILE_BYTES, TILE_BYTES);
#pragma unroll
    for (int hf = 0; hf < 2; ++hf) {
      uint8_t* it = seg + hf * ITEM_BYTES;
      const unsigned it0 = (unsigned)(wave * 2 + hf) * ITEM_BYTES;
      const unsigned scratch = it0 + lane * 4u;
      unsigned next = it0 + 256u;
      unsigned bits = 0;
#pragma unroll
      for (int e = 0; e < 32; ++e) {
        const unsigned val = v[hf * 8 + (e >> 2)][e & 3];
        const bool nz = val != 0u;
        const unsigned long long m = __ballot(nz);
        unsigned a = (__builtin_amdgcn_mbcnt_hi((unsigned)(m >> 32), __builtin_amdgcn_mbcnt_lo((unsigned)m, 0u)) << 2) + next;
        asm("" : "+v"(a));
        *(unsigned*)(smem + (nz ? a : scratch)) = val;
        next += (unsigned)__popcll(m) * 4u;
        asm("" : "+s"(next));
        bits |= nz ? (1u << e) : 0u;
      }
      const unsigned base = next - it0;
      *(unsigned*)(it + lane * 4) = bits;
      if (lane == 0) lds_word[16 + wave * 2 + hf] = (base - 256u) >> 2;
      const unsigned go = (unsigned)(wave * 2 + hf) * ITEM_BYTES;
      asm volatile("" ::: "memory");
#pragma unroll 3
      for (unsigned o = lane * 16u; o < base; o += 1024u)
        __builtin_amdgcn_raw_buffer_store_b128(*(const u32x4*)(it + o), stg, go + o, 0, AUX_SYS);
    }
    if (tid < 16) lds_word[32 + tid] = 0u;
  }
  const unsigned* gate = (const unsigned*)(myws + OFF_PD) + arel * 32;
  unsigned* tick = (unsigned*)(myws + OFF_TK) + arel * 32;
  unsigned t = 0;
  bool drawn = false;
  if (wave == 7 && RFL(ld_sys(gate)) >= (call - 1u) * 32u + GATE_N) { t = draw(tick, lane); drawn = true; }
  WAIT_VM(0);
  RAW_BARRIER();
  const unsigned tag = (call & 0x7FFFu) << 17;
  if (wave == 7) {
    if (!drawn) {
      (void)wave_wait_flags(gate, lane == 0, (call - 1u) * 32u + GATE_N, tmo, call);
      t = draw(tick, lane);
    }
    t &= 31u;
    if (lane < 16) st_sys((unsigned*)(p.ws[arel] + OFF_SF) + (arel * 32 + t) * 16 + lane, tag | ((unsigned)nt << 12) | lds_word[16 + lane]);
    if (lane == 0) st_agent(ctr, call);
  }
  if (b < 160) return;

  {
    const int eng = wave < 4 ? (wave & 1) : ((wave - 4) >> 1);
    const int g = (b - 160) * 2 + eng;
    const int rel = 2 + g % 6;
    const unsigned src = (unsigned)(p.rank ^ rel) & 7u;
    volatile lds_u32* c = lds_word + 32 + eng * 8;
    uint8_t* ring = smem + eng * 65536;
    constexpr unsigned END = 0xFFFFFFFFu;
    if (wave < 2) {
      const uint8_t* rstg = p.ws[rel] + OFF_STG + (unsigned)(rel * 32) * TILE_BYTES + lane * 16;
#pragma unroll 1
      for (unsigned k = 0;; ++k) {
        if (RFL(c[0]) <= k) {
          WAIT_VM(0);
          if (lane == 0) c[1] = k;
          lds_wait_ge(c, k + 1);
        }
        const unsigned d = RFL(c[4 + (k & 3)]);
        if (d == END) { WAIT_VM(0); if (lane == 0) c[1] = k; break; }
        const unsigned nbytes = 256u + (d & 0xFFFu) * 4u;
        const uint8_t* gsrc = rstg + ((d >> 12) & 511u) * ITEM_BYTES;
        uint8_t* l = ring + (k & 3) * ITEM_BYTES;
        const int nch = (int)((nbytes + 1023u) >> 10);
        const bool fast = (d >> 30) != 0u;
#pragma unroll 1
        for (int ch = 0; ch < nch; ++ch) {
          if ((unsigned)ch * 1024u + lane * 16u < nbytes)
            __builtin_amdgcn_global_load_lds((const unsigned*)(gsrc + ch * 1024), (lds_u32*)(l + ch * 1024), 16, 0, 17);
          if (ch == 0 || !fast) WAIT_VM(1); else WAIT_VM(PULL_VM2);
          if (ch == 0 && lane == 0) c[1] = k;
        }
      }
    } else if (wave < 4) {
      unsigned* head = (unsigned*)(myws + OFF_H2) + rel * 32;
      arm_counter(head, call, lane);
      const unsigned* q = (const unsigned*)(myws + OFF_SF) + rel * 512;
      (void)wave_wait_flags((const unsigned*)(myws + OFF_PD) + rel * 32, lane == 0, (call - 1u) * 32u + GATE_N2 + (unsigned)(g / 6) * GATE_RAMP / 32u, tmo, call);
      unsigned i = (unsigned)(g / 6);
#pragma unroll 1
      for (unsigned k = 0;; ++k) {
        for (;;) {
          const unsigned u0 = c[2], u1 = c[3], ld = c[1];
          if (k < RFL(min(min(u0, u1) + 4u, ld + PULL_AHEAD))) break;
          __builtin_amdgcn_s_sleep(1);
        }
        if (k > 0) i = draw(head, lane) + 32u;
        unsigned d = END;
        if (i < 512u) {
          const unsigned long long t0 = __builtin_amdgcn_s_memrealtime();
          for (unsigned spin = 1;; ++spin) {
            const unsigned w = RFL(ld_sys(q + i));
            if ((w & 0xFFFE0000u) == tag) {
              d = ((((w >> 12) & 31u) * 16u + (i & 15u)) << 12) | min(w & 0xFFFu, 2048u) | (i >= PULL_FAST ? 1u << 30 : 0u);
              break;
            }
            if (give_up(spin, t0, tmo, call)) {
              if (lane == 0) p.out[(p.rank >> 1) * NS] = __builtin_nanf("");
              break;
            }
            __builtin_amdgcn_s_sleep(1);
          }
        }
        if (lane == 0) c[4 + (k & 3)] = d;
        WAIT_LGKM0();
        if (lane == 0) c[0] = k + 1;
        if (d == END) break;
      }
    } else {
      const int u = wave & 1;
#pragma unroll 1
      for (unsigned k = 0;; ++k) {
        lds_wait_ge(c, k + 1);
        const unsigned d = RFL(c[4 + (k & 3)]);
        if (d == END) break;
        lds_wait_ge(c + 1, k + 1);
        const uint8_t* l = ring + (k & 3) * ITEM_BYTES;
        const unsigned bits = *(const unsigned*)(l + lane * 4);
        unsigned base = 256;
        if (u) {
          const unsigned below = __popc(bits & 0xFFFFu);
#pragma unroll
          for (int bt = 0; bt < 5; ++bt) base += (unsigned)__popcll(__ballot((below >> bt) & 1u)) << (bt + 2);
        }
        const unsigned mine = bits >> (16 * u);
        u32x4 v[4];
#pragma unroll
        for (int e = 0; e < 16; ++e) {
          const bool bit = (mine >> e) & 1u;
          const unsigned long long m = __ballot(bit);
          const unsigned pos = __builtin_amdgcn_mbcnt_hi((unsigned)(m >> 32), __builtin_amdgcn_mbcnt_lo((unsigned)m, 0u));
          const unsigned x = *(const unsigned*)(l + base + pos * 4u);
          v[e >> 2][e & 3] = bit ? x : 0u;
          base += (unsigned)__popcll(m) * 4u;
        }
        WAIT_LGKM0();
        if (lane == 0) c[2 + u] = k + 1;
        const unsigned tq = (d >> 12) & 511u;
        const unsigned ob = ((src & 1u) * 256u + (tq & 15u) * 16u + u * 8u + (lane >> 5)) * (NTOT * 4u) +
                            ((src >> 1) * NS + (tq >> 4) * 128u) * 4u + (lane & 31) * 16u;
#pragma unroll
        for (int bb = 0; bb < 4; ++bb)
          __builtin_amdgcn_raw_buffer_store_b128(v[bb], outr, ob + bb * 2u * (NTOT * 4u), 0, AUX_OUT);
      }
    }
  }
}

}

extern "C" void kernel_launch(int rank, int nranks, ncclComm_t comm,
                              hipStream_t stream,
                              void* const* d_in, const int* in_sizes, int n_in,
                              void* d_out, int out_size,
                              const int* in_dtypes, int out_dtype,
                              void* const* d_out_all,
                              void* const* d_ws_all, size_t ws_bytes) {
  (void)comm; (void)in_sizes; (void)in_dtypes; (void)out_dtype; (void)d_out_all;
  if (nranks != NR || (unsigned)rank >= (unsigned)NR || ws_bytes < WS_NEED || n_in < 4 || out_size != 512 * NTOT) {
    fprintf(stderr, "kernel_launch: unsupported configuration\n");
    return;
  }
  Params p;
  p.x = (const uint8_t*)d_in[0];
  p.w = (const uint8_t*)d_in[1];
  p.sx = (const float*)d_in[2];
  p.sw = (const float*)d_in[3];
  for (int i = 0; i < NR; ++i) p.ws[i] = (uint8_t*)d_ws_all[rank ^ i];
  p.out = (float*)d_out;
  p.rank = rank;
  {
    uint8_t map[NWG];
    auto put = [&](int ii, int x, int arel, int nt) { map[ii * 8 + x] = (uint8_t)(arel << 5 | nt); };
    for (int x = 0; x < 8; ++x)
      for (int j = 0; j < 12; ++j) put(20 + j, x, 2 + 3 * (x & 1) + j % 3, 4 * (x >> 1) + j / 3);
    for (int x = 0; x < 8; ++x) put(7, x, 0, 4 * (x >> 1) + (x & 1));
    int sidx = 0;
    for (int nt = 0; nt < 32; ++nt)
      for (int a = 0; a < 2; ++a) {
        if (a == 0 && nt < 16 && (nt & 3) < 2) continue;
        put(sidx % 7, sidx / 7, a, nt);
        ++sidx;
      }
    for (int x = 0; x < 8; ++x)
      for (int j = 0; j < 12; ++j) put(8 + j, x, 2 + 3 * (x & 1) + j % 3, 16 + 4 * (x >> 1) + j / 3);
    for (int i = 0; i < 64; ++i)
      p.tilemap[i] = map[4 * i] | map[4 * i + 1] << 8 | map[4 * i + 2] << 16 | (unsigned)map[4 * i + 3] << 24;
  }
  hipLaunchKernelGGL(gemm_rs_kernel, dim3(NWG), dim3(NTHREADS), 0, stream, p);
  HIP_CHECK(hipGetLastError());
}
```
